```python
import numpy as np

kernel_name = "gpu_gemm_ar_m4096_k4096_n16384_e5m2e5m2_dqrelu"


ABSMAX_RTOL = 0.05

COMPUTE_DTYPE = "float32"
EPILOGUE = "dequant_relu"
IN_QUANT = None
OUT_QUANT = None
IN_FP8 = "e5m2e5m2"
IN_DTYPES = {"x": "e5m2", "w_mat": "e5m2",
             "scale_x": "float32", "scale_w": "float32"}
WORLD_SIZE = 8

def _snap_fp8(x, mant_bits, max_abs, min_normal):
    a = np.abs(x.astype(np.float64))
    _m, _e = np.frexp(a)
    q = np.where(a >= min_normal, np.ldexp(1.0, _e - 1 - mant_bits),
                 min_normal * 2.0 ** -mant_bits)
    snapped = np.minimum(np.round(a / q) * q, max_abs)
    return (np.sign(x) * snapped).astype(np.float32)

def setup_inputs():

    x = _snap_fp8(np.random.randn(4096, 4096).astype(np.float32), 2, 57344.0, 2.0 ** -14)
    w_mat = _snap_fp8(np.random.randn(4096, 16384).astype(np.float32), 2, 57344.0, 2.0 ** -14)
    scale_x = (0.02 * (1.0 + np.random.rand(1))).astype(np.float32)
    scale_w = (0.02 * (1.0 + np.random.rand(1))).astype(np.float32)
    return {"x": x, "w_mat": w_mat, "scale_x": scale_x, "scale_w": scale_w}

def reference(x, w_mat, scale_x, scale_w):
    acc = (x @ w_mat).astype(np.float32)
    y = (acc * (scale_x * scale_w)).astype(np.float32)
    return np.maximum(y, 0.0).astype(np.float32)
```

```cpp
#include <hip/hip_runtime.h>
#include <rccl/rccl.h>
#include <cstdio>

#define HIP_CHECK(expr) do { hipError_t _e = (expr); if (_e != hipSuccess) { \
    fprintf(stderr, "HIP error %s at %s:%d\n", hipGetErrorString(_e), __FILE__, __LINE__); \
    return; } } while (0)

#define LAS __attribute__((address_space(3)))
#define GAS __attribute__((address_space(1)))
typedef unsigned char u8;
typedef unsigned short u16;
typedef int v8i __attribute__((ext_vector_type(8)));
typedef int v4i __attribute__((ext_vector_type(4)));
typedef unsigned v4u __attribute__((ext_vector_type(4)));
typedef float v4f __attribute__((ext_vector_type(4)));

constexpr int M = 4096, N = 16384, K = 4096, NR = 8, KS = K / NR;
constexpr int NKT = 32;
constexpr int BLK = 32768, HT = 16384;
constexpr int NPA = 16, NPB = 64, NPANEL = NPA + NPB;
constexpr unsigned PANEL_BYTES = NKT * BLK;
constexpr unsigned STG_BYTES = NPANEL * PANEL_BYTES;
constexpr int GRID = 256, NCOMM = 16, NGEMM = GRID - NCOMM, CW = 2, NCWAVES = NCOMM * CW;
constexpr int MAXT = 5;
constexpr int NPIECE = NPANEL * 4, NUNIT = NPIECE * 8;
constexpr size_t WS_CNT = 0;
constexpr size_t WS_TMO = GRID * 128;
constexpr size_t WS_FLAG = 65536;
constexpr size_t WS_DBG = 131072;
constexpr size_t WS_STG = 1u << 20;
constexpr size_t WS_TOTAL = WS_STG + 2 * (size_t)STG_BYTES;
constexpr unsigned long long DEADLINE = 50000000ull;
constexpr int LDS_BYTES = 8 * HT + 64;

struct Params {
  const u8* x; const u8* w; const float* sx; const float* sw; float* out;
  u8* ws[NR];
  int rank; int pad;
  unsigned tile[NGEMM][(MAXT + 1) / 2];
  unsigned stream[NPIECE / 2];
};
__device__ __forceinline__ int half_of(const unsigned* t, int i) { return (int)((t[i >> 1] >> ((i & 1) * 16)) & 0xFFFFu); }

__device__ __forceinline__ v8i ldfrag(LAS u8* p) {
  v4i lo = *(const LAS v4i*)p, hi = *(const LAS v4i*)(p + 1024);
  return (v8i){lo[0], lo[1], lo[2], lo[3], hi[0], hi[1], hi[2], hi[3]};
}
#define WAIT_V(n) asm volatile("s_waitcnt vmcnt(" #n ")" ::: "memory")
#define WAIT_L(n) asm volatile("s_waitcnt lgkmcnt(" #n ")" ::: "memory")
#define BAR __builtin_amdgcn_s_barrier()
#define SCHED __builtin_amdgcn_sched_barrier(0)

struct Raw { unsigned r[16]; };

__device__ __forceinline__ void comm_load(const Params& p, int n, int lane, Raw& d) {
  const int piece = half_of(p.stream, n >> 3), u = n & 7, panel = piece >> 2, q = piece & 3;
  const int g = lane >> 4, c = lane & 15;
  const bool isx = panel < NPA;
  const __amdgpu_buffer_rsrc_t rs = __builtin_amdgcn_make_buffer_rsrc((void*)(isx ? p.x : p.w), 0, isx ? M * KS : KS * N, 0x00020000);
  const unsigned vo = isx ? (unsigned)(c * KS + g * 16) : (unsigned)(g * 16 * N + 4 * c);
  const unsigned so = isx ? (unsigned)((panel * 256 + u * 32) * KS + q * 128) : (unsigned)((q * 128 + (u & 1) * 64) * N + (panel - NPA) * 256 + (u >> 1) * 64);
  const unsigned sa = isx ? 4u : (unsigned)N, sb = isx ? 64u : 4u * N, sc = isx ? 16u * KS : 8u * N;
#pragma unroll
  for (int j = 0; j < 16; ++j) d.r[j] = __builtin_amdgcn_raw_buffer_load_b32(rs, vo, so + (j & 3) * sa + ((j >> 2) & 1) * sb + (j >> 3) * sc, 0);
}

__device__ __forceinline__ void comm_role(const Params& p, unsigned call, int wv, int lane) {
  const unsigned par = call & 1u;
  __amdgpu_buffer_rsrc_t rs[NR];
#pragma unroll
  for (int i = 0; i < NR; ++i) {
    const int d = (p.rank + 1 + i) & 7; const unsigned rot = (unsigned)((p.rank - d) & 7) * BLK;
    rs[i] = __builtin_amdgcn_make_buffer_rsrc(p.ws[d] + WS_STG + (size_t)par * STG_BYTES + rot, 0, STG_BYTES - rot, 0x00020000);
  }
  u8* fl = p.ws[lane & 7] + WS_FLAG + (unsigned)((p.rank - lane) & 7) * 4;
  Raw raw;
  int n = wv;
  comm_load(p, n, lane, raw);
  __builtin_amdgcn_s_waitcnt(0x0F70);
  int prev = -1;
  for (; n < NUNIT; n += NCWAVES) {
    const int piece = half_of(p.stream, n >> 3), u = n & 7, panel = piece >> 2, q = piece & 3;
    v4u o[4]; unsigned off = (unsigned)panel * PANEL_BYTES + (unsigned)q * 8u * BLK + (unsigned)lane * 16u, s1, s2;
    if (panel < NPA) {
#pragma unroll
      for (int i = 0; i < 4; ++i) o[i] = (v4u){raw.r[i * 4], raw.r[i * 4 + 1], raw.r[i * 4 + 2], raw.r[i * 4 + 3]};
      off += u * 4096; s1 = 1024; s2 = 2048;
    } else {
#pragma unroll
      for (int d = 0; d < 4; ++d) {
        const unsigned r0 = raw.r[4 * d], r1 = raw.r[4 * d + 1], r2 = raw.r[4 * d + 2], r3 = raw.r[4 * d + 3];
        const unsigned t0 = __builtin_amdgcn_perm(r1, r0, 0x05010400u), t1 = __builtin_amdgcn_perm(r1, r0, 0x07030602u);
        const unsigned t2 = __builtin_amdgcn_perm(r3, r2, 0x05010400u), t3 = __builtin_amdgcn_perm(r3, r2, 0x07030602u);
        o[0][d] = __builtin_amdgcn_perm(t2, t0, 0x05040100u); o[1][d] = __builtin_amdgcn_perm(t2, t0, 0x07060302u);
        o[2][d] = __builtin_amdgcn_perm(t3, t1, 0x05040100u); o[3][d] = __builtin_amdgcn_perm(t3, t1, 0x07060302u);
      }
      off += (u >> 1) * 4096 + (u & 1) * 1024; s1 = 2048; s2 = 16384;
    }
    comm_load(p, n + NCWAVES < NUNIT ? n + NCWAVES : n, lane, raw);
    asm volatile("" ::: "memory");
#pragma unroll
    for (int i = 0; i < NR; ++i) {
      __builtin_amdgcn_raw_buffer_store_b128(o[0], rs[i], off, 0, 17);
      __builtin_amdgcn_raw_buffer_store_b128(o[1], rs[i], off, s1, 17);
      __builtin_amdgcn_raw_buffer_store_b128(o[2], rs[i], off, s2, 17);
      __builtin_amdgcn_raw_buffer_store_b128(o[3], rs[i], off, s1 + s2, 17);
    }
    asm volatile("" ::: "memory");
    __builtin_amdgcn_s_waitcnt(0x8F70);
    WAIT_V(32);
    if (prev >= 0 && lane < NR) __hip_atomic_fetch_add((unsigned*)(fl + prev), 1u, __ATOMIC_RELAXED, __HIP_MEMORY_SCOPE_SYSTEM);
    prev = panel * 128 + q * 32;
  }
  WAIT_V(0);
  if (prev >= 0 && lane < NR) __hip_atomic_fetch_add((unsigned*)(fl + prev), 1u, __ATOMIC_RELAXED, __HIP_MEMORY_SCOPE_SYSTEM);
}

__global__ __launch_bounds__(512, 2) void fused_kernel(const Params p) {
  __shared__ __attribute__((aligned(16))) u8 shm[LDS_BYTES];
  LAS u8* lds = (LAS u8*)shm;
  const int tid = threadIdx.x, wid = __builtin_amdgcn_readfirstlane(tid >> 6), lane = tid & 63;
  u8* const myws = p.ws[p.rank];
  unsigned* cnt = (unsigned*)(myws + WS_CNT + blockIdx.x * 128);
  const unsigned call = (unsigned)__builtin_amdgcn_readfirstlane(__hip_atomic_load(cnt, __ATOMIC_RELAXED, __HIP_MEMORY_SCOPE_AGENT)) + 1u;
  if (wid == 0) __builtin_amdgcn_fence(__ATOMIC_ACQUIRE, "");
  if (tid == 0) *(LAS float*)(lds + 8 * HT) = p.sx[0] * p.sw[0];
  asm volatile("s_waitcnt vmcnt(0) lgkmcnt(0)" ::: "memory");
  BAR;
  if (tid == 0) __hip_atomic_store(cnt, call, __ATOMIC_RELAXED, __HIP_MEMORY_SCOPE_AGENT);
  if (blockIdx.x < NCOMM) {
    if (wid < CW) comm_role(p, call, wid * NCOMM + blockIdx.x, lane);
    return;
  }
  const int gw = blockIdx.x - NCOMM;
  const int wr = wid >> 2, wc = wid & 3;
  const unsigned voff = (unsigned)(wid * 2048 + lane * 16);
  const unsigned ldsw = (unsigned)wid * 2048u;
  const int aoff = wr * 8192 + lane * 16, boff = wc * 4096 + lane * 16;
  const u8* const stg = myws + WS_STG + (size_t)(call & 1u) * STG_BYTES;
  const unsigned need = call * 8u;
  unsigned* const tmo = (unsigned*)(myws + WS_TMO);
#define SA(b, h) (((b) * 2 + (h)) * HT)
#define SB(b, h) ((4 + (b) * 2 + (h)) * HT)
#define STAGE(bufoff, gbase) do { \
    __builtin_amdgcn_global_load_lds((const unsigned*)((gbase) + voff), (LAS unsigned*)(lds + (bufoff) + ldsw), 16, 0, 0); \
    __builtin_amdgcn_global_load_lds((const unsigned*)((gbase) + voff), (LAS unsigned*)(lds + (bufoff) + ldsw), 16, 1024, 0); } while (0)
#define LDA(dst, b, h) do { _Pragma("unroll") for (int m = 0; m < 4; ++m) dst[m] = ldfrag(lds + SA(b, h) + aoff + m * 2048); } while (0)
#define LDB(dst, b, h) do { _Pragma("unroll") for (int n = 0; n < 2; ++n) dst[n] = ldfrag(lds + SB(b, h) + boff + n * 2048); } while (0)
#define MFMA1(c, a, b) asm volatile("v_mfma_f32_16x16x128_f8f6f4 %0, %1, %2, %0 cbsz:1 blgp:1" : "+v"(c) : "v"(a), "v"(b))
#define MMA(ai, bj, At, Bt) do { __builtin_amdgcn_s_setprio(1); _Pragma("unroll") for (int m = 0; m < 4; ++m) _Pragma("unroll") for (int n = 0; n < 2; ++n) \
    MFMA1(acc[ai][bj][m][n], At[m], Bt[n]); __builtin_amdgcn_s_setprio(0); } while (0)
#define FLAGP(t) ((const unsigned*)(myws + WS_FLAG + ((lane < 32) ? ((t) & 15) : NPA + ((t) >> 4)) * 128 + (lane & 31) * 4))
#define MASK_OF(v) ({ const unsigned long long _b = __builtin_amdgcn_ballot_w64((v) >= need); (unsigned)_b & (unsigned)(_b >> 32); })
#define ENSURE(t, bits) do { if ((rdy & (bits)) != (bits)) { \
      const unsigned long long _t0 = __builtin_amdgcn_s_memrealtime(); const unsigned* _fp = FLAGP(t); \
      for (;;) { const unsigned _v = __hip_atomic_load(_fp, __ATOMIC_RELAXED, __HIP_MEMORY_SCOPE_SYSTEM); rdy = MASK_OF(_v); \
        if ((rdy & (bits)) == (bits)) break; \
        if (__builtin_amdgcn_readfirstlane(__hip_atomic_load(tmo, __ATOMIC_RELAXED, __HIP_MEMORY_SCOPE_AGENT)) == call || __builtin_amdgcn_s_memrealtime() - _t0 > DEADLINE) { \
          __hip_atomic_store(tmo, call, __ATOMIC_RELAXED, __HIP_MEMORY_SCOPE_AGENT); *(LAS float*)(lds + 8 * HT) = __builtin_nanf(""); rdy = 0xFFFFFFFFu; break; } \
        __builtin_amdgcn_s_sleep(4); } \
      __builtin_amdgcn_s_waitcnt(0xC07F); } } while (0)

  int cur = half_of(p.tile[gw], 0);
  unsigned rdy = 0;
  v4f acc[2][2][4][2];
#define ZERO_ACC() do { _Pragma("unroll") for (int a = 0; a < 2; ++a) _Pragma("unroll") for (int b = 0; b < 2; ++b) _Pragma("unroll") for (int m = 0; m < 4; ++m) _Pragma("unroll") for (int n = 0; n < 2; ++n) acc[a][b][m][n] = (v4f){0.f, 0.f, 0.f, 0.f}; } while (0)
  ZERO_ACC();
  v8i At[4], B0[2], B1[2];
  const u8* cA = stg + (size_t)(cur & 15) * PANEL_BYTES;
  const u8* cB = stg + (size_t)(NPA + (cur >> 4)) * PANEL_BYTES;
  if (wid == 0) ENSURE(cur, 3u);
  asm volatile("" ::: "memory"); BAR; asm volatile("" ::: "memory");
  STAGE(SB(0, 0), cB); STAGE(SA(0, 0), cA); STAGE(SB(0, 1), cB + HT); STAGE(SA(0, 1), cA + HT);
  if (wr == 1) BAR;
  WAIT_V(4); BAR;
  STAGE(SB(1, 0), cB + BLK); STAGE(SA(1, 0), cA + BLK); STAGE(SB(1, 1), cB + BLK + HT);
  WAIT_V(6); BAR;
  for (int r = 0;; ++r) {
    const int nx = half_of(p.tile[gw], r + 1);
    const bool has_next = nx != 0xFFFF;
    const int nxt = has_next ? nx : cur;
    const u8* nA = stg + (size_t)(nxt & 15) * PANEL_BYTES;
    const u8* nB = stg + (size_t)(NPA + (nxt >> 4)) * PANEL_BYTES;
    for (int t = 0; t < NKT; t += 2) {
      const bool last = (t == NKT - 2);
      const u8* a1 = cA + (size_t)(t + 1) * BLK;
      const u8* a2 = last ? nA : cA + (size_t)(t + 2) * BLK;
      const u8* b2 = last ? nB : cB + (size_t)(t + 2) * BLK;
      const u8* a3 = a2 + BLK; const u8* b3 = b2 + BLK;
      if (wid == 0) {
        if (last) rdy = 0;
        const int tt = last ? nxt : cur; const unsigned bits = last ? 3u : (3u << (t + 2));
        ENSURE(tt, bits);
      }
      LDB(B0, 0, 0); SCHED; LDA(At, 0, 0); STAGE(SA(1, 1), a1 + HT);
      WAIT_L(8); BAR; WAIT_L(0); MMA(0, 0, At, B0); BAR; SCHED;
      LDB(B1, 0, 1); STAGE(SB(0, 0), b2);
      BAR; WAIT_L(0); MMA(0, 1, At, B1); BAR;
      LDA(At, 0, 1); STAGE(SA(0, 0), a2);
      BAR; WAIT_L(0); MMA(1, 0, At, B0); BAR; SCHED;
      STAGE(SB(0, 1), b2 + HT);
      WAIT_V(6); BAR; MMA(1, 1, At, B1); BAR;
      LDB(B0, 1, 0); SCHED; LDA(At, 1, 0); STAGE(SA(0, 1), a2 + HT);
      WAIT_L(8); BAR; WAIT_L(0); MMA(0, 0, At, B0); BAR; SCHED;
      LDB(B1, 1, 1); STAGE(SB(1, 0), b3);
      BAR; WAIT_L(0); MMA(0, 1, At, B1); BAR;
      LDA(At, 1, 1); STAGE(SA(1, 0), a3);
      BAR; WAIT_L(0); MMA(1, 0, At, B0); BAR; SCHED;
      STAGE(SB(1, 1), b3 + HT);
      WAIT_V(6); BAR; MMA(1, 1, At, B1); BAR;
    }
    asm volatile("s_nop 15\n\ts_nop 7" ::: "memory"); SCHED;
    {
      int tz = threadIdx.x; asm volatile("" : "+v"(tz));
      const int l = tz & 63, fq = l >> 4, c = l & 15;
      const float scale = *(LAS float*)(lds + 8 * HT);
      float* base = p.out + (size_t)((cur & 15) * 256 + wr * 64 + 4 * fq) * N + (cur >> 4) * 256 + wc * 64 + 4 * c;
#pragma unroll
      for (int ai = 0; ai < 2; ++ai)
#pragma unroll
        for (int m = 0; m < 4; ++m)
#pragma unroll
          for (int j = 0; j < 4; ++j) {
            v4f o;
            o[0] = __builtin_elementwise_maximum(acc[ai][0][m][0][j] * scale, 0.f); o[1] = __builtin_elementwise_maximum(acc[ai][0][m][1][j] * scale, 0.f);
            o[2] = __builtin_elementwise_maximum(acc[ai][1][m][0][j] * scale, 0.f); o[3] = __builtin_elementwise_maximum(acc[ai][1][m][1][j] * scale, 0.f);
            __builtin_nontemporal_store(o, (v4f*)(base + (size_t)(ai * 128 + m * 16 + j) * N));
          }
    }
    if (!has_next) break;
    ZERO_ACC();
    cur = nxt; cA = nA; cB = nB;
  }
  WAIT_V(0);
  if (wr == 0) BAR;
}

static void build_schedule(Params& p) {
  for (int b = NCOMM; b < GRID; ++b) {
    const int x = b & 7, c = (b >> 3) - NCOMM / 8, xr = x & 3, xc = x >> 2, W = NGEMM / 8;
    auto tile_of = [&](int o) {
      int ri, ci;
      if (o < 16) { ri = (o >> 3) * 2 + (o & 1); ci = (o & 7) >> 1; } else { ri = o & 3; ci = o >> 2; }
      return (u16)((xr + 4 * ri) | ((ci * 2 + xc) << 4));
    };
    int k = 0; u16* t = (u16*)p.tile[b - NCOMM];
    const int extra = 128 - 4 * W;
    t[k++] = tile_of(c);
    if (c < extra) t[k++] = tile_of(W + c);
    for (int r = 0; r < 3; ++r) t[k++] = tile_of(W + extra + r * W + c);
    while (k <= MAXT) t[k++] = 0xFFFF;
  }
  int s = 0;
  auto put = [&](int panel, int q) { ((u16*)p.stream)[s++] = (u16)(panel << 2 | q); };
  for (int q = 0; q < 4; ++q) for (int i = 0; i < 8; ++i) { put(i, q); put(NPA + i, q); }
  for (int q = 0; q < 4; ++q) { for (int i = 8; i < 16; ++i) put(i, q); for (int i = 8; i < 20; ++i) put(NPA + i, q); }
  const int lim[4] = {20, 34, 50, 64};
  for (int r = 0; r < 3; ++r) for (int q = 0; q < 4; ++q) for (int i = lim[r]; i < lim[r + 1]; ++i) put(NPA + i, q);
}

extern "C" void kernel_launch(int rank, int nranks, ncclComm_t comm, hipStream_t stream,
                              void* const* d_in, const int* in_sizes, int n_in,
                              void* d_out, int out_size, const int* in_dtypes, int out_dtype,
                              void* const* d_out_all, void* const* d_ws_all, size_t ws_bytes) {
  if (nranks != NR || n_in != 4 || ws_bytes < WS_TOTAL || (unsigned)rank >= (unsigned)NR ||
      in_sizes[0] != M * KS || in_sizes[1] != KS * N || out_size != M * N) { fprintf(stderr, "unexpected problem shape\n"); return; }
  Params p;
  p.x = (const u8*)d_in[0]; p.w = (const u8*)d_in[1]; p.sx = (const float*)d_in[2]; p.sw = (const float*)d_in[3];
  p.out = (float*)d_out; p.rank = rank; p.pad = 0;
  for (int i = 0; i < NR; ++i) p.ws[i] = (u8*)d_ws_all[i];
  build_schedule(p);
  hipLaunchKernelGGL(fused_kernel, dim3(GRID), dim3(512), 0, stream, p);
  HIP_CHECK(hipGetLastError());
}
```

```cpp
#include <hip/hip_runtime.h>
#include <rccl/rccl.h>
#include <cstdio>

#define HIP_CHECK(expr) do { hipError_t _e = (expr); if (_e != hipSuccess) { \
    fprintf(stderr, "HIP error %s at %s:%d\n", hipGetErrorString(_e), __FILE__, __LINE__); \
    return; } } while (0)

#define LAS __attribute__((address_space(3)))
typedef unsigned char u8;
typedef unsigned short u16;
typedef int v8i __attribute__((ext_vector_type(8)));
typedef int v4i __attribute__((ext_vector_type(4)));
typedef unsigned v4u __attribute__((ext_vector_type(4)));
typedef float v4f __attribute__((ext_vector_type(4)));
typedef __amdgpu_buffer_rsrc_t rsrc_t;

#define XV 21
#define JS 6
#define LAUX 0
constexpr int M = 4096, N = 16384, K = 4096, NR = 8, KS = K / NR;
constexpr int NKT = 32;
constexpr int BLK = 32768;
constexpr int NPA = 16, NPB = 64, NPANEL = NPA + NPB;
constexpr unsigned Q_BYTES = NPANEL * BLK;
constexpr unsigned SLOT_BYTES = 4 * Q_BYTES;
constexpr unsigned STG_BYTES = NR * SLOT_BYTES;
constexpr int GRID = 256, NCOMM = 16, NGEMM = GRID - NCOMM, NCWAVES = NCOMM * 4, NGWAVES = NGEMM * 4;
constexpr int MAXT = 5;
constexpr int NPIECE = NPANEL * 4, NUNIT = NPIECE * 8;
constexpr size_t WS_CNT = 0;
constexpr size_t WS_TMO = GRID * 128;
constexpr size_t WS_TICKET = WS_TMO + 128;
constexpr size_t WS_FLAG = 65536;
constexpr size_t WS_DBG = 131072;
constexpr size_t WS_STG = 1u << 20;
constexpr size_t WS_TOTAL = WS_STG + 2 * (size_t)STG_BYTES;
constexpr unsigned long long DEADLINE = 50000000ull;
constexpr int LDS_A = 0, LDS_B = 2 * BLK, LDS_X = 4 * BLK, LDS_BYTES = LDS_X + 64;

struct Params {
  const u8* x; const u8* w; const float* sx; const float* sw; float* out;
  u8* ws[NR];
  int rank; int pad;
  unsigned tile[NGEMM][(MAXT + 1) / 2];
  unsigned stream[NPIECE / 2];
};
__device__ __forceinline__ int half_of(const unsigned* t, int i) { return (int)((t[i >> 1] >> ((i & 1) * 16)) & 0xFFFFu); }

__device__ __forceinline__ v8i ldfrag(LAS u8* p, int sw) {
  v4i lo = *(const LAS v4i*)(p + sw * 1024), hi = *(const LAS v4i*)(p + 1024 - sw * 1024);
  return (v8i){lo[0], lo[1], lo[2], lo[3], hi[0], hi[1], hi[2], hi[3]};
}
__device__ __forceinline__ void dma16(rsrc_t rs, LAS u8* l, int vo, unsigned so, int k) {
  switch (k) {
    case 0: __builtin_amdgcn_raw_ptr_buffer_load_lds(rs, (LAS void*)l, 16, vo, so, 0, LAUX); break;
    case 1: __builtin_amdgcn_raw_ptr_buffer_load_lds(rs, (LAS void*)l, 16, vo, so, 1024, LAUX); break;
    case 2: __builtin_amdgcn_raw_ptr_buffer_load_lds(rs, (LAS void*)l, 16, vo, so, 2048, LAUX); break;
    default: __builtin_amdgcn_raw_ptr_buffer_load_lds(rs, (LAS void*)l, 16, vo, so, 3072, LAUX); break;
  }
}
#define STR2(x) #x
#define STR(x) STR2(x)
#define WAIT_V(n) asm volatile("s_waitcnt vmcnt(" STR(n) ")" ::: "memory")
#define WAIT_L(n) asm volatile("s_waitcnt lgkmcnt(" STR(n) ")" ::: "memory")
#define BAR __builtin_amdgcn_s_barrier()
#define FBAR do { asm volatile("" ::: "memory"); __builtin_amdgcn_s_barrier(); asm volatile("" ::: "memory"); } while (0)
#define SCHED __builtin_amdgcn_sched_barrier(0)
#define MFA(c, a, b) asm volatile("v_mfma_f32_16x16x128_f8f6f4 %0, %1, %2, %0 cbsz:1 blgp:1" : "+a"(c) : "v"(a), "v"(b))
#define MFA0(c, a, b) asm volatile("v_mfma_f32_16x16x128_f8f6f4 %0, %1, %2, 0 cbsz:1 blgp:1" : "=a"(c) : "v"(a), "v"(b))
#define STAMP() do {} while (0)

struct Raw { unsigned r[16]; };
struct Unit { v4u o[4]; unsigned off, so[4], flag; };

__device__ __forceinline__ void unit_load(const Params& p, int n, int lane, Raw& d) {
  const int piece = half_of(p.stream, n >> 3), u = n & 7, panel = piece >> 2, q = piece & 3;
  const int g = lane >> 4, c = lane & 15;
  const bool isx = panel < NPA;
  const rsrc_t rs = __builtin_amdgcn_make_buffer_rsrc((void*)(isx ? p.x : p.w), 0, isx ? M * KS : KS * N, 0x00020000);
  const unsigned vo = isx ? (unsigned)(c * KS + g * 16) : (unsigned)(g * 16 * N + 4 * c);
  const unsigned so = isx ? (unsigned)((panel * 256 + u * 32) * KS + q * 128) : (unsigned)((q * 128 + (u & 1) * 64) * N + (panel - NPA) * 256 + (u >> 1) * 64);
  const unsigned sa = isx ? 4u : (unsigned)N, sb = isx ? 64u : 4u * N, sc = isx ? 16u * KS : 8u * N;
#pragma unroll
  for (int j = 0; j < 16; ++j) d.r[j] = __builtin_amdgcn_raw_buffer_load_b32(rs, vo, so + (j & 3) * sa + ((j >> 2) & 1) * sb + (j >> 3) * sc, 0);
}
__device__ __forceinline__ void unit_pack(const Params& p, int n, int lane, const Raw& raw, Unit& t) {
  const int piece = half_of(p.stream, n >> 3), u = n & 7, panel = piece >> 2, q = piece & 3;
  t.off = (unsigned)q * Q_BYTES + (unsigned)panel * BLK + (unsigned)lane * 16u;
  t.flag = (unsigned)(panel * 128 + q * 32);
  t.so[0] = 0;
  if (panel < NPA) {
#pragma unroll
    for (int i = 0; i < 4; ++i) t.o[i] = (v4u){raw.r[i * 4], raw.r[i * 4 + 1], raw.r[i * 4 + 2], raw.r[i * 4 + 3]};
    t.off += u * 4096; t.so[1] = 1024; t.so[2] = 2048; t.so[3] = 3072;
  } else {
#pragma unroll
    for (int d = 0; d < 4; ++d) {
      const unsigned r0 = raw.r[4 * d], r1 = raw.r[4 * d + 1], r2 = raw.r[4 * d + 2], r3 = raw.r[4 * d + 3];
      const unsigned t0 = __builtin_amdgcn_perm(r1, r0, 0x05010400u), t1 = __builtin_amdgcn_perm(r1, r0, 0x07030602u);
      const unsigned t2 = __builtin_amdgcn_perm(r3, r2, 0x05010400u), t3 = __builtin_amdgcn_perm(r3, r2, 0x07030602u);
      t.o[0][d] = __builtin_amdgcn_perm(t2, t0, 0x05040100u); t.o[1][d] = __builtin_amdgcn_perm(t2, t0, 0x07060302u);
      t.o[2][d] = __builtin_amdgcn_perm(t3, t1, 0x05040100u); t.o[3][d] = __builtin_amdgcn_perm(t3, t1, 0x07060302u);
    }
    t.off += (u >> 1) * 8192 + (u & 1) * 1024; t.so[1] = 2048; t.so[2] = (u & 1) ? 4096 - 1024 : 4096 + 1024; t.so[3] = t.so[2] + 2048;
  }
}

__global__ __launch_bounds__(256, 1) void fused_kernel(const Params p) {
  __shared__ __attribute__((aligned(16))) u8 shm[LDS_BYTES];
  LAS u8* lds = (LAS u8*)shm;
  const int tid = threadIdx.x, wid = __builtin_amdgcn_readfirstlane(tid >> 6), lane = tid & 63;
  u8* const myws = p.ws[p.rank];
  STAMP();
  unsigned* const cnt = (unsigned*)(myws + WS_CNT + blockIdx.x * 128);
  if (blockIdx.x < NCOMM) {
    int n = wid * NCOMM + blockIdx.x;
    Raw raw;
    unit_load(p, n, lane, raw);
    const unsigned call = (unsigned)__builtin_amdgcn_readfirstlane(__hip_atomic_load(cnt, __ATOMIC_RELAXED, __HIP_MEMORY_SCOPE_AGENT)) + 1u;
    rsrc_t rs[NR - 1];
#pragma unroll
    for (int i = 0; i < NR - 1; ++i) {
      const int d = (p.rank + 1 + i) & 7;
      rs[i] = __builtin_amdgcn_make_buffer_rsrc(p.ws[d] + WS_STG + (size_t)(call & 1u) * STG_BYTES + (unsigned)((p.rank - d - 1) & 7) * SLOT_BYTES, 0, SLOT_BYTES, 0x00020000);
    }
    u8* const fl = p.ws[lane & 7] + WS_FLAG + (unsigned)((p.rank - lane - 1) & 7) * 4;
    const bool fl_on = lane < NR && lane != p.rank;
    unsigned* const ticket = (unsigned*)(myws + WS_TICKET);
    const unsigned tbase = (call - 1u) * (unsigned)(NUNIT - NCWAVES) - (unsigned)(2 * NCWAVES);
    unsigned tk = (unsigned)(n + NCWAVES) + tbase;
    bool first = true;
    for (;;) {
      Unit t;
      unit_pack(p, n, lane, raw, t);
      asm volatile("" : "+v"(t.o[0]), "+v"(t.o[1]), "+v"(t.o[2]), "+v"(t.o[3]));
      if (first) { for (int i = 0; i < wid; ++i) FBAR; } else FBAR;
#pragma unroll
      for (int k = 0; k < 28; ++k) { const int d = k % 7, i = (k / 7 + d) & 3; __builtin_amdgcn_raw_buffer_store_b128(t.o[i], rs[d], t.off, t.so[i], 17); }
      WAIT_V(XV); FBAR;
      asm volatile("s_waitcnt vmcnt(0)" : "+v"(tk) :: "memory");
      if (fl_on) __hip_atomic_fetch_add((unsigned*)(fl + t.flag), 1u, __ATOMIC_RELAXED, __HIP_MEMORY_SCOPE_SYSTEM);
      if (first && tid == 0) __hip_atomic_store(cnt, call, __ATOMIC_RELAXED, __HIP_MEMORY_SCOPE_AGENT);
      first = false;
      const unsigned nn = (unsigned)__builtin_amdgcn_readfirstlane(tk) - tbase;
      if (nn >= (unsigned)NUNIT) break;
      n = (int)nn;
      asm volatile("s_mov_b64 exec, 1\n\tglobal_atomic_add %0, %1, %2, %3 sc0\n\ts_mov_b64 exec, -1" : "=v"(tk) : "v"(0u), "v"(1u), "s"(ticket) : "memory");
      unit_load(p, n, lane, raw);
      FBAR; FBAR;
    }
    return;
  }
  const int gw = blockIdx.x - NCOMM;
  unsigned call;
  {
    Raw raw[3];
    const int n0 = wid * NGEMM + gw;
#pragma unroll
    for (int i = 0; i < 3; ++i) { const int n = n0 + i * NGWAVES; unit_load(p, n < NUNIT ? n : n0, lane, raw[i]); }
    call = (unsigned)__builtin_amdgcn_readfirstlane(__hip_atomic_load(cnt, __ATOMIC_RELAXED, __HIP_MEMORY_SCOPE_AGENT)) + 1u - (p.pad ? 1u : 0u);
    if (p.pad > 1) { unsigned long long t0 = __builtin_amdgcn_s_memrealtime(); while (__builtin_amdgcn_s_memrealtime() - t0 < (unsigned long long)(((gw >> 3) >> 2) * p.pad)) __builtin_amdgcn_s_sleep(8); }
    const rsrc_t rl = __builtin_amdgcn_make_buffer_rsrc(myws + WS_STG + (size_t)(call & 1u) * STG_BYTES + 7u * SLOT_BYTES, 0, SLOT_BYTES, 0x00020000);
    unsigned fo = 0;
#pragma unroll
    for (int i = 0; i < 3; ++i) {
      const int n = n0 + i * NGWAVES;
      if (n < NUNIT && !p.pad) {
        Unit t; unit_pack(p, n, lane, raw[i], t);
#pragma unroll
        for (int k = 0; k < 4; ++k) __builtin_amdgcn_raw_buffer_store_b128(t.o[k], rl, t.off, t.so[k], 17);
        fo = lane == i ? t.flag : fo;
      }
    }
    if (wid == 0) __builtin_amdgcn_fence(__ATOMIC_ACQUIRE, "");
    if (tid == 0) *(LAS float*)(lds + LDS_X + 32) = p.sx[0] * p.sw[0];
    WAIT_V(0);
    if (lane < 3 && n0 + lane * NGWAVES < NUNIT && !p.pad) __hip_atomic_fetch_add((unsigned*)(myws + WS_FLAG + fo + 7 * 4), 1u, __ATOMIC_RELAXED, __HIP_MEMORY_SCOPE_SYSTEM);
  }
  const int wr = wid >> 1, wc = wid & 1;
  const rsrc_t rs = __builtin_amdgcn_make_buffer_rsrc(myws + WS_STG + (size_t)(call & 1u) * STG_BYTES, 0, STG_BYTES, 0x00020000);
  const rsrc_t rf = __builtin_amdgcn_make_buffer_rsrc(myws + WS_FLAG, 0, NPANEL * 128, 0x00020000);
  const int vo = wid * 8192 + lane * 16, vob = wid * 4096 + lane * 16;
  const unsigned need = call * 8u;
  unsigned* const tmo = (unsigned*)(myws + WS_TMO);
#define KOFF(j) (((j) & 7u) * SLOT_BYTES + ((j) >> 3) * Q_BYTES)
#define DMA_A(i, slot, so) dma16(rs, lds + LDS_A + (slot) * BLK + wid * 8192 + ((i) >> 2) * 4096, vo, (so) + ((i) >> 2) * 4096, (i) & 3)
#define DMA_B(i, slot, h, so) dma16(rs, lds + LDS_B + (slot) * BLK + (h) * 16384 + wid * 4096, vob, (so) + (h) * 16384, i)
#define FLAGO(t) ((unsigned)(((lane < 32) ? ((t) & 15) : NPA + ((t) >> 4)) * 128 + (lane & 31) * 4 + wid * 65536))
#define POLL(dst, off) asm volatile("buffer_load_dword %0, %1, %2, 0 offen sc0 sc1" : "=v"(dst) : "v"(off), "s"(rf) : "memory")
#define MASK_OF(v) ({ const unsigned long long _b = __builtin_amdgcn_ballot_w64((v) >= need); (unsigned)_b & (unsigned)(_b >> 32); })
#define DIAG_WAITED() do {} while (0)
#define ENSURE(cand) do { if (__builtin_expect(wid == 0 && (cand) == 0, 0)) { \
      const unsigned long long _t0 = __builtin_amdgcn_s_memrealtime(); \
      for (;;) { unsigned _v; POLL(_v, fvo); asm volatile("s_waitcnt vmcnt(0)" : "+v"(_v) :: "memory"); rdy = MASK_OF(_v); cand = rdy & ~done; \
        if (cand) break; \
        if (__builtin_amdgcn_readfirstlane(__hip_atomic_load(tmo, __ATOMIC_RELAXED, __HIP_MEMORY_SCOPE_AGENT)) == call || __builtin_amdgcn_s_memrealtime() - _t0 > DEADLINE) { \
          __hip_atomic_store(tmo, call, __ATOMIC_RELAXED, __HIP_MEMORY_SCOPE_AGENT); *(LAS float*)(lds + LDS_X + 32) = __builtin_nanf(""); cand = ~done; break; } \
        __builtin_amdgcn_s_sleep(2); } \
      DIAG_WAITED(); __builtin_amdgcn_s_waitcnt(0xC07F); } } while (0)
#define PICK(cand) ({ const unsigned _r = (cand) & 0x7F7F7F7Fu; (unsigned)__builtin_ctz((_r ? _r : (cand)) | 0x80000000u); })

  int cur = half_of(p.tile[gw], 0);
  unsigned done = 0, rdy = 0, fvo = FLAGO(cur), pv;
  unsigned sA = (unsigned)(cur & 15) * BLK, sB = (unsigned)(NPA + (cur >> 4)) * BLK;
  unsigned bsrc;
  v4f acc[8][8];
  v8i A[8], B[4];
  LAS u8* ard = lds + LDS_A + wr * 16384 + lane * 16;
  LAS u8* brd = lds + LDS_B + wc * 8192 + lane * 16;
  LAS u8* xrd = lds + LDS_X;
  LAS u8* xwr = lds + LDS_X + wid * 4;
  asm volatile("" : "+v"(ard), "+v"(brd), "+v"(xrd), "+v"(xwr));
#define BFRAG(slot, n) ldfrag(brd + (slot) * BLK + ((n) >> 2) * 16384 + ((n) & 3) * 2048, ((n) >> 1) & 1)
  {
    unsigned c0_ = 0; ENSURE(c0_); const unsigned j0 = PICK(c0_); done = 1u << j0;
    unsigned c1_ = c0_ & ~done; ENSURE(c1_); const unsigned j1 = PICK(c1_);
    if (tid == 0) *(LAS unsigned*)(lds + LDS_X + 48) = j0 | (j1 << 8);
    WAIT_L(0); FBAR;
    if (tid == 0) __hip_atomic_store(cnt, call, __ATOMIC_RELAXED, __HIP_MEMORY_SCOPE_AGENT);
    STAMP();
    const unsigned jj = (unsigned)__builtin_amdgcn_readfirstlane(*(LAS unsigned*)(lds + LDS_X + 48));
    const unsigned k0 = KOFF(jj & 255u), k1 = KOFF(jj >> 8);
    done = (1u << (jj & 255u)) | (1u << (jj >> 8));
    POLL(pv, fvo);
#pragma unroll
    for (int i = 0; i < 8; ++i) DMA_A(i, 0, sA + k0);
#pragma unroll
    for (int i = 0; i < 8; ++i) DMA_B(i & 3, 0, i >> 2, sB + k0);
#pragma unroll
    for (int i = 0; i < 8; ++i) DMA_A(i, 1, sA + k1);
#pragma unroll
    for (int i = 0; i < 4; ++i) DMA_B(i, 1, 0, sB + k1);
    bsrc = sB + k1;
    WAIT_V(0); FBAR;
#pragma unroll
    for (int m = 0; m < 8; ++m) A[m] = ldfrag(ard + m * 2048, 0);
    B[0] = BFRAG(0, 0); B[1] = BFRAG(0, 1);
    DMA_B(0, 1, 1, bsrc); DMA_B(1, 1, 1, bsrc);
  }
#define KTILE(par, FIRST, SW, VP, VY, VX) do { \
    unsigned jv, asrc, cand; \
    _Pragma("unroll") for (int np = 0; np < 4; ++np) { \
      const int cb = (np & 1) * 2, nb = 2 - cb; \
      if (np == 1) { WAIT_V(VY); WAIT_L(0); BAR; SCHED; } \
      if (np == 3) { WAIT_V(VX); WAIT_L(0); BAR; SCHED; } \
      _Pragma("unroll") for (int m = 0; m < 8; ++m) { \
        _Pragma("unroll") for (int e = 0; e < 2; ++e) { \
          const int n = np * 2 + e, s = m * 2 + e; \
          if (FIRST) MFA0(acc[m][n], A[m], B[cb + e]); else MFA(acc[m][n], A[m], B[cb + e]); \
          if (np == 1 && s == 0) jv = *(LAS unsigned*)(xrd + (par) * 16); \
          if (s == 0 || s == 4) { const int f = s >> 2; \
            if (np < 3) B[nb + f] = BFRAG(par, np * 2 + 2 + f); else B[nb + f] = BFRAG((par) ^ 1, f); } \
          if (np == 3 && e == 1) A[m] = ldfrag(ard + ((par) ^ 1) * BLK + m * 2048, 0); \
          if (np == 0 && s == 1) { asm volatile("s_waitcnt vmcnt(" STR(VP) ")" : "+v"(pv) :: "memory"); rdy |= MASK_OF(pv); } \
          if (np == 0 && (s == 2 || s == 10)) DMA_B(2 + (s >> 3), (par) ^ 1, 1, bsrc); \
          if (np == 0 && s == 5) { cand = rdy & ~done; ENSURE(cand); } \
          if (np == 0 && s == 8) *(LAS unsigned*)(xwr + (par) * 16) = PICK(cand); \
          if (np == 1 && s == 2) { const unsigned po = (SW) ? nfvo : (rdy == 0xFFFFFFFFu ? 0x40000u : fvo); POLL(pv, po); } \
          if (np == 1 && s == 5) { asm volatile("" : "+v"(jv)); const unsigned j = (unsigned)__builtin_amdgcn_readfirstlane(jv), ko = KOFF(j); \
            bsrc = sB + ko; asrc = sA + ko; done |= 1u << j; \
            if (SW) { done = 0; rdy = 0; sA = nsA; sB = nsB; fvo = nfvo; } } \
          if (np == 1 && s >= 7 && e == 1) DMA_A((s - 7) >> 1, par, asrc); \
          if (np == 2 && e == 1 && m < 3) DMA_A(5 + m, par, asrc); \
          if (np == 2 && e == 1 && m >= 3 && m < 7) DMA_B(m - 3, par, 0, bsrc); \
          if (np == 3 && (s == 2 || s == 10)) DMA_B(s >> 3, par, 1, bsrc); \
          SCHED; \
        } } } } while (0)
  for (int r = 0;; ++r) {
    const int nx = half_of(p.tile[gw], r + 1);
    const bool has_next = nx != 0xFFFF;
    const int nxt = has_next ? nx : cur;
    const unsigned nsA = (unsigned)(nxt & 15) * BLK, nsB = (unsigned)(NPA + (nxt >> 4)) * BLK, nfvo = FLAGO(nxt);
    KTILE(0, true, false, 63, 63, 63);
    KTILE(1, false, false, 14, 17, 17);
    for (int t = 2; t < NKT; t += 2) {
      KTILE(0, false, false, 14, 17, 17);
      KTILE(1, false, t == NKT - 4, 14, 17, 17);
    }
    STAMP();
    asm volatile("s_nop 15\n\ts_nop 7" ::: "memory"); SCHED;
    {
      const float scale = *(LAS float*)(xrd + 32);
      const rsrc_t rc = __builtin_amdgcn_make_buffer_rsrc(p.out + (size_t)((cur & 15) * 256 + wr * 128) * N + (cur >> 4) * 256 + wc * 64, 0, 128 * N * 4, 0x00020000);
      const int co = (4 * (lane >> 4) * N + 4 * (lane & 15)) * 4;
#pragma unroll
      for (int m = 0; m < 8; ++m)
#pragma unroll
        for (int j = 0; j < 4; ++j)
#pragma unroll
          for (int g = 0; g < 2; ++g) {
            v4f ov;
#pragma unroll
            for (int i = 0; i < 4; ++i) {
              float v;
              asm volatile("v_accvgpr_read_b32 %0, %1" : "=v"(v) : "a"(acc[m][g * 4 + i][j]));
              ov[i] = __builtin_elementwise_maximum(v * scale, 0.f);
            }
            __builtin_amdgcn_raw_buffer_store_b128(__builtin_bit_cast(v4i, ov), rc, co + g * 512, (m * 16 + j) * N * 4, 2);
            SCHED;
          }
    }
    STAMP();
    if (!has_next) break;
    cur = nxt;
  }
  WAIT_V(0);
  STAMP();
}

static int g_pad = 0;
static void build_schedule(Params& p) {
  for (int b = NCOMM; b < GRID; ++b) {
    const int x = b & 7, c = (b >> 3) - NCOMM / 8, xr = x & 3, xc = x >> 2, W = NGEMM / 8;
    auto tile_of = [&](int o) {
      int ri, ci;
      if (o < 16) { ri = (o >> 3) * 2 + (o & 1); ci = (o & 7) >> 1; } else { ri = o & 3; ci = o >> 2; }
      return (u16)((xr + 4 * ri) | ((ci * 2 + xc) << 4));
    };
    int k = 0; u16* t = (u16*)p.tile[b - NCOMM];
    const int extra = 128 - 4 * W;
    t[k++] = tile_of(c);
    if (c < extra) t[k++] = tile_of(W + c);
    for (int r = 0; r < 3; ++r) t[k++] = tile_of(W + extra + r * W + (c + W - extra) % W);
    while (k <= MAXT) t[k++] = 0xFFFF;
  }
  int s = 0;
  auto put = [&](int panel, int q) { ((u16*)p.stream)[s++] = (u16)(panel << 2 | q); };
  for (int q = 0; q < 4; ++q) for (int i = 0; i < 8; ++i) { put(i, q); put(NPA + i, q); }
  for (int q = 0; q < 4; ++q) { for (int i = 8; i < 16; ++i) put(i, q); for (int i = 8; i < 20; ++i) put(NPA + i, q); }
  const int lim[4] = {20, 34, 50, 64};
  for (int r = 0; r < 3; ++r) for (int q = 0; q < 4; ++q) for (int i = lim[r]; i < lim[r + 1]; ++i) put(NPA + i, q);
}

extern "C" void kernel_launch(int rank, int nranks, ncclComm_t comm, hipStream_t stream,
                              void* const* d_in, const int* in_sizes, int n_in,
                              void* d_out, int out_size, const int* in_dtypes, int out_dtype,
                              void* const* d_out_all, void* const* d_ws_all, size_t ws_bytes) {
  if (nranks != NR || n_in != 4 || ws_bytes < WS_TOTAL || (unsigned)rank >= (unsigned)NR ||
      in_sizes[0] != M * KS || in_sizes[1] != KS * N || out_size != M * N) { fprintf(stderr, "unexpected problem shape\n"); return; }
  Params p;
  p.x = (const u8*)d_in[0]; p.w = (const u8*)d_in[1]; p.sx = (const float*)d_in[2]; p.sw = (const float*)d_in[3];
  p.out = (float*)d_out; p.rank = rank; p.pad = g_pad;
  for (int i = 0; i < NR; ++i) p.ws[i] = (u8*)d_ws_all[i];
  build_schedule(p);
  hipLaunchKernelGGL(fused_kernel, dim3(GRID), dim3(256), 0, stream, p);
  HIP_CHECK(hipGetLastError());
}
```

```cpp
#include <hip/hip_runtime.h>
#include <rccl/rccl.h>
#include <cstdio>

#define HIP_CHECK(expr) do { hipError_t _e = (expr); if (_e != hipSuccess) { \
    fprintf(stderr, "HIP error %s at %s:%d\n", hipGetErrorString(_e), __FILE__, __LINE__); \
    return; } } while (0)

#define LAS __attribute__((address_space(3)))
typedef unsigned char u8;
typedef unsigned short u16;
typedef int v8i __attribute__((ext_vector_type(8)));
typedef int v4i __attribute__((ext_vector_type(4)));
typedef unsigned v4u __attribute__((ext_vector_type(4)));
typedef float v4f __attribute__((ext_vector_type(4)));
typedef float v2f __attribute__((ext_vector_type(2)));
typedef __amdgpu_buffer_rsrc_t rsrc_t;

#define XV 21
#define DEEP 14
#define PDELAY 0
#define CWV 4
#define SLP 2
#define JS 6
#define LAUX 0
constexpr int M = 4096, N = 16384, K = 4096, NR = 8, KS = K / NR;
constexpr int NKT = 32;
constexpr int BLK = 32768;
constexpr int NPA = 16, NPB = 64, NPANEL = NPA + NPB;
constexpr unsigned KT_BYTES = NPANEL * BLK;
constexpr unsigned STG_BYTES = NKT * KT_BYTES;
constexpr int GRID = 256, NCOMM = 16, NGEMM = GRID - NCOMM, CW = CWV, NCWAVES = NCOMM * CW, NGWAVES = NGEMM * 4;
constexpr int MAXT = 5;
constexpr int NPIECE = NPANEL * 4, NUNIT = NPIECE * 8;
constexpr size_t WS_CNT = 0;
constexpr size_t WS_TICKET = GRID * 128;
constexpr size_t WS_FLAG = 65536;
constexpr unsigned TMO_OFF = NPANEL * 128;
constexpr size_t WS_DBG = 131072;
constexpr size_t WS_STG = 1u << 20;
constexpr size_t WS_TOTAL = WS_STG + 2 * (size_t)STG_BYTES;
constexpr unsigned DEADLINE = 50000000u;
constexpr int LDS_A = 0, LDS_B = 2 * BLK, LDS_X = 4 * BLK, LDS_BYTES = LDS_X + 64;

struct Params {
  const u8* x; const u8* w; const float* sx; const float* sw; float* out;
  u8* peer[NR];
  int pad; int pad2;
  unsigned tile[NGEMM][(MAXT + 1) / 2];
  unsigned stream[NPIECE / 2];
};
__device__ __forceinline__ int half_of(const unsigned* t, int i) { return (int)((t[i >> 1] >> ((i & 1) * 16)) & 0xFFFFu); }

#define STR2(x) #x
#define STR(x) STR2(x)
#define WAIT_V(n) asm volatile("s_waitcnt vmcnt(" STR(n) ")" ::: "memory")
#define WAIT_L(n) asm volatile("s_waitcnt lgkmcnt(" STR(n) ")" ::: "memory")
#define BAR __builtin_amdgcn_s_barrier()
#define FBAR do { asm volatile("" ::: "memory"); __builtin_amdgcn_s_barrier(); asm volatile("" ::: "memory"); } while (0)
#define SCHED __builtin_amdgcn_sched_barrier(0)
#define STAMP() do {} while (0)

#define DMAV 0
#define PU 2
constexpr int dma_slot(int k) {
  if (DMAV == 0) return 22 + 4 * k;
  if (DMAV == 1) return k < 12 ? 22 + 2 * k : 50 + 8 * (k - 12);
  if (DMAV == 2) return k < 12 ? 22 + 3 * k - (k & 1) : 58 + 4 * (k - 12) + (k > 13 ? 4 : 0);
  return 0;
}
constexpr int POLL_SLOT = 17;
constexpr int ND = 32 / PU;
struct Slots { int dma[64], mx[128], st[128], park[64], tail; bool ok; };
constexpr Slots make_slots() {
  Slots t{};
  for (int i = 0; i < 64; ++i) t.dma[i] = t.park[i] = -1;
  for (int i = 0; i < 128; ++i) t.mx[i] = t.st[i] = -1;
  for (int k = 0; k < 16; ++k) t.dma[dma_slot(k) % 64] = k;
  t.ok = dma_slot(0) >= 22 && dma_slot(12) >= 48 && dma_slot(15) < 64 + 20 && t.dma[POLL_SLOT] < 0;
  int fr[64] = {}, nf = 0;
  for (int w = 23; w < 78; ++w) if (t.dma[w % 64] < 0) fr[nf++] = w;
  for (int u = 0; u < 32; ++u) {
    const int x = fr[u * nf / 32], y = fr[(u + 1) * nf / 32 - 1];
    t.mx[x] = u; t.st[y] = u; t.tail += y < 64;
    if (x < 21 + 2 * (u >> 2) || x >= 64 + 2 * (u >> 2)) t.ok = false;
  }
  int c = 0;
  for (int w = 15; w < 64 && c < 32; ++w) if (t.dma[w] < 0) {
    t.park[w] = ((c >> 1) & 7) * 4 + (c >> 4) * 2 + (c & 1);
    if (w >= 32 + (c >> 4) * 16 + 2 * ((c >> 1) & 7)) t.ok = false;
    ++c;
  }
  if (c != 32) t.ok = false;
  return t;
}
constexpr Slots SL = make_slots();
static_assert(SL.ok, "drain schedule violates a deadline");
constexpr int dma_at(int s) { return SL.dma[s]; }
constexpr int g0_mx(int mode, int s) { return mode == 1 || mode == 2 ? SL.mx[(mode - 1) * 64 + s] : -1; }
constexpr int g0_st(int mode, int s) { return mode == 1 || mode == 2 ? SL.st[(mode - 1) * 64 + s] : -1; }
constexpr int park_frag_at(int mode, int s) { return mode == 2 ? SL.park[s] : -1; }
constexpr int g1_op(int mode, int s, int op) {
  if (mode < 3) return -1;
  const int q = s & 15, np = s >> 4;
  int k = op == 3 ? (q == 8 ? 0 : q == 12 ? 1 : -1) : (q == 1 + 2 * op ? 0 : q == 7 + 2 * op ? 1 : -1);
  if (DMAV != 0 && op == 3) k = q == 13 ? 0 : q == 15 ? 1 : -1;
  if (k < 0) return -1;
  if (PU == 8) return (mode - 3) * 8 + np * 2 + k;
  if (PU == 4) return np == 1 || np == 2 ? (mode - 3) * 4 + (np - 1) * 2 + k : -1;
  return np == 2 ? (mode - 3) * 2 + k : -1;
}
constexpr int nvm(int mode, int prev, int lo, int hi) {
  int c = 0;
  for (int s = lo + 1; s < hi; ++s) {
    const int q = (s + 128) % 64, md = s >= 0 ? mode : s >= -64 ? prev : 0;
    c += (dma_at(q) >= 0) + (q == POLL_SLOT) + (g0_st(md, q) >= 0) + (g1_op(md, q, 3) >= 0);
  }
  return c;
}
template <int V> struct IC { static constexpr int value = V; };
template <int I, int E, class F> __device__ __forceinline__ void static_for(F&& f) { if constexpr (I < E) { f(IC<I>{}); static_for<I + 1, E>(f); } }

enum : int { V_XWR = 15, V_ARD = 16, V_BRD, V_XRD, V_VL, V_CO, V_FVO, V_PV, V_TMP, V_T = 24, V_A = 32, V_B = 96, V_ACC = 128, A_ACC = 0, A_PARK = 128 };
#define C10(p, t) #p #t "0", #p #t "1", #p #t "2", #p #t "3", #p #t "4", #p #t "5", #p #t "6", #p #t "7", #p #t "8", #p #t "9"
#define C20_255(p) C10(p, 2), C10(p, 3), C10(p, 4), C10(p, 5), C10(p, 6), C10(p, 7), C10(p, 8), C10(p, 9), C10(p, 10), C10(p, 11), C10(p, 12), C10(p, 13), \
    C10(p, 14), C10(p, 15), C10(p, 16), C10(p, 17), C10(p, 18), C10(p, 19), C10(p, 20), C10(p, 21), C10(p, 22), C10(p, 23), C10(p, 24), \
    #p "250", #p "251", #p "252", #p "253", #p "254", #p "255"
#define OWNED_REGS "v15", "v16", "v17", "v18", "v19", C20_255(v), C10(a, ), C10(a, 1), C20_255(a)
template <int m, int n, int slot, bool zero> __device__ __forceinline__ void mfma() {
  constexpr int c = (n < 4 ? V_ACC : A_ACC) + m * 16 + (n & 3) * 4, a = V_A + m * 8, b = V_B + slot * 8;
  if constexpr (n < 4 && !zero) asm volatile("v_mfma_f32_16x16x128_f8f6f4 v[%c0:%c1], v[%c2:%c3], v[%c4:%c5], v[%c0:%c1] cbsz:1 blgp:1" :: "n"(c), "n"(c + 3), "n"(a), "n"(a + 7), "n"(b), "n"(b + 7));
  if constexpr (n < 4 && zero) asm volatile("v_mfma_f32_16x16x128_f8f6f4 v[%c0:%c1], v[%c2:%c3], v[%c4:%c5], 0 cbsz:1 blgp:1" :: "n"(c), "n"(c + 3), "n"(a), "n"(a + 7), "n"(b), "n"(b + 7));
  if constexpr (n >= 4 && !zero) asm volatile("v_mfma_f32_16x16x128_f8f6f4 a[%c0:%c1], v[%c2:%c3], v[%c4:%c5], a[%c0:%c1] cbsz:1 blgp:1" :: "n"(c), "n"(c + 3), "n"(a), "n"(a + 7), "n"(b), "n"(b + 7));
  if constexpr (n >= 4 && zero) asm volatile("v_mfma_f32_16x16x128_f8f6f4 a[%c0:%c1], v[%c2:%c3], v[%c4:%c5], 0 cbsz:1 blgp:1" :: "n"(c), "n"(c + 3), "n"(a), "n"(a + 7), "n"(b), "n"(b + 7));
}
template <int dst, int adr, int off, int sw> __device__ __forceinline__ void ldfrag() {
  asm volatile("ds_read_b128 v[%c0:%c1], v%c4 offset:%c5\n\tds_read_b128 v[%c2:%c3], v%c4 offset:%c6"
               :: "n"(dst), "n"(dst + 3), "n"(dst + 4), "n"(dst + 7), "n"(adr), "n"(off + sw * 1024), "n"(off + 1024 - sw * 1024) : "memory");
}
template <int m, int slot> __device__ __forceinline__ void ld_a() { ldfrag<V_A + m * 8, V_ARD, slot * BLK + m * 2048, 0>(); }
template <int r, int slot, int n> __device__ __forceinline__ void ld_b() { ldfrag<V_B + r * 8, V_BRD, slot * BLK + (n >> 2) * 16384 + (n & 3) * 2048, (n >> 1) & 1>(); }
__device__ __forceinline__ void set_m0(unsigned v) { asm volatile("s_mov_b32 m0, %0" :: "s"(v) : "memory"); }
template <int k> __device__ __forceinline__ void dma(rsrc_t rs, unsigned so) {
  asm volatile("buffer_load_dwordx4 v%c0, %1, %2 offen offset:%c3 lds" :: "n"(V_VL), "s"(rs), "s"(so), "n"(k * 1024) : "memory");
}
template <int n> __device__ __forceinline__ void wait_v() { asm volatile("s_waitcnt vmcnt(%c0)" :: "n"(n > 63 ? 63 : n) : "memory"); }
template <int n> __device__ __forceinline__ void wait_l() { asm volatile("s_waitcnt lgkmcnt(%c0)" :: "n"(n) : "memory"); }
template <int off> __device__ __forceinline__ void x_read() { asm volatile("ds_read_b32 v%c0, v%c1 offset:%c2" :: "n"(V_TMP), "n"(V_XRD), "n"(off) : "memory"); }
template <int nl> __device__ __forceinline__ unsigned x_value() { unsigned r; asm volatile("s_waitcnt lgkmcnt(%c1)\n\tv_readfirstlane_b32 %0, v%c2" : "=s"(r) : "n"(nl), "n"(V_TMP) : "memory"); return r; }
template <int off> __device__ __forceinline__ void x_write(unsigned v) {
  asm volatile("v_mov_b32 v%c0, %3\n\tds_write_b32 v%c1, v%c0 offset:%c2" :: "n"(V_TMP), "n"(V_XWR), "n"(off), "s"(v) : "memory");
}
__device__ __forceinline__ unsigned pick(unsigned cand, unsigned& done) {
  unsigned t;
  asm volatile("s_and_b32 %0, %2, 0x7f7f7f7f\n\ts_cselect_b32 %0, %0, %2\n\ts_ff1_i32_b32 %0, %0\n\ts_bitset1_b32 %1, %0\n\ts_mul_i32 %0, %0, %3" : "=&s"(t), "+s"(done) : "s"(cand), "n"(KT_BYTES) : "scc");
  return t;
}
__device__ __forceinline__ void poll(rsrc_t rf, unsigned so) { asm volatile("buffer_load_dword v%c0, v%c1, %2, %3 offen sc0 sc1" :: "n"(V_PV), "n"(V_FVO), "s"(rf), "s"(so) : "memory"); }
template <int nv> __device__ __forceinline__ unsigned poll_mask(unsigned need) {
  unsigned long long b; asm volatile("s_waitcnt vmcnt(%c1)\n\tv_cmp_le_u32 %0, %2, v%c3" : "=s"(b) : "n"(nv > 63 ? 63 : nv), "s"(need), "n"(V_PV) : "memory");
  return (unsigned)b & (unsigned)(b >> 32);
}
__device__ __forceinline__ void aim_poll(int t) {
  const unsigned lb = (unsigned)(NPA - 1 + (t >> 4)) * 128u, la = (unsigned)(t & 15) * 128u - lb;
  asm volatile("v_lshrrev_b32 v%c0, 2, v%c1\n\tv_add_u32 v%c0, %2, v%c0\n\ts_mov_b64 exec, 0xffffffff\n\tv_add_u32 v%c0, %3, v%c0\n\ts_mov_b64 exec, -1" :: "n"(V_FVO), "n"(V_VL), "s"(lb), "s"(la));
}
template <int t, int u> __device__ __forceinline__ void q_g0(unsigned scale) {
  constexpr int d = V_T + t * 4, c = V_ACC + (u >> 2) * 16 + (u & 3);
  asm volatile("v_mul_f32 v%c0, %8, v%c4\n\tv_mul_f32 v%c1, %8, v%c5\n\tv_mul_f32 v%c2, %8, v%c6\n\tv_mul_f32 v%c3, %8, v%c7\n\t"
               "v_maximum3_f32 v%c0, v%c0, 0, 0\n\tv_maximum3_f32 v%c1, v%c1, 0, 0\n\tv_maximum3_f32 v%c2, v%c2, 0, 0\n\tv_maximum3_f32 v%c3, v%c3, 0, 0"
               :: "n"(d), "n"(d + 1), "n"(d + 2), "n"(d + 3), "n"(c), "n"(c + 4), "n"(c + 8), "n"(c + 12), "s"(scale));
}
template <int t, int base, int u> __device__ __forceinline__ void q_read() {
  constexpr int d = V_T + t * 4, c = base + (u >> 2) * 16 + (u & 3);
  asm volatile("v_accvgpr_read_b32 v%c0, a%c4\n\tv_accvgpr_read_b32 v%c1, a%c5\n\tv_accvgpr_read_b32 v%c2, a%c6\n\tv_accvgpr_read_b32 v%c3, a%c7"
               :: "n"(d), "n"(d + 1), "n"(d + 2), "n"(d + 3), "n"(c), "n"(c + 4), "n"(c + 8), "n"(c + 12));
}
template <int t> __device__ __forceinline__ void q_scale(unsigned scale) {
  constexpr int d = V_T + t * 4;
  asm volatile("v_mul_f32 v%c0, %4, v%c0\n\tv_mul_f32 v%c1, %4, v%c1\n\tv_mul_f32 v%c2, %4, v%c2\n\tv_mul_f32 v%c3, %4, v%c3" :: "n"(d), "n"(d + 1), "n"(d + 2), "n"(d + 3), "s"(scale));
}
template <int t> __device__ __forceinline__ void q_clamp() {
  constexpr int d = V_T + t * 4;
  asm volatile("v_maximum3_f32 v%c0, v%c0, 0, 0\n\tv_maximum3_f32 v%c1, v%c1, 0, 0\n\tv_maximum3_f32 v%c2, v%c2, 0, 0\n\tv_maximum3_f32 v%c3, v%c3, 0, 0" :: "n"(d), "n"(d + 1), "n"(d + 2), "n"(d + 3));
}
template <int t, int u, int g> __device__ __forceinline__ void q_store(rsrc_t rc) {
  constexpr int d = V_T + t * 4;
  asm volatile("buffer_store_dwordx4 v[%c0:%c1], v%c2, %3, %4 offen offset:%c5 nt" :: "n"(d), "n"(d + 3), "n"(V_CO), "s"(rc), "s"(((u >> 2) * 16 + (u & 3)) * N * 4), "n"(g * 512) : "memory");
}
template <int f> __device__ __forceinline__ void park_frag() {
  constexpr int d = A_PARK + f * 4, c = A_ACC + f * 4;
  asm volatile("v_accvgpr_mov_b32 a%c0, a%c4\n\tv_accvgpr_mov_b32 a%c1, a%c5\n\tv_accvgpr_mov_b32 a%c2, a%c6\n\tv_accvgpr_mov_b32 a%c3, a%c7"
               :: "n"(d), "n"(d + 1), "n"(d + 2), "n"(d + 3), "n"(c), "n"(c + 1), "n"(c + 2), "n"(c + 3));
}

#define KSTAMP(i) do {} while (0)
struct Raw { unsigned r[16]; };
struct Unit { v4u o[4]; unsigned off, so[4], flag; };

__device__ __forceinline__ void unit_load(const Params& p, int n, int lane, Raw& d) {
  const int piece = half_of(p.stream, n >> 3), u = n & 7, panel = piece >> 2, q = piece & 3;
  const int g = lane >> 4, c = lane & 15;
  const bool isx = panel < NPA;
  const rsrc_t rs = __builtin_amdgcn_make_buffer_rsrc((void*)(isx ? p.x : p.w), 0, isx ? M * KS : KS * N, 0x00020000);
  const unsigned vo = isx ? (unsigned)(c * KS + g * 16) : (unsigned)(g * 16 * N + 4 * c);
  const unsigned so = isx ? (unsigned)((panel * 256 + u * 32) * KS + q * 128) : (unsigned)((q * 128 + (u & 1) * 64) * N + (panel - NPA) * 256 + (u >> 1) * 64);
  const unsigned sa = isx ? 4u : (unsigned)N, sb = isx ? 64u : 4u * N, sc = isx ? 16u * KS : 8u * N;
#pragma unroll
  for (int j = 0; j < 16; ++j) d.r[j] = __builtin_amdgcn_raw_buffer_load_b32(rs, vo, so + (j & 3) * sa + ((j >> 2) & 1) * sb + (j >> 3) * sc, 0);
}
__device__ __forceinline__ void unit_pack(const Params& p, int n, int lane, const Raw& raw, Unit& t) {
  const int piece = half_of(p.stream, n >> 3), u = n & 7, panel = piece >> 2, q = piece & 3;
  t.off = (unsigned)q * (8u * KT_BYTES) + (unsigned)panel * BLK + (unsigned)lane * 16u;
  t.flag = (unsigned)(panel * 128 + q * 32);
  t.so[0] = 0;
  if (panel < NPA) {
#pragma unroll
    for (int i = 0; i < 4; ++i) t.o[i] = (v4u){raw.r[i * 4], raw.r[i * 4 + 1], raw.r[i * 4 + 2], raw.r[i * 4 + 3]};
    t.off += u * 4096; t.so[1] = 1024; t.so[2] = 2048; t.so[3] = 3072;
  } else {
#pragma unroll
    for (int d = 0; d < 4; ++d) {
      const unsigned r0 = raw.r[4 * d], r1 = raw.r[4 * d + 1], r2 = raw.r[4 * d + 2], r3 = raw.r[4 * d + 3];
      const unsigned t0 = __builtin_amdgcn_perm(r1, r0, 0x05010400u), t1 = __builtin_amdgcn_perm(r1, r0, 0x07030602u);
      const unsigned t2 = __builtin_amdgcn_perm(r3, r2, 0x05010400u), t3 = __builtin_amdgcn_perm(r3, r2, 0x07030602u);
      t.o[0][d] = __builtin_amdgcn_perm(t2, t0, 0x05040100u); t.o[1][d] = __builtin_amdgcn_perm(t2, t0, 0x07060302u);
      t.o[2][d] = __builtin_amdgcn_perm(t3, t1, 0x05040100u); t.o[3][d] = __builtin_amdgcn_perm(t3, t1, 0x07060302u);
    }
    t.off += (u >> 1) * 8192 + (u & 1) * 1024; t.so[1] = 2048; t.so[2] = (u & 1) ? 4096 - 1024 : 4096 + 1024; t.so[3] = t.so[2] + 2048;
  }
}

__global__ __launch_bounds__(256, 1) void fused_kernel(const Params p) {
  __shared__ __attribute__((aligned(16))) u8 shm[LDS_BYTES];
  LAS u8* lds = (LAS u8*)shm;
  const int tid = threadIdx.x, wid = __builtin_amdgcn_readfirstlane(tid >> 6), lane = tid & 63;
  u8* const myws = p.peer[NR - 1];
  STAMP();
  unsigned* const cnt = (unsigned*)(myws + WS_CNT + blockIdx.x * 128);
  if (blockIdx.x < NCOMM) {
    int n = wid * NCOMM + blockIdx.x;
    Raw raw;
    unit_load(p, n, lane, raw);
    const unsigned call = (unsigned)__builtin_amdgcn_readfirstlane(__hip_atomic_load(cnt, __ATOMIC_RELAXED, __HIP_MEMORY_SCOPE_AGENT)) + 1u;
    rsrc_t rs[NR - 1];
#pragma unroll
    for (int i = 0; i < NR - 1; ++i)
      rs[i] = __builtin_amdgcn_make_buffer_rsrc(p.peer[i] + WS_STG + (size_t)(call & 1u) * STG_BYTES + (unsigned)(6 - i) * KT_BYTES, 0, STG_BYTES - 7u * KT_BYTES, 0x00020000);
    u8* const fl = p.peer[lane < NR - 1 ? lane : 0] + WS_FLAG + (unsigned)((6 - lane) & 7) * 4;
    unsigned* const ticket = (unsigned*)(myws + WS_TICKET);
    const unsigned tbase = (call - 1u) * (unsigned)(NUNIT - NCWAVES) - (unsigned)(2 * NCWAVES);
    unsigned tk = (unsigned)(n + NCWAVES) + tbase;
    bool first = true;
    for (;;) {
      Unit t;
      unit_pack(p, n, lane, raw, t);
      asm volatile("" : "+v"(t.o[0]), "+v"(t.o[1]), "+v"(t.o[2]), "+v"(t.o[3]));
      if (first) { for (int i = 0; i < wid; ++i) FBAR; } else FBAR;
#pragma unroll
      for (int k = 0; k < 28; ++k) { const int d = k % 7, i = (k / 7 + d) & 3; __builtin_amdgcn_raw_buffer_store_b128(t.o[i], rs[d], t.off, t.so[i], 17); }
      FBAR;
      WAIT_V(DEEP); FBAR;
      __builtin_amdgcn_s_waitcnt(0x0F70);
      asm volatile("s_waitcnt vmcnt(0)" : "+v"(tk) :: "memory");
      if (lane < NR - 1) __hip_atomic_fetch_add((unsigned*)(fl + t.flag), 1u, __ATOMIC_RELAXED, __HIP_MEMORY_SCOPE_SYSTEM);
      if (first && tid == 0) __hip_atomic_store(cnt, call, __ATOMIC_RELAXED, __HIP_MEMORY_SCOPE_AGENT);
      first = false;
      const unsigned nn = (unsigned)__builtin_amdgcn_readfirstlane(tk) - tbase;
      if (nn >= (unsigned)NUNIT) break;
      n = (int)nn;
      asm volatile("s_mov_b64 exec, 1\n\tglobal_atomic_add %0, %1, %2, %3 sc0\n\ts_mov_b64 exec, -1" : "=v"(tk) : "v"(0u), "v"(1u), "s"(ticket) : "memory");
      unit_load(p, n, lane, raw);
      FBAR;
    }
    return;
  }
  const int gw = blockIdx.x - NCOMM;
  unsigned call;
  {
    if (PDELAY) { unsigned long long t0 = __builtin_amdgcn_s_memrealtime(); while (__builtin_amdgcn_s_memrealtime() - t0 < PDELAY) __builtin_amdgcn_s_sleep(8); }
    Raw raw[3];
    const int n0 = wid * NGEMM + gw;
#pragma unroll
    for (int i = 0; i < 3; ++i) { const int n = n0 + i * NGWAVES; unit_load(p, n < NUNIT ? n : n0, lane, raw[i]); }
    call = (unsigned)__builtin_amdgcn_readfirstlane(__hip_atomic_load(cnt, __ATOMIC_RELAXED, __HIP_MEMORY_SCOPE_AGENT)) + 1u - (p.pad ? 1u : 0u);
    if (p.pad > 1) { unsigned long long t0 = __builtin_amdgcn_s_memrealtime(); while (__builtin_amdgcn_s_memrealtime() - t0 < (unsigned long long)(((gw >> 3) >> 2) * p.pad)) __builtin_amdgcn_s_sleep(8); }
    const rsrc_t rl = __builtin_amdgcn_make_buffer_rsrc(myws + WS_STG + (size_t)(call & 1u) * STG_BYTES + 7u * KT_BYTES, 0, STG_BYTES - 7u * KT_BYTES, 0x00020000);
    unsigned fo = 0;
#pragma unroll
    for (int i = 0; i < 3; ++i) {
      const int n = n0 + i * NGWAVES;
      if (n < NUNIT && !p.pad) {
        Unit t; unit_pack(p, n, lane, raw[i], t);
#pragma unroll
        for (int k = 0; k < 4; ++k) __builtin_amdgcn_raw_buffer_store_b128(t.o[k], rl, t.off, t.so[k], 17);
        fo = lane == i ? t.flag : fo;
      }
    }
    if (wid == 0) __builtin_amdgcn_fence(__ATOMIC_ACQUIRE, "");
    if (tid == 0) *(LAS float*)(lds + LDS_X + 32) = p.sx[0] * p.sw[0];
    WAIT_V(0);
    if (lane < 3 && n0 + lane * NGWAVES < NUNIT && !p.pad) __hip_atomic_fetch_add((unsigned*)(myws + WS_FLAG + fo + 7 * 4), 1u, __ATOMIC_RELAXED, __HIP_MEMORY_SCOPE_SYSTEM);
  }
  WAIT_L(0); FBAR;
  if (tid == 0) __hip_atomic_store(cnt, call, __ATOMIC_RELAXED, __HIP_MEMORY_SCOPE_AGENT);
  const int wr = wid >> 1, wc = wid & 1;
  const rsrc_t rs = __builtin_amdgcn_make_buffer_rsrc(myws + WS_STG + (size_t)(call & 1u) * STG_BYTES, 0, STG_BYTES, 0x00020000);
  const rsrc_t rf = __builtin_amdgcn_make_buffer_rsrc(myws + WS_FLAG, 0, TMO_OFF + 128, 0x00020000);
  const unsigned need = call * 8u;
  const unsigned lbase = (unsigned)(size_t)lds;
  const unsigned rinit = wid ? 0xFFFFFFFFu : 0u;
  unsigned long long tlo = p.tile[gw][0] | ((unsigned long long)p.tile[gw][1] << 32); unsigned thi = p.tile[gw][2];
  asm volatile("v_mov_b32 v%c0, %5\n\tv_mov_b32 v%c1, %6\n\tv_mov_b32 v%c2, %7\n\tv_mov_b32 v%c3, %8\n\tv_mov_b32 v%c4, %9\n\tv_mov_b32 v%c10, %11"
               :: "n"(V_ARD), "n"(V_BRD), "n"(V_XRD), "n"(V_VL), "n"(V_CO),
                  "v"(lbase + LDS_A + wr * 16384 + lane * 16), "v"(lbase + LDS_B + wc * 8192 + lane * 16), "v"(lbase + LDS_X), "v"(lane * 16),
                  "v"((4 * (lane >> 4) * N + 4 * (lane & 15)) * 4), "n"(V_XWR), "v"(lbase + LDS_X + wid * 4) : OWNED_REGS, "memory");
#define M0_A(slot, h) (lbase + LDS_A + (slot) * BLK + wid * 8192 + (h) * 4096)
#define M0_B(slot, h) (lbase + LDS_B + (slot) * BLK + (h) * 16384 + wid * 4096)
#define SRC_A(t) ((unsigned)(((t) & 15) * BLK + wid * 8192))
#define SRC_B(t) ((unsigned)((NPA + ((t) >> 4)) * BLK + wid * 4096))
#define DIAG_WAITED() do {} while (0)
#define ENSURE(cand) do { if (__builtin_expect((cand) == 0, 0)) { \
      const unsigned _t0 = (unsigned)__builtin_amdgcn_s_memrealtime(); \
      for (;;) { unsigned _w; \
        poll(rf, 0u); asm volatile("buffer_load_dword v%c0, off, %1, %2 sc0 sc1" :: "n"(V_TMP), "s"(rf), "s"(TMO_OFF) : "memory"); \
        rdy = poll_mask<0>(need); cand = rdy & ~done; \
        if (cand) break; \
        asm volatile("v_readfirstlane_b32 %0, v%c1" : "=s"(_w) : "n"(V_TMP)); \
        if (_w == call || (unsigned)__builtin_amdgcn_s_memrealtime() - _t0 > DEADLINE) { \
          asm volatile("v_mov_b32 v%c0, %1\n\tbuffer_store_dword v%c0, off, %2, %3 sc0 sc1\n\ts_nop 1" :: "n"(V_TMP), "s"(call), "s"(rf), "s"(TMO_OFF) : "memory"); \
          x_write<32>(0x7FC00000u); cand = ~done; break; } \
        __builtin_amdgcn_s_sleep(SLP); } \
      DIAG_WAITED(); __builtin_amdgcn_s_waitcnt(0xC07F); } } while (0)
#define POLL_OFF (rdy == 0xFFFFFFFFu ? 0x40000u : 0u)

  int cur = (int)(tlo & 0xFFFFu);
  unsigned done = 0, rdy = rinit;
  unsigned sA = SRC_A(cur), sB = SRC_B(cur);
  unsigned bsrc;
  aim_poll(cur);
  {
    unsigned c_ = rdy; ENSURE(c_); x_write<0>(pick(c_, done));
    c_ = rdy & ~done; ENSURE(c_); x_write<16>(pick(c_, done));
    WAIT_L(0); FBAR;
    STAMP();
    x_read<0>(); const unsigned k0 = x_value<0>();
    x_read<16>(); const unsigned k1 = x_value<0>();
    poll(rf, POLL_OFF);
#define DMA4(m0v, so) do { set_m0(m0v); asm volatile("s_nop 0"); dma<0>(rs, so); dma<1>(rs, so); dma<2>(rs, so); dma<3>(rs, so); } while (0)
    DMA4(M0_A(0, 0), sA + k0); DMA4(M0_A(0, 1), sA + k0 + 4096); DMA4(M0_B(0, 0), sB + k0); DMA4(M0_B(0, 1), sB + k0 + 16384);
    DMA4(M0_A(1, 0), sA + k1); DMA4(M0_A(1, 1), sA + k1 + 4096); DMA4(M0_B(1, 0), sB + k1);
    if (dma_slot(12) < 64) DMA4(M0_B(1, 1), sB + k1 + 16384);
    bsrc = sB + k1;
    WAIT_V(0); FBAR;
    ld_b<0, 0, 0>(); ld_a<0, 0>(); ld_a<1, 0>(); ld_b<1, 0, 1>(); ld_a<2, 0>(); ld_a<3, 0>(); ld_a<4, 0>(); ld_a<5, 0>(); ld_a<6, 0>(); ld_a<7, 0>();
  }
#define KTILE(par, MODE, PREV, SW) do { \
    unsigned asrc, cand, jp; \
    static_for<0, 64>([&](auto gsc) __attribute__((always_inline)) { \
      constexpr int gs = decltype(gsc)::value, np = gs >> 4, s = gs & 15, m = s >> 1, e = s & 1; \
      constexpr int cb = (np & 1) * 2, nb = 2 - cb, z = e ^ (m & 1); \
      if constexpr (gs == 16) { wait_v<nvm(MODE, PREV, dma_slot(15) - 128, 16)>(); wait_l<0>(); BAR; } \
      if constexpr (gs == 48) { wait_v<nvm(MODE, PREV, dma_slot(11) - 64, 48)>(); wait_l<0>(); BAR; } \
      if constexpr (gs <= 1) wait_l<14>(); \
      if constexpr (gs == 4 || gs == 6) wait_l<12>(); \
      if constexpr (gs == 8) wait_l<10>(); \
      if constexpr (gs == 10 || gs == 12 || gs == 14) wait_l<19 - gs>(); \
      if constexpr (gs == 32 || gs == 33) wait_l<2>(); \
      mfma<m, np * 2 + z, cb + z, (MODE) == 2>(); SCHED; \
      if constexpr (gs == 16) x_read<(par) * 16>(); \
      if constexpr (gs == 20) { const unsigned ko = x_value<2>(); bsrc = sB + ko; asrc = sA + ko; } \
      if constexpr ((MODE) == 1 && gs == 20) x_read<32>(); \
      if constexpr ((MODE) == 1 && gs == 22) scale = x_value<2>(); \
      if constexpr (s == 0 || s == 4) { if constexpr (np < 3) ld_b<nb + (s >> 2), par, np * 2 + 2 + (s >> 2)>(); else ld_b<nb + (s >> 2), (par) ^ 1, (s >> 2)>(); } \
      if constexpr (np == 3 && e == 1) ld_a<m, (par) ^ 1>(); \
      if constexpr (gs == 1) rdy |= poll_mask<nvm(MODE, PREV, POLL_SLOT - 64, 1)>(need); \
      if constexpr (gs == 3) { cand = rdy & ~done; ENSURE(cand); } \
      if constexpr (gs == 7) jp = pick(cand, done); \
      if constexpr (gs == 9) x_write<(par) * 16>(jp); \
      if constexpr (gs == 11 && (SW)) { done = 0; rdy = rinit; aim_poll(nxt); } \
      if constexpr (gs == POLL_SLOT) poll(rf, POLL_OFF); \
      if constexpr (dma_at((gs + 1) % 64) >= 0 && dma_at((gs + 1) % 64) % 4 == 0) { constexpr int k = dma_at((gs + 1) % 64), sl = dma_slot(k) > 64 ? (par) ^ 1 : (par); \
        set_m0(k < 8 ? M0_A(sl, k >> 2) : M0_B(sl, (k >> 2) - 2)); } \
      if constexpr (dma_at(gs) >= 0) { constexpr int k = dma_at(gs); dma<k & 3>(rs, k < 8 ? asrc + (k >> 2) * 4096 : bsrc + ((k >> 2) - 2) * 16384); } \
      if constexpr (gs == 29 && (SW)) { sA = SRC_A(nxt); sB = SRC_B(nxt); } \
      if constexpr (g0_mx(MODE, gs) >= 0) q_g0<g0_mx(MODE, gs) & 1, g0_mx(MODE, gs)>(scale); \
      if constexpr (g0_st(MODE, gs) >= 0) q_store<g0_st(MODE, gs) & 1, g0_st(MODE, gs), 0>(rc); \
      if constexpr (park_frag_at(MODE, gs) >= 0) park_frag<park_frag_at(MODE, gs)>(); \
      if constexpr (g1_op(MODE, gs, 0) >= 0) q_read<g1_op(MODE, gs, 0) & 1, A_PARK, g1_op(MODE, gs, 0)>(); \
      if constexpr (g1_op(MODE, gs, 1) >= 0) q_scale<g1_op(MODE, gs, 1) & 1>(scale); \
      if constexpr (g1_op(MODE, gs, 2) >= 0) q_clamp<g1_op(MODE, gs, 2) & 1>(); \
      if constexpr (g1_op(MODE, gs, 3) >= 0) q_store<g1_op(MODE, gs, 3) & 1, g1_op(MODE, gs, 3), 1>(rc); \
      SCHED; \
    }); } while (0)
  unsigned scale = 0;
  rsrc_t rc = __builtin_amdgcn_make_buffer_rsrc(p.out, 0, 0, 0x00020000);
  for (int r = 0;; ++r) {
    tlo = (tlo >> 16) | ((unsigned long long)thi << 48); thi = (thi >> 16) | 0xFFFF0000u;
    const int nx = (int)(tlo & 0xFFFFu);
    const bool has_next = nx != 0xFFFF;
    const int nxt = has_next ? nx : cur;
    KSTAMP(0);
    KTILE(0, 2, 1, false); KSTAMP(1);
    static_for<1, ND + 2>([&](auto tc) __attribute__((always_inline)) { constexpr int t = decltype(tc)::value;
      KTILE(t & 1, (t <= ND ? 2 + t : 0), 1 + t, false); if constexpr (t <= 4) KSTAMP(1 + t); });
    KSTAMP(6);
    for (int t = ND + 2; t < NKT - 4; t += 2) {
      KTILE(0, 0, 0, false);
      KTILE(1, 0, 0, false);
    }
    KTILE(0, 0, 0, false);
    KTILE(1, 0, 0, true);
    KSTAMP(7);
    KTILE(0, 0, 0, false); KSTAMP(8);
    rc = __builtin_amdgcn_make_buffer_rsrc(p.out + (size_t)((cur & 15) * 256 + wr * 128) * N + (cur >> 4) * 256 + wc * 64, 0, 128 * N * 4, 0x00020000);
    KTILE(1, 1, 0, false); KSTAMP(9);
    STAMP();
    STAMP();
    if (!has_next) break;
    cur = nxt;
  }
  static_for<SL.tail, 32>([&](auto uc) __attribute__((always_inline)) { constexpr int u = decltype(uc)::value; q_g0<u & 1, u>(scale); q_store<u & 1, u, 0>(rc); });
  static_for<0, 32>([&](auto uc) __attribute__((always_inline)) { constexpr int u = decltype(uc)::value; q_read<u & 1, A_ACC, u>(); q_scale<u & 1>(scale); q_clamp<u & 1>(); q_store<u & 1, u, 1>(rc); });
  asm volatile("s_waitcnt vmcnt(0) ; end of owned registers" ::: "memory");
  STAMP();
}

static int g_pad = 0;
static void build_schedule(Params& p) {
  for (int b = NCOMM; b < GRID; ++b) {
    const int x = b & 7, c = (b >> 3) - NCOMM / 8, xr = x & 3, xc = x >> 2, W = NGEMM / 8;
    auto tile_of = [&](int o) {
      int ri, ci;
      if (o < 16) { ri = (o >> 3) * 2 + (o & 1); ci = (o & 7) >> 1; } else { ri = o & 3; ci = o >> 2; }
      return (u16)((xr + 4 * ri) | ((ci * 2 + xc) << 4));
    };
    int k = 0; u16* t = (u16*)p.tile[b - NCOMM];
    const int extra = 128 - 4 * W;
    t[k++] = tile_of(c);
    if (c < extra) t[k++] = tile_of(W + c);
    for (int r = 0; r < 3; ++r) t[k++] = tile_of(W + extra + r * W + (c + W - extra) % W);
    while (k <= MAXT) t[k++] = 0xFFFF;
  }
  int s = 0;
  auto put = [&](int panel, int q) { ((u16*)p.stream)[s++] = (u16)(panel << 2 | q); };
  for (int q = 0; q < 4; ++q) for (int i = 0; i < 8; ++i) { put(i, q); put(NPA + i, q); }
  for (int q = 0; q < 4; ++q) { for (int i = 8; i < 16; ++i) put(i, q); for (int i = 8; i < 20; ++i) put(NPA + i, q); }
  const int lim[4] = {20, 34, 50, 64};
  for (int r = 0; r < 3; ++r) for (int q = 0; q < 4; ++q) for (int i = lim[r]; i < lim[r + 1]; ++i) put(NPA + i, q);
}

extern "C" void kernel_launch(int rank, int nranks, ncclComm_t comm, hipStream_t stream,
                              void* const* d_in, const int* in_sizes, int n_in,
                              void* d_out, int out_size, const int* in_dtypes, int out_dtype,
                              void* const* d_out_all, void* const* d_ws_all, size_t ws_bytes) {
  if (nranks != NR || n_in != 4 || ws_bytes < WS_TOTAL || (unsigned)rank >= (unsigned)NR ||
      in_sizes[0] != M * KS || in_sizes[1] != KS * N || out_size != M * N) { fprintf(stderr, "unexpected problem shape\n"); return; }
  Params p;
  p.x = (const u8*)d_in[0]; p.w = (const u8*)d_in[1]; p.sx = (const float*)d_in[2]; p.sw = (const float*)d_in[3];
  p.out = (float*)d_out; p.pad = g_pad; p.pad2 = 0;
  for (int i = 0; i < NR; ++i) p.peer[i] = (u8*)d_ws_all[(rank + 1 + i) % NR];
  build_schedule(p);
  hipLaunchKernelGGL(fused_kernel, dim3(GRID), dim3(256), 0, stream, p);
  HIP_CHECK(hipGetLastError());
}
```

```cpp
#include <hip/hip_runtime.h>
#include <rccl/rccl.h>
#include <cstdio>

#define HIP_CHECK(expr) do { hipError_t _e = (expr); if (_e != hipSuccess) { \
    fprintf(stderr, "HIP error %s at %s:%d\n", hipGetErrorString(_e), __FILE__, __LINE__); \
    return; } } while (0)

#define LAS __attribute__((address_space(3)))
typedef unsigned char u8;
typedef unsigned short u16;
typedef int v8i __attribute__((ext_vector_type(8)));
typedef int v4i __attribute__((ext_vector_type(4)));
typedef unsigned v4u __attribute__((ext_vector_type(4)));
typedef float v4f __attribute__((ext_vector_type(4)));
typedef float v2f __attribute__((ext_vector_type(2)));
typedef __amdgpu_buffer_rsrc_t rsrc_t;

#define HV 28
#define DW 3
#define CWV 1
#define SLP 2
#define NDIRECT 4
constexpr int M = 4096, N = 16384, K = 4096, NR = 8, KS = K / NR;
constexpr int NKT = 32;
constexpr int BLK = 32768;
constexpr int NPA = 16, NPB = 64, NPANEL = NPA + NPB;
constexpr unsigned KT_BYTES = NPANEL * BLK;
constexpr unsigned STG_BYTES = NKT * KT_BYTES;
constexpr int GRID = 256, NCOMM = 16, NGEMM = GRID - NCOMM, CW = CWV, NCWAVES = NCOMM * CW, NGWAVES = NGEMM * 4;
constexpr int WAITER = 64;
constexpr int MAXT = 5;
constexpr int NPIECE = NPANEL * 4, NUNIT = NPIECE * 8;
constexpr int SC = 20, NOUT = 256, NTOT = NUNIT + NOUT;
constexpr size_t WS_CNT = 0;
constexpr size_t WS_TICKET = GRID * 128;
constexpr size_t WS_FLAG = 65536;
constexpr unsigned TMO_OFF = NPANEL * 128;
constexpr unsigned OUT_OFF = TMO_OFF + 128;
constexpr unsigned SH_OFF = OUT_OFF + 128;
constexpr unsigned FLAG_BYTES = SH_OFF + 128;
constexpr size_t WS_DBG = 131072;
constexpr size_t WS_STG = 1u << 20;
constexpr size_t WS_TOTAL = WS_STG + 2 * (size_t)STG_BYTES;
constexpr unsigned DEADLINE = 50000000u;
constexpr int LDS_A = 0, LDS_B = 2 * BLK, LDS_X = 4 * BLK, LDS_BYTES = LDS_X + 64;

struct Params {
  const u8* x; const u8* w; const float* sx; const float* sw; float* out;
  u8* peer[NR];
  float* share[NR];
  int pad; int pad2;
  unsigned tile[NGEMM][(MAXT + 1) / 2];
  unsigned stream[NPIECE / 2];
};
__device__ __forceinline__ int half_of(const unsigned* t, int i) { return (int)((t[i >> 1] >> ((i & 1) * 16)) & 0xFFFFu); }

#define STR2(x) #x
#define STR(x) STR2(x)
#define WAIT_V(n) asm volatile("s_waitcnt vmcnt(" STR(n) ")" ::: "memory")
#define WAIT_L(n) asm volatile("s_waitcnt lgkmcnt(" STR(n) ")" ::: "memory")
#define BAR __builtin_amdgcn_s_barrier()
#define FBAR do { asm volatile("" ::: "memory"); __builtin_amdgcn_s_barrier(); asm volatile("" ::: "memory"); } while (0)
#define SCHED __builtin_amdgcn_sched_barrier(0)
#define STAMP() do {} while (0)
#define GSTAMP() do {} while (0)

#define DMAV 0
#define PU 2
constexpr int dma_slot(int k) {
  if (DMAV == 0) return 22 + 4 * k;
  if (DMAV == 1) return k < 12 ? 22 + 2 * k : 50 + 8 * (k - 12);
  if (DMAV == 2) return k < 12 ? 22 + 3 * k - (k & 1) : 58 + 4 * (k - 12) + (k > 13 ? 4 : 0);
  return 0;
}
constexpr int POLL_SLOT = 17;
constexpr int ND = 32 / PU;
struct Slots { int dma[64], mx[128], st[128], park[64], tail; bool ok; };
constexpr Slots make_slots() {
  Slots t{};
  for (int i = 0; i < 64; ++i) t.dma[i] = t.park[i] = -1;
  for (int i = 0; i < 128; ++i) t.mx[i] = t.st[i] = -1;
  for (int k = 0; k < 16; ++k) t.dma[dma_slot(k) % 64] = k;
  t.ok = dma_slot(0) >= 22 && dma_slot(12) >= 48 && dma_slot(15) < 64 + 20 && t.dma[POLL_SLOT] < 0;
  int fr[64] = {}, nf = 0;
  for (int w = 23; w < 78; ++w) if (t.dma[w % 64] < 0) fr[nf++] = w;
  for (int u = 0; u < 32; ++u) {
    const int x = fr[u * nf / 32], y = fr[(u + 1) * nf / 32 - 1];
    t.mx[x] = u; t.st[y] = u; t.tail += y < 64;
    if (x < 21 + 2 * (u >> 2) || x >= 64 + 2 * (u >> 2)) t.ok = false;
  }
  int c = 0;
  for (int w = 15; w < 64 && c < 32; ++w) if (t.dma[w] < 0) {
    t.park[w] = ((c >> 1) & 7) * 4 + (c >> 4) * 2 + (c & 1);
    if (w >= 32 + (c >> 4) * 16 + 2 * ((c >> 1) & 7)) t.ok = false;
    ++c;
  }
  if (c != 32) t.ok = false;
  return t;
}
constexpr Slots SL = make_slots();
static_assert(SL.ok, "drain schedule violates a deadline");
constexpr int dma_at(int s) { return SL.dma[s]; }
constexpr int g0_mx(int mode, int s) { return mode == 1 || mode == 2 ? SL.mx[(mode - 1) * 64 + s] : -1; }
constexpr int g0_st(int mode, int s) { return mode == 1 || mode == 2 ? SL.st[(mode - 1) * 64 + s] : -1; }
constexpr int park_frag_at(int mode, int s) { return mode == 2 ? SL.park[s] : -1; }
constexpr int g1_op(int mode, int s, int op) {
  if (mode < 3) return -1;
  const int q = s & 15, np = s >> 4;
  int k = op == 3 ? (q == 8 ? 0 : q == 12 ? 1 : -1) : (q == 1 + 2 * op ? 0 : q == 7 + 2 * op ? 1 : -1);
  if (DMAV != 0 && op == 3) k = q == 13 ? 0 : q == 15 ? 1 : -1;
  if (k < 0) return -1;
  if (PU == 8) return (mode - 3) * 8 + np * 2 + k;
  if (PU == 4) return np == 1 || np == 2 ? (mode - 3) * 4 + (np - 1) * 2 + k : -1;
  return np == 2 ? (mode - 3) * 2 + k : -1;
}
constexpr int nvm(int mode, int prev, int lo, int hi) {
  int c = 0;
  for (int s = lo + 1; s < hi; ++s) {
    const int q = (s + 128) % 64, md = s >= 0 ? mode : s >= -64 ? prev : 0;
    c += (dma_at(q) >= 0) + (q == POLL_SLOT) + (g0_st(md, q) >= 0) + (g1_op(md, q, 3) >= 0);
  }
  return c;
}
template <int V> struct IC { static constexpr int value = V; };
template <int I, int E, class F> __device__ __forceinline__ void static_for(F&& f) { if constexpr (I < E) { f(IC<I>{}); static_for<I + 1, E>(f); } }

enum : int { V_XWR = 15, V_ARD = 16, V_BRD, V_XRD, V_VL, V_CO, V_FVO, V_PV, V_TMP, V_T = 24, V_A = 32, V_B = 96, V_ACC = 128, A_ACC = 0, A_PARK = 128 };
#define C10(p, t) #p #t "0", #p #t "1", #p #t "2", #p #t "3", #p #t "4", #p #t "5", #p #t "6", #p #t "7", #p #t "8", #p #t "9"
#define C20_255(p) C10(p, 2), C10(p, 3), C10(p, 4), C10(p, 5), C10(p, 6), C10(p, 7), C10(p, 8), C10(p, 9), C10(p, 10), C10(p, 11), C10(p, 12), C10(p, 13), \
    C10(p, 14), C10(p, 15), C10(p, 16), C10(p, 17), C10(p, 18), C10(p, 19), C10(p, 20), C10(p, 21), C10(p, 22), C10(p, 23), C10(p, 24), \
    #p "250", #p "251", #p "252", #p "253", #p "254", #p "255"
#define OWNED_REGS "v14", "v15", "v16", "v17", "v18", "v19", C20_255(v), C10(a, ), C10(a, 1), C20_255(a)
template <int m, int n, int slot, bool zero> __device__ __forceinline__ void mfma() {
  constexpr int c = (n < 4 ? V_ACC : A_ACC) + m * 16 + (n & 3) * 4, a = V_A + m * 8, b = V_B + slot * 8;
  if constexpr (n < 4 && !zero) asm volatile("v_mfma_f32_16x16x128_f8f6f4 v[%c0:%c1], v[%c2:%c3], v[%c4:%c5], v[%c0:%c1] cbsz:1 blgp:1" :: "n"(c), "n"(c + 3), "n"(a), "n"(a + 7), "n"(b), "n"(b + 7));
  if constexpr (n < 4 && zero) asm volatile("v_mfma_f32_16x16x128_f8f6f4 v[%c0:%c1], v[%c2:%c3], v[%c4:%c5], 0 cbsz:1 blgp:1" :: "n"(c), "n"(c + 3), "n"(a), "n"(a + 7), "n"(b), "n"(b + 7));
  if constexpr (n >= 4 && !zero) asm volatile("v_mfma_f32_16x16x128_f8f6f4 a[%c0:%c1], v[%c2:%c3], v[%c4:%c5], a[%c0:%c1] cbsz:1 blgp:1" :: "n"(c), "n"(c + 3), "n"(a), "n"(a + 7), "n"(b), "n"(b + 7));
  if constexpr (n >= 4 && zero) asm volatile("v_mfma_f32_16x16x128_f8f6f4 a[%c0:%c1], v[%c2:%c3], v[%c4:%c5], 0 cbsz:1 blgp:1" :: "n"(c), "n"(c + 3), "n"(a), "n"(a + 7), "n"(b), "n"(b + 7));
}
template <int dst, int adr, int off, int sw> __device__ __forceinline__ void ldfrag() {
  asm volatile("ds_read_b128 v[%c0:%c1], v%c4 offset:%c5\n\tds_read_b128 v[%c2:%c3], v%c4 offset:%c6"
               :: "n"(dst), "n"(dst + 3), "n"(dst + 4), "n"(dst + 7), "n"(adr), "n"(off + sw * 1024), "n"(off + 1024 - sw * 1024) : "memory");
}
template <int m, int slot> __device__ __forceinline__ void ld_a() { ldfrag<V_A + m * 8, V_ARD, slot * BLK + m * 2048, 0>(); }
template <int r, int slot, int n> __device__ __forceinline__ void ld_b() { ldfrag<V_B + r * 8, V_BRD, slot * BLK + (n >> 2) * 16384 + (n & 3) * 2048, (n >> 1) & 1>(); }
__device__ __forceinline__ void set_m0(unsigned v) { asm volatile("s_mov_b32 m0, %0" :: "s"(v) : "memory"); }
template <int k> __device__ __forceinline__ void dma(rsrc_t rs, unsigned so) {
  asm volatile("buffer_load_dwordx4 v%c0, %1, %2 offen offset:%c3 lds" :: "n"(V_VL), "s"(rs), "s"(so), "n"(k * 1024) : "memory");
}
template <int n> __device__ __forceinline__ void wait_v() { asm volatile("s_waitcnt vmcnt(%c0)" :: "n"(n > 63 ? 63 : n) : "memory"); }
template <int n> __device__ __forceinline__ void wait_l() { asm volatile("s_waitcnt lgkmcnt(%c0)" :: "n"(n) : "memory"); }
template <int off> __device__ __forceinline__ void x_read() { asm volatile("ds_read_b32 v%c0, v%c1 offset:%c2" :: "n"(V_TMP), "n"(V_XRD), "n"(off) : "memory"); }
template <int nl> __device__ __forceinline__ unsigned x_value() { unsigned r; asm volatile("s_waitcnt lgkmcnt(%c1)\n\tv_readfirstlane_b32 %0, v%c2" : "=s"(r) : "n"(nl), "n"(V_TMP) : "memory"); return r; }
template <int off> __device__ __forceinline__ void x_write(unsigned v) {
  asm volatile("v_mov_b32 v%c0, %3\n\tds_write_b32 v%c1, v%c0 offset:%c2" :: "n"(V_TMP), "n"(V_XWR), "n"(off), "s"(v) : "memory");
}
__device__ __forceinline__ unsigned pick(unsigned cand, unsigned& done) {
  unsigned t;
  asm volatile("s_and_b32 %0, %2, 0x7f7f7f7f\n\ts_cselect_b32 %0, %0, %2\n\ts_ff1_i32_b32 %0, %0\n\ts_bitset1_b32 %1, %0\n\ts_mul_i32 %0, %0, %3" : "=&s"(t), "+s"(done) : "s"(cand), "n"(KT_BYTES) : "scc");
  return t;
}
__device__ __forceinline__ void poll(rsrc_t rf, unsigned so) { asm volatile("buffer_load_dword v%c0, v%c1, %2, %3 offen sc0 sc1" :: "n"(V_PV), "n"(V_FVO), "s"(rf), "s"(so) : "memory"); }
template <int nv> __device__ __forceinline__ unsigned poll_mask(unsigned need) {
  unsigned long long b; asm volatile("s_waitcnt vmcnt(%c1)\n\tv_cmp_le_u32 %0, %2, v%c3" : "=s"(b) : "n"(nv > 63 ? 63 : nv), "s"(need), "n"(V_PV) : "memory");
  return (unsigned)b & (unsigned)(b >> 32);
}
__device__ __forceinline__ void aim_poll(int t) {
  const unsigned lb = (unsigned)(NPA - 1 + (t >> 4)) * 128u, la = (unsigned)(t & 15) * 128u - lb;
  asm volatile("v_lshrrev_b32 v%c0, 2, v%c1\n\tv_add_u32 v%c0, %2, v%c0\n\ts_mov_b64 exec, 0xffffffff\n\tv_add_u32 v%c0, %3, v%c0\n\ts_mov_b64 exec, -1" :: "n"(V_FVO), "n"(V_VL), "s"(lb), "s"(la));
}
template <int t, int u> __device__ __forceinline__ void q_g0(unsigned scale) {
  constexpr int d = V_T + t * 4, c = V_ACC + (u >> 2) * 16 + (u & 3);
  asm volatile("v_mul_f32 v%c0, %8, v%c4\n\tv_mul_f32 v%c1, %8, v%c5\n\tv_mul_f32 v%c2, %8, v%c6\n\tv_mul_f32 v%c3, %8, v%c7\n\t"
               "v_maximum3_f32 v%c0, v%c0, 0, 0\n\tv_maximum3_f32 v%c1, v%c1, 0, 0\n\tv_maximum3_f32 v%c2, v%c2, 0, 0\n\tv_maximum3_f32 v%c3, v%c3, 0, 0"
               :: "n"(d), "n"(d + 1), "n"(d + 2), "n"(d + 3), "n"(c), "n"(c + 4), "n"(c + 8), "n"(c + 12), "s"(scale));
}
template <int t, int base, int u> __device__ __forceinline__ void q_read() {
  constexpr int d = V_T + t * 4, c = base + (u >> 2) * 16 + (u & 3);
  asm volatile("v_accvgpr_read_b32 v%c0, a%c4\n\tv_accvgpr_read_b32 v%c1, a%c5\n\tv_accvgpr_read_b32 v%c2, a%c6\n\tv_accvgpr_read_b32 v%c3, a%c7"
               :: "n"(d), "n"(d + 1), "n"(d + 2), "n"(d + 3), "n"(c), "n"(c + 4), "n"(c + 8), "n"(c + 12));
}
template <int t> __device__ __forceinline__ void q_scale(unsigned scale) {
  constexpr int d = V_T + t * 4;
  asm volatile("v_mul_f32 v%c0, %4, v%c0\n\tv_mul_f32 v%c1, %4, v%c1\n\tv_mul_f32 v%c2, %4, v%c2\n\tv_mul_f32 v%c3, %4, v%c3" :: "n"(d), "n"(d + 1), "n"(d + 2), "n"(d + 3), "s"(scale));
}
template <int t> __device__ __forceinline__ void q_clamp() {
  constexpr int d = V_T + t * 4;
  asm volatile("v_maximum3_f32 v%c0, v%c0, 0, 0\n\tv_maximum3_f32 v%c1, v%c1, 0, 0\n\tv_maximum3_f32 v%c2, v%c2, 0, 0\n\tv_maximum3_f32 v%c3, v%c3, 0, 0" :: "n"(d), "n"(d + 1), "n"(d + 2), "n"(d + 3));
}
template <int t, int u, int g, bool through = false> __device__ __forceinline__ void q_store(rsrc_t rc) {
  constexpr int d = V_T + t * 4;
  if constexpr (through) asm volatile("buffer_store_dwordx4 v[%c0:%c1], v%c2, %3, %4 offen offset:%c5 sc0 sc1" :: "n"(d), "n"(d + 3), "n"(V_CO), "s"(rc), "s"(((u >> 2) * 16 + (u & 3)) * N * 4), "n"(g * 512) : "memory");
  else asm volatile("buffer_store_dwordx4 v[%c0:%c1], v%c2, %3, %4 offen offset:%c5 nt" :: "n"(d), "n"(d + 3), "n"(V_CO), "s"(rc), "s"(((u >> 2) * 16 + (u & 3)) * N * 4), "n"(g * 512) : "memory");
}
template <int f> __device__ __forceinline__ void park_frag() {
  constexpr int d = A_PARK + f * 4, c = A_ACC + f * 4;
  asm volatile("v_accvgpr_mov_b32 a%c0, a%c4\n\tv_accvgpr_mov_b32 a%c1, a%c5\n\tv_accvgpr_mov_b32 a%c2, a%c6\n\tv_accvgpr_mov_b32 a%c3, a%c7"
               :: "n"(d), "n"(d + 1), "n"(d + 2), "n"(d + 3), "n"(c), "n"(c + 1), "n"(c + 2), "n"(c + 3));
}

#define KSTAMP(i) do {} while (0)
struct Raw { unsigned r[16]; };
struct Unit { v4u o[4]; unsigned off, so[4], flag; };
constexpr unsigned SH_BYTES = 255u * N * 4u + 4096u;

__device__ __forceinline__ void unit_load(const Params& p, int n, int lane, Raw& d) {
  const int piece = half_of(p.stream, n >> 3), u = n & 7, panel = piece >> 2, q = piece & 3;
  const int g = lane >> 4, c = lane & 15;
  const bool isx = panel < NPA;
  const rsrc_t rs = __builtin_amdgcn_make_buffer_rsrc((void*)(isx ? p.x : p.w), 0, isx ? M * KS : KS * N, 0x00020000);
  const unsigned vo = isx ? (unsigned)(c * KS + g * 16) : (unsigned)(g * 16 * N + 4 * c);
  const unsigned so = isx ? (unsigned)((panel * 256 + u * 32) * KS + q * 128) : (unsigned)((q * 128 + (u & 1) * 64) * N + (panel - NPA) * 256 + (u >> 1) * 64);
  const unsigned sa = isx ? 4u : (unsigned)N, sb = isx ? 64u : 4u * N, sc = isx ? 16u * KS : 8u * N;
#pragma unroll
  for (int j = 0; j < 16; ++j) d.r[j] = __builtin_amdgcn_raw_buffer_load_b32(rs, vo, so + (j & 3) * sa + ((j >> 2) & 1) * sb + (j >> 3) * sc, 0);
}
__device__ __forceinline__ void unit_pack(const Params& p, int n, int lane, const Raw& raw, Unit& t) {
  const int piece = half_of(p.stream, n >> 3), u = n & 7, panel = piece >> 2, q = piece & 3;
  t.off = (unsigned)q * (8u * KT_BYTES) + (unsigned)panel * BLK + (unsigned)lane * 16u;
  t.flag = (unsigned)(panel * 128 + q * 32);
  t.so[0] = 0;
  if (panel < NPA) {
#pragma unroll
    for (int i = 0; i < 4; ++i) t.o[i] = (v4u){raw.r[i * 4], raw.r[i * 4 + 1], raw.r[i * 4 + 2], raw.r[i * 4 + 3]};
    t.off += u * 4096; t.so[1] = 1024; t.so[2] = 2048; t.so[3] = 3072;
  } else {
#pragma unroll
    for (int d = 0; d < 4; ++d) {
      const unsigned r0 = raw.r[4 * d], r1 = raw.r[4 * d + 1], r2 = raw.r[4 * d + 2], r3 = raw.r[4 * d + 3];
      const unsigned t0 = __builtin_amdgcn_perm(r1, r0, 0x05010400u), t1 = __builtin_amdgcn_perm(r1, r0, 0x07030602u);
      const unsigned t2 = __builtin_amdgcn_perm(r3, r2, 0x05010400u), t3 = __builtin_amdgcn_perm(r3, r2, 0x07030602u);
      t.o[0][d] = __builtin_amdgcn_perm(t2, t0, 0x05040100u); t.o[1][d] = __builtin_amdgcn_perm(t2, t0, 0x07060302u);
      t.o[2][d] = __builtin_amdgcn_perm(t3, t1, 0x05040100u); t.o[3][d] = __builtin_amdgcn_perm(t3, t1, 0x07060302u);
    }
    t.off += (u >> 1) * 8192 + (u & 1) * 1024; t.so[1] = 2048; t.so[2] = (u & 1) ? 4096 - 1024 : 4096 + 1024; t.so[3] = t.so[2] + 2048;
  }
}

template <int NW> __device__ __forceinline__ void wait_local(u8* myws, unsigned off, unsigned step, unsigned want, unsigned call) {
  unsigned* const f = (unsigned*)(myws + WS_FLAG);
  const unsigned t0 = (unsigned)__builtin_amdgcn_s_memrealtime();
  for (;;) {
    bool low = false;
#pragma unroll
    for (int i = 0; i < NW; ++i) low |= __hip_atomic_load(f + (off + i * step) / 4, __ATOMIC_RELAXED, __HIP_MEMORY_SCOPE_SYSTEM) < want;
    if (__builtin_amdgcn_ballot_w64(low) == 0) break;
    if (__builtin_amdgcn_readfirstlane(__hip_atomic_load(f + TMO_OFF / 4, __ATOMIC_RELAXED, __HIP_MEMORY_SCOPE_SYSTEM)) == call || (unsigned)__builtin_amdgcn_s_memrealtime() - t0 > DEADLINE) {
      __hip_atomic_store(f + TMO_OFF / 4, call, __ATOMIC_RELAXED, __HIP_MEMORY_SCOPE_SYSTEM); break; }
    __builtin_amdgcn_s_sleep(32);
  }
}

__global__ __launch_bounds__(256, 1) void fused_kernel(const Params p) {
  __shared__ __attribute__((aligned(16))) u8 shm[LDS_BYTES];
  LAS u8* lds = (LAS u8*)shm;
  const int tid = threadIdx.x, wid = __builtin_amdgcn_readfirstlane(tid >> 6), lane = tid & 63;
  u8* const myws = p.peer[NR - 1];
  STAMP();
  unsigned* const cnt = (unsigned*)(myws + WS_CNT + blockIdx.x * 128);
  if (blockIdx.x < NCOMM) {
    volatile LAS unsigned* const stage = (volatile LAS unsigned*)(lds + LDS_X);
    const int n0 = wid * NCOMM + blockIdx.x;
    Raw raw;
    const unsigned cv = __hip_atomic_load(cnt, __ATOMIC_RELAXED, __HIP_MEMORY_SCOPE_AGENT);
    if (wid < DW) unit_load(p, n0, lane, raw);
    if (tid == 0) *stage = 0;
    const unsigned call = (unsigned)__builtin_amdgcn_readfirstlane(cv) + 1u;
    WAIT_L(0); FBAR;
    if (wid >= DW) {
      if (wid == 3) {
        if (lane == 0) __hip_atomic_store(cnt, call, __ATOMIC_RELAXED, __HIP_MEMORY_SCOPE_AGENT);
        wait_local<NPIECE / 64>(myws, lane * 32 + 7 * 4, 64 * 32, call * 8u, call);
        *stage = 1;
        wait_local<1>(myws, SH_OFF, 0, call * 16u, call);
        *stage = 2;
      }
      return;
    }
    unsigned tab[3];
#pragma unroll
    for (int i = 0; i < 3; ++i) tab[i] = p.stream[i * 64 + lane < NPIECE / 2 ? i * 64 + lane : 0];
    rsrc_t rs[NR];
#pragma unroll
    for (int i = 0; i < NR; ++i)
      rs[i] = __builtin_amdgcn_make_buffer_rsrc(p.peer[i] + WS_STG + (size_t)(call & 1u) * STG_BYTES + (unsigned)((6 - i) & 7) * KT_BYTES, 0, STG_BYTES - 7u * KT_BYTES, 0x00020000);
    u8* const fl = p.peer[lane < NR - 1 ? lane : 0] + WS_FLAG + (unsigned)((6 - lane) & 7) * 4;
    unsigned* const ticket = (unsigned*)(myws + WS_TICKET);
    constexpr int NDW = NCOMM * DW, NSTAT = NDIRECT * NDW + NCWAVES;
    const unsigned tbase = (call - 1u) * (unsigned)(NTOT - NSTAT + NCWAVES) - (unsigned)NSTAT;
    const unsigned vl = (unsigned)lane * 16u;
#define DRAW() asm volatile("s_mov_b64 exec, 1\n\tglobal_atomic_add %0, %1, %2, %3 sc0\n\ts_mov_b64 exec, -1" : "+v"(tk) : "v"(0u), "v"(1u), "s"(ticket) : "memory")
#define PUBLISH(h) do { asm volatile("" ::: "memory"); __builtin_amdgcn_s_waitcnt(0x0F70 | ((h) & 15) | (((h) >> 4) << 14)); asm volatile("s_waitcnt vmcnt(" STR(h) ")" : "+v"(tk) :: "memory"); \
      if (lane < NR - 1 && pflag != ~0u) __hip_atomic_fetch_add((unsigned*)(fl + pflag), 1u, __ATOMIC_RELAXED, __HIP_MEMORY_SCOPE_SYSTEM); } while (0)
#define AWAIT(v) do { while (*stage < (v)) __builtin_amdgcn_s_sleep(1); } while (0)
    auto locate = [&](unsigned n, unsigned& off, unsigned& flag) {
      const unsigned pi = n >> 3, w = pi >> 1;
      const unsigned a = __builtin_amdgcn_readlane(tab[0], w & 63), b = __builtin_amdgcn_readlane(tab[1], w & 63), c = __builtin_amdgcn_readlane(tab[2], w & 63);
      const unsigned e = ((w < 64 ? a : w < 128 ? b : c) >> ((pi & 1) * 16)) & 0xFFFFu;
      off = n >= (unsigned)NUNIT ? (n - NUNIT) * (N * 4u) : (e & 3) * (8u * KT_BYTES) + (e >> 2) * BLK + (n & 7) * 4096u;
      flag = n >= (unsigned)NUNIT ? OUT_OFF : e * 32u;
    };
    unsigned tk = 0, n = (unsigned)(n0 + NDIRECT * NDW), off, flag, pflag = ~0u;
    v4u d[4];
    __builtin_amdgcn_s_waitcnt(0x0F70);
    for (int i = 0; i < NDIRECT; ++i) {
      Unit t;
      unit_pack(p, n0 + i * NDW, lane, raw, t);
      asm volatile("" : "+v"(t.o[0]), "+v"(t.o[1]), "+v"(t.o[2]), "+v"(t.o[3]));
      unit_load(p, n0 + (i + 1 < NDIRECT ? i + 1 : i) * NDW, lane, raw);
#pragma unroll
      for (int k = 0; k < 28; ++k) { const int q = k % 7, j = (k / 7 + q) & 3; __builtin_amdgcn_raw_buffer_store_b128(t.o[j], rs[q], t.off, t.so[j], 17); }
      if (i == NDIRECT - 1 && wid < CW) {
        AWAIT(1);
        locate(n, off, flag);
#pragma unroll
        for (int k = 0; k < 4; ++k) d[k] = __builtin_amdgcn_raw_buffer_load_b128(rs[7], vl + k * 1024, off, 17);
        DRAW();
      }
      PUBLISH(28); pflag = t.flag;
    }
    PUBLISH(0); pflag = ~0u;
    if (wid < CW) for (;;) {
      const v4u o[4] = {d[0], d[1], d[2], d[3]};
      const unsigned coff = off, cflag = flag;
      const unsigned nn = (unsigned)__builtin_amdgcn_readfirstlane(tk) - tbase;
      const bool more = nn < (unsigned)NTOT, sw = n < (unsigned)NUNIT && nn >= (unsigned)NUNIT;
      if (more) {
        if (sw) { AWAIT(2); rs[7] = __builtin_amdgcn_make_buffer_rsrc(p.share[7], 0, SH_BYTES, 0x00020000); }
        locate(nn, off, flag);
#pragma unroll
        for (int k = 0; k < 4; ++k) d[k] = __builtin_amdgcn_raw_buffer_load_b128(rs[7], vl + k * 1024, off, 17);
        DRAW();
      }
#pragma unroll
      for (int k = 0; k < 28; ++k) {
        const int q = k % 7, j = (k / 7 + q) & 3; __builtin_amdgcn_raw_buffer_store_b128(o[j], rs[q], vl + j * 1024, coff, 17);
        if (k == HV - 1) PUBLISH(HV);
      }
      pflag = cflag;
      if (!more) break;
      if (sw) {
#pragma unroll
        for (int i = 0; i < NR - 1; ++i) rs[i] = __builtin_amdgcn_make_buffer_rsrc(p.share[i], 0, SH_BYTES, 0x00020000);
      }
      n = nn;
    }
    PUBLISH(0);
    return;
  }
  const int gw = blockIdx.x - NCOMM;
  unsigned call;
  {
    Raw raw[3];
    const int n0 = wid * NGEMM + gw;
#pragma unroll
    for (int i = 0; i < 3; ++i) { const int n = n0 + i * NGWAVES; unit_load(p, n < NUNIT ? n : n0, lane, raw[i]); }
    call = (unsigned)__builtin_amdgcn_readfirstlane(__hip_atomic_load(cnt, __ATOMIC_RELAXED, __HIP_MEMORY_SCOPE_AGENT)) + 1u - (p.pad ? 1u : 0u);
    if (p.pad > 1) { unsigned long long t0 = __builtin_amdgcn_s_memrealtime(); while (__builtin_amdgcn_s_memrealtime() - t0 < (unsigned long long)(((gw >> 3) >> 2) * p.pad)) __builtin_amdgcn_s_sleep(8); }
    const rsrc_t rl = __builtin_amdgcn_make_buffer_rsrc(myws + WS_STG + (size_t)(call & 1u) * STG_BYTES + 7u * KT_BYTES, 0, STG_BYTES - 7u * KT_BYTES, 0x00020000);
    unsigned fo = 0;
#pragma unroll
    for (int i = 0; i < 3; ++i) {
      const int n = n0 + i * NGWAVES;
      if (n < NUNIT && !p.pad) {
        Unit t; unit_pack(p, n, lane, raw[i], t);
#pragma unroll
        for (int k = 0; k < 4; ++k) __builtin_amdgcn_raw_buffer_store_b128(t.o[k], rl, t.off, t.so[k], 17);
        fo = lane == i ? t.flag : fo;
      }
    }
    if (wid == 0) __builtin_amdgcn_fence(__ATOMIC_ACQUIRE, "");
    if (tid == 0) *(LAS float*)(lds + LDS_X + 32) = p.sx[0] * p.sw[0];
    WAIT_V(0);
    if (lane < 3 && n0 + lane * NGWAVES < NUNIT && !p.pad) __hip_atomic_fetch_add((unsigned*)(myws + WS_FLAG + fo + 7 * 4), 1u, __ATOMIC_RELAXED, __HIP_MEMORY_SCOPE_SYSTEM);
  }
  WAIT_L(0); FBAR;
  if (tid == 0) __hip_atomic_store(cnt, call, __ATOMIC_RELAXED, __HIP_MEMORY_SCOPE_AGENT);
  const int wr = wid >> 1, wc = wid & 1;
  const rsrc_t rs = __builtin_amdgcn_make_buffer_rsrc(myws + WS_STG + (size_t)(call & 1u) * STG_BYTES, 0, STG_BYTES, 0x00020000);
  const rsrc_t rf = __builtin_amdgcn_make_buffer_rsrc(myws + WS_FLAG, 0, FLAG_BYTES, 0x00020000);
  const unsigned need = call * 8u;
  const unsigned lbase = (unsigned)(size_t)lds;
  const unsigned rinit = wid ? 0xFFFFFFFFu : 0u;
  unsigned long long tlo = p.tile[gw][0] | ((unsigned long long)p.tile[gw][1] << 32); unsigned thi = p.tile[gw][2];
  asm volatile("v_mov_b32 v%c0, %5\n\tv_mov_b32 v%c1, %6\n\tv_mov_b32 v%c2, %7\n\tv_mov_b32 v%c3, %8\n\tv_mov_b32 v%c4, %9\n\tv_mov_b32 v%c10, %11"
               :: "n"(V_ARD), "n"(V_BRD), "n"(V_XRD), "n"(V_VL), "n"(V_CO),
                  "v"(lbase + LDS_A + wr * 16384 + lane * 16), "v"(lbase + LDS_B + wc * 8192 + lane * 16), "v"(lbase + LDS_X), "v"(lane * 16),
                  "v"((4 * (lane >> 4) * N + 4 * (lane & 15)) * 4), "n"(V_XWR), "v"(lbase + LDS_X + wid * 4) : OWNED_REGS, "memory");
#define M0_A(slot, h) (lbase + LDS_A + (slot) * BLK + wid * 8192 + (h) * 4096)
#define M0_B(slot, h) (lbase + LDS_B + (slot) * BLK + (h) * 16384 + wid * 4096)
#define SRC_A(t) ((unsigned)(((t) & 15) * BLK + wid * 8192))
#define SRC_B(t) ((unsigned)((NPA + ((t) >> 4)) * BLK + wid * 4096))
#define DIAG_WAITED() do {} while (0)
#define ENSURE(cand) do { if (__builtin_expect((cand) == 0, 0)) { \
      const unsigned _t0 = (unsigned)__builtin_amdgcn_s_memrealtime(); \
      for (;;) { unsigned _w; \
        poll(rf, 0u); asm volatile("buffer_load_dword v%c0, off, %1, %2 sc0 sc1" :: "n"(V_TMP), "s"(rf), "s"(TMO_OFF) : "memory"); \
        rdy = poll_mask<0>(need); cand = rdy & ~done; \
        if (cand) break; \
        asm volatile("v_readfirstlane_b32 %0, v%c1" : "=s"(_w) : "n"(V_TMP)); \
        if (_w == call || (unsigned)__builtin_amdgcn_s_memrealtime() - _t0 > DEADLINE) { \
          asm volatile("v_mov_b32 v%c0, %1\n\tbuffer_store_dword v%c0, off, %2, %3 sc0 sc1\n\ts_nop 1" :: "n"(V_TMP), "s"(call), "s"(rf), "s"(TMO_OFF) : "memory"); \
          x_write<32>(0x7FC00000u); cand = ~done; break; } \
        __builtin_amdgcn_s_sleep(SLP); } \
      DIAG_WAITED(); __builtin_amdgcn_s_waitcnt(0xC07F); } } while (0)
#define POLL_OFF (rdy == 0xFFFFFFFFu ? 0x40000u : 0u)

  int cur = (int)(tlo & 0xFFFFu);
  unsigned done = 0, rdy = rinit;
  unsigned sA = SRC_A(cur), sB = SRC_B(cur);
  unsigned bsrc;
  aim_poll(cur);
  {
    unsigned c_ = rdy; ENSURE(c_); x_write<0>(pick(c_, done));
    c_ = rdy & ~done; ENSURE(c_); x_write<16>(pick(c_, done));
    WAIT_L(0); FBAR;
    GSTAMP();
    x_read<0>(); const unsigned k0 = x_value<0>();
    x_read<16>(); const unsigned k1 = x_value<0>();
    poll(rf, POLL_OFF);
#define DMA4(m0v, so) do { set_m0(m0v); asm volatile("s_nop 0"); dma<0>(rs, so); dma<1>(rs, so); dma<2>(rs, so); dma<3>(rs, so); } while (0)
    DMA4(M0_A(0, 0), sA + k0); DMA4(M0_A(0, 1), sA + k0 + 4096); DMA4(M0_B(0, 0), sB + k0); DMA4(M0_B(0, 1), sB + k0 + 16384);
    DMA4(M0_A(1, 0), sA + k1); DMA4(M0_A(1, 1), sA + k1 + 4096); DMA4(M0_B(1, 0), sB + k1);
    if (dma_slot(12) < 64) DMA4(M0_B(1, 1), sB + k1 + 16384);
    bsrc = sB + k1;
    WAIT_V(0); FBAR;
    ld_b<0, 0, 0>(); ld_a<0, 0>(); ld_a<1, 0>(); ld_b<1, 0, 1>(); ld_a<2, 0>(); ld_a<3, 0>(); ld_a<4, 0>(); ld_a<5, 0>(); ld_a<6, 0>(); ld_a<7, 0>();
  }
#define KTILE(par, MODE, PREV, SW) do { \
    unsigned asrc, cand, jp; \
    static_for<0, 64>([&](auto gsc) __attribute__((always_inline)) { \
      constexpr int gs = decltype(gsc)::value, np = gs >> 4, s = gs & 15, m = s >> 1, e = s & 1; \
      constexpr int cb = (np & 1) * 2, nb = 2 - cb, z = e ^ (m & 1); \
      if constexpr (gs == 16) { wait_v<nvm(MODE, PREV, dma_slot(15) - 128, 16)>(); wait_l<0>(); BAR; } \
      if constexpr (gs == 48) { wait_v<nvm(MODE, PREV, dma_slot(11) - 64, 48)>(); wait_l<0>(); BAR; } \
      if constexpr (gs <= 1) wait_l<14>(); \
      if constexpr (gs == 4 || gs == 6) wait_l<12>(); \
      if constexpr (gs == 8) wait_l<10>(); \
      if constexpr (gs == 10 || gs == 12 || gs == 14) wait_l<19 - gs>(); \
      if constexpr (gs == 32 || gs == 33) wait_l<2>(); \
      mfma<m, np * 2 + z, cb + z, (MODE) == 2>(); SCHED; \
      if constexpr (gs == 16) x_read<(par) * 16>(); \
      if constexpr (gs == 20) { const unsigned ko = x_value<2>(); bsrc = sB + ko; asrc = sA + ko; } \
      if constexpr ((MODE) == 1 && gs == 20) x_read<32>(); \
      if constexpr ((MODE) == 1 && gs == 22) scale = x_value<2>(); \
      if constexpr (s == 0 || s == 4) { if constexpr (np < 3) ld_b<nb + (s >> 2), par, np * 2 + 2 + (s >> 2)>(); else ld_b<nb + (s >> 2), (par) ^ 1, (s >> 2)>(); } \
      if constexpr (np == 3 && e == 1) ld_a<m, (par) ^ 1>(); \
      if constexpr (gs == 1) rdy |= poll_mask<nvm(MODE, PREV, POLL_SLOT - 64, 1)>(need); \
      if constexpr (gs == 3) { cand = rdy & ~done; ENSURE(cand); } \
      if constexpr (gs == 7) jp = pick(cand, done); \
      if constexpr (gs == 9) x_write<(par) * 16>(jp); \
      if constexpr (gs == 11 && (SW)) { done = 0; rdy = rinit; aim_poll(nxt); } \
      if constexpr (gs == POLL_SLOT) poll(rf, POLL_OFF); \
      if constexpr (dma_at((gs + 1) % 64) >= 0 && dma_at((gs + 1) % 64) % 4 == 0) { constexpr int k = dma_at((gs + 1) % 64), sl = dma_slot(k) > 64 ? (par) ^ 1 : (par); \
        set_m0(k < 8 ? M0_A(sl, k >> 2) : M0_B(sl, (k >> 2) - 2)); } \
      if constexpr (dma_at(gs) >= 0) { constexpr int k = dma_at(gs); dma<k & 3>(rs, k < 8 ? asrc + (k >> 2) * 4096 : bsrc + ((k >> 2) - 2) * 16384); } \
      if constexpr (gs == 29 && (SW)) { sA = SRC_A(nxt); sB = SRC_B(nxt); } \
      if constexpr (g0_mx(MODE, gs) >= 0) q_g0<g0_mx(MODE, gs) & 1, g0_mx(MODE, gs)>(scale); \
      if constexpr (g0_st(MODE, gs) >= 0) q_store<g0_st(MODE, gs) & 1, g0_st(MODE, gs), 0>(rc); \
      if constexpr (park_frag_at(MODE, gs) >= 0) park_frag<park_frag_at(MODE, gs)>(); \
      if constexpr (g1_op(MODE, gs, 0) >= 0) q_read<g1_op(MODE, gs, 0) & 1, A_PARK, g1_op(MODE, gs, 0)>(); \
      if constexpr (g1_op(MODE, gs, 1) >= 0) q_scale<g1_op(MODE, gs, 1) & 1>(scale); \
      if constexpr (g1_op(MODE, gs, 2) >= 0) q_clamp<g1_op(MODE, gs, 2) & 1>(); \
      if constexpr (g1_op(MODE, gs, 3) >= 0) q_store<g1_op(MODE, gs, 3) & 1, g1_op(MODE, gs, 3), 1>(rc); \
      SCHED; \
    }); } while (0)
  unsigned scale = 0;
  rsrc_t rc = __builtin_amdgcn_make_buffer_rsrc(p.out, 0, 0, 0x00020000);
  for (int r = 0;; ++r) {
    tlo = (tlo >> 16) | ((unsigned long long)thi << 48); thi = (thi >> 16) | 0xFFFF0000u;
    const int nx = (int)(tlo & 0xFFFFu);
    const bool has_next = nx != 0xFFFF;
    const int nxt = has_next ? nx : cur;
    KSTAMP(0);
    KTILE(0, 2, 1, false); KSTAMP(1);
    static_for<1, ND + 2>([&](auto tc) __attribute__((always_inline)) { constexpr int t = decltype(tc)::value;
      KTILE(t & 1, (t <= ND ? 2 + t : 0), 1 + t, false); if constexpr (t <= 4) KSTAMP(1 + t); });
    KSTAMP(6);
    for (int t = ND + 2; t < NKT - 4; t += 2) {
      KTILE(0, 0, 0, false);
      KTILE(1, 0, 0, false);
    }
    KTILE(0, 0, 0, false);
    KTILE(1, 0, 0, true);
    KSTAMP(7);
    KTILE(0, 0, 0, false); KSTAMP(8);
    const bool shared = (cur & 15) >= 8 && (cur >> 6) == SC / 4;
    float* const cp = p.out + (size_t)((cur & 15) * 256 + wr * 128) * N + (cur >> 4) * 256 + wc * 64;
    rc = __builtin_amdgcn_make_buffer_rsrc(cp, 0, shared ? 0 : 128 * N * 4, 0x00020000);
    KTILE(1, 1, 0, false); KSTAMP(9);
    if (shared) {
      const rsrc_t rt = __builtin_amdgcn_make_buffer_rsrc(cp, 0, 128 * N * 4, 0x00020000);
      static_for<0, 32>([&](auto uc) __attribute__((always_inline)) { constexpr int u = decltype(uc)::value; q_g0<u & 1, u>(scale); q_store<u & 1, u, 0, true>(rt); });
      static_for<0, 32>([&](auto uc) __attribute__((always_inline)) { constexpr int u = decltype(uc)::value; q_read<u & 1, A_ACC, u>(); q_scale<u & 1>(scale); q_clamp<u & 1>(); q_store<u & 1, u, 1, true>(rt); });
      if (!p.pad) asm volatile("v_mov_b32 v%c0, 1\n\ts_waitcnt vmcnt(0)\n\ts_mov_b64 exec, 1\n\tbuffer_atomic_add v%c0, off, %1, %2 sc1\n\ts_mov_b64 exec, -1" :: "n"(V_TMP), "s"(rf), "s"(SH_OFF) : "memory");
    }
    GSTAMP();
    GSTAMP();
    if (!has_next) break;
    cur = nxt;
  }
  static_for<SL.tail, 32>([&](auto uc) __attribute__((always_inline)) { constexpr int u = decltype(uc)::value; q_g0<u & 1, u>(scale); q_store<u & 1, u, 0>(rc); });
  static_for<0, 32>([&](auto uc) __attribute__((always_inline)) { constexpr int u = decltype(uc)::value; q_read<u & 1, A_ACC, u>(); q_scale<u & 1>(scale); q_clamp<u & 1>(); q_store<u & 1, u, 1>(rc); });
  asm volatile("s_waitcnt vmcnt(0) ; end of owned registers" ::: "memory");
  if (gw == WAITER && wid == 0) {
    unsigned* const f = (unsigned*)(myws + WS_FLAG);
    const unsigned t0 = (unsigned)__builtin_amdgcn_s_memrealtime();
    bool bad = false;
    for (;;) {
      const unsigned v = __hip_atomic_load(f + OUT_OFF / 4 + (lane < NR - 1 ? lane : 0), __ATOMIC_RELAXED, __HIP_MEMORY_SCOPE_SYSTEM);
      bad = __hip_atomic_load(f + TMO_OFF / 4, __ATOMIC_RELAXED, __HIP_MEMORY_SCOPE_SYSTEM) == call || (unsigned)__builtin_amdgcn_s_memrealtime() - t0 > DEADLINE;
      if (bad || __builtin_amdgcn_ballot_w64(v < call * (unsigned)NOUT) == 0) break;
      __builtin_amdgcn_s_sleep(2);
    }
    __builtin_amdgcn_fence(__ATOMIC_ACQUIRE, "");
    if (bad && lane == 0) {
      __hip_atomic_store(f + TMO_OFF / 4, call, __ATOMIC_RELAXED, __HIP_MEMORY_SCOPE_SYSTEM);
      p.out[(size_t)(cur & 15) * 256 * N + (cur >> 4) * 256] = __builtin_nanf("");
    }
  }
}

static int g_pad = 0;
#define expt(n, d) (d)
static void build_schedule(Params& p, int rank) {
  constexpr int W = NGEMM / 8, PR = 2, PC = 3, C0 = 18;
  for (int x = 0; x < 8; ++x) {
    const int xr = x & 3, xc = x >> 2;
    u16 list[128]; int n = 0;
    for (int pass = 0; pass < 2; ++pass)
      for (int ci = 0; ci < 32; ++ci) for (int ri = 0; ri < 4; ++ri) {
        const int pm = xr + 4 * ri, pn = 2 * ci + xc;
        if ((ri < PR && ci < PC) != (pass == 0)) continue;
        if (pm >= 8 && pn / 4 == SC / 4 && pm != 8 + rank) continue;
        list[n++] = (u16)(pm | pn << 4);
      }
    const int extra = n - 4 * W;
    for (int c = 0; c < W; ++c) {
      int k = 0; u16* t = (u16*)p.tile[c * 8 + x];
      t[k++] = list[c];
      if (c < extra) t[k++] = list[W + c];
      for (int r = 0; r < 3; ++r) t[k++] = list[W + extra + r * W + (c + W - extra) % W];
      while (k <= MAXT) t[k++] = 0xFFFF;
    }
  }
  int s = 0;
  auto put = [&](int panel, int q) { ((u16*)p.stream)[s++] = (u16)(panel << 2 | q); };
  for (int q = 0; q < 4; ++q) for (int i = 0; i < 16; ++i) { if (i < 4 * PR) put(i, q); if (i < 2 * PC) put(NPA + i, q); }
  for (int q = 0; q < 4; ++q) { for (int i = 4 * PR; i < 16; ++i) put(i, q); for (int i = 2 * PC; i < C0; ++i) put(NPA + i, q); }
  const int lim[4] = {C0, 34, 50, 64};
  for (int r = 0; r < 3; ++r) for (int q = 0; q < 4; ++q) for (int i = lim[r]; i < lim[r + 1]; ++i) put(NPA + i, q);
}

extern "C" void kernel_launch(int rank, int nranks, ncclComm_t comm, hipStream_t stream,
                              void* const* d_in, const int* in_sizes, int n_in,
                              void* d_out, int out_size, const int* in_dtypes, int out_dtype,
                              void* const* d_out_all, void* const* d_ws_all, size_t ws_bytes) {
  if (nranks != NR || n_in != 4 || ws_bytes < WS_TOTAL || (unsigned)rank >= (unsigned)NR ||
      in_sizes[0] != M * KS || in_sizes[1] != KS * N || out_size != M * N) { fprintf(stderr, "unexpected problem shape\n"); return; }
  Params p;
  p.x = (const u8*)d_in[0]; p.w = (const u8*)d_in[1]; p.sx = (const float*)d_in[2]; p.sw = (const float*)d_in[3];
  p.out = (float*)d_out; p.pad = g_pad; p.pad2 = 0;
  for (int i = 0; i < NR; ++i) {
    p.peer[i] = (u8*)d_ws_all[(rank + 1 + i) % NR];
    p.share[i] = (float*)d_out_all[(rank + 1 + i) % NR] + (size_t)(8 + rank) * 256 * N + SC * 256;
  }
  build_schedule(p, rank);
  hipLaunchKernelGGL(fused_kernel, dim3(GRID), dim3(256), 0, stream, p);
  HIP_CHECK(hipGetLastError());
}
```

```cpp
#include <hip/hip_runtime.h>
#include <rccl/rccl.h>
#include <cstdio>

#define HIP_CHECK(expr) do { hipError_t _e = (expr); if (_e != hipSuccess) { \
    fprintf(stderr, "HIP error %s at %s:%d\n", hipGetErrorString(_e), __FILE__, __LINE__); \
    return; } } while (0)

#define LAS __attribute__((address_space(3)))
typedef unsigned char u8;
typedef unsigned short u16;
typedef __attribute__((address_space(1))) unsigned gu32;
typedef unsigned v4u __attribute__((ext_vector_type(4)));
typedef __amdgpu_buffer_rsrc_t rsrc_t;

constexpr int M = 4096, N = 16384, K = 4096, NR = 8, KS = K / NR;
constexpr int NKT = 32;
constexpr int BLK = 32768;
constexpr int NPA = 16, NPB = 64, NPANEL = NPA + NPB;
constexpr unsigned KT_BYTES = NPANEL * BLK;
constexpr unsigned STG_BYTES = NKT * KT_BYTES;
constexpr int GRID = 256, NCOMM = 16, NGEMM = GRID - NCOMM, NGWAVES = NGEMM * 4;
constexpr int DW = 3, NDIRECT = 4, NDW = NCOMM * DW;
constexpr int WAITER = 64;
constexpr int MAXT = 5;
constexpr int NPIECE = NPANEL * 4, NUNIT = NPIECE * 8;
constexpr int SC = 20, NSH = 3, NOUT = NSH * 64, NTOT = NUNIT + NOUT;
constexpr unsigned SH_BYTES = 255u * N * 4u + NSH * 1024u;
constexpr size_t WS_CNT = 0;
constexpr size_t WS_TICKET = GRID * 128;
constexpr size_t WS_FLAG = 65536;
constexpr unsigned TMO_OFF = NPANEL * 128;
constexpr unsigned OUT_OFF = TMO_OFF + 128;
constexpr unsigned SH_OFF = OUT_OFF + 128;
constexpr unsigned FLAG_BYTES = SH_OFF + 128;
constexpr size_t WS_STG = 1u << 20;
constexpr size_t WS_TOTAL = WS_STG + 2 * (size_t)STG_BYTES;
constexpr unsigned DEADLINE = 50000000u;
constexpr int LDS_A = 0, LDS_B = 2 * BLK, LDS_X = 4 * BLK, LDS_BYTES = LDS_X + 64;

struct Params {
  const u8* x; const u8* w; const float* sx; const float* sw; float* out;
  u8* peer[NR];
  float* share[NR];
  unsigned tile[NGEMM][(MAXT + 1) / 2];
  unsigned stream[NPIECE / 2];
};

#define STR2(x) #x
#define STR(x) STR2(x)
#define WAIT_V(n) asm volatile("s_waitcnt vmcnt(" STR(n) ")" ::: "memory")
#define WAIT_L(n) asm volatile("s_waitcnt lgkmcnt(" STR(n) ")" ::: "memory")
#define BAR __builtin_amdgcn_s_barrier()
#define FBAR do { asm volatile("" ::: "memory"); __builtin_amdgcn_s_barrier(); asm volatile("" ::: "memory"); } while (0)
#define SCHED __builtin_amdgcn_sched_barrier(0)
#define RLX_SYS __ATOMIC_RELAXED, __HIP_MEMORY_SCOPE_SYSTEM
#define RLX_AGT __ATOMIC_RELAXED, __HIP_MEMORY_SCOPE_AGENT

constexpr int dma_slot(int k) { return 22 + 4 * k; }
constexpr int POLL_SLOT = 17;
constexpr int ND = 16;
struct Slots { int dma[64], mx[128], st[128], park[64], tail; bool ok; };
constexpr Slots make_slots() {
  Slots t{};
  for (int i = 0; i < 64; ++i) t.dma[i] = t.park[i] = -1;
  for (int i = 0; i < 128; ++i) t.mx[i] = t.st[i] = -1;
  for (int k = 0; k < 16; ++k) t.dma[dma_slot(k) % 64] = k;
  t.ok = dma_slot(0) >= 22 && dma_slot(12) >= 48 && dma_slot(15) < 64 + 20 && t.dma[POLL_SLOT] < 0;
  int fr[64] = {}, nf = 0;
  for (int w = 23; w < 78; ++w) if (t.dma[w % 64] < 0) fr[nf++] = w;
  for (int u = 0; u < 32; ++u) {
    const int x = fr[u * nf / 32], y = fr[(u + 1) * nf / 32 - 1];
    t.mx[x] = u; t.st[y] = u; t.tail += y < 64;
    if (x < 21 + 2 * (u >> 2) || x >= 64 + 2 * (u >> 2)) t.ok = false;
  }
  int c = 0;
  for (int w = 15; w < 64 && c < 32; ++w) if (t.dma[w] < 0) {
    t.park[w] = ((c >> 1) & 7) * 4 + (c >> 4) * 2 + (c & 1);
    if (w >= 32 + (c >> 4) * 16 + 2 * ((c >> 1) & 7)) t.ok = false;
    ++c;
  }
  if (c != 32) t.ok = false;
  return t;
}
constexpr Slots SL = make_slots();
static_assert(SL.ok, "drain schedule violates a deadline");
static_assert(NDW % 8 == 0 && NGWAVES % 8 == 0 && NCOMM % 8 == 0 && 3 * NGWAVES >= NUNIT, "unit dealing");
constexpr int dma_at(int s) { return SL.dma[s]; }
constexpr int g0_mx(int mode, int s) { return mode == 1 || mode == 2 ? SL.mx[(mode - 1) * 64 + s] : -1; }
constexpr int g0_st(int mode, int s) { return mode == 1 || mode == 2 ? SL.st[(mode - 1) * 64 + s] : -1; }
constexpr int park_frag_at(int mode, int s) { return mode == 2 ? SL.park[s] : -1; }
constexpr int g1_op(int mode, int s, int op) {
  const int k = op == 3 ? (s == 40 ? 0 : s == 44 ? 1 : -1) : (s == 33 + 2 * op ? 0 : s == 39 + 2 * op ? 1 : -1);
  return mode >= 3 && k >= 0 ? (mode - 3) * 2 + k : -1;
}
constexpr int nvm(int mode, int prev, int lo, int hi) {
  int c = 0;
  for (int s = lo + 1; s < hi; ++s) {
    const int q = (s + 128) % 64, md = s >= 0 ? mode : s >= -64 ? prev : 0;
    c += (dma_at(q) >= 0) + (q == POLL_SLOT) + (g0_st(md, q) >= 0) + (g1_op(md, q, 3) >= 0);
  }
  return c;
}
template <int V> struct IC { static constexpr int value = V; };
template <int I, int E, class F> __device__ __forceinline__ void static_for(F&& f) { if constexpr (I < E) { f(IC<I>{}); static_for<I + 1, E>(f); } }

enum : int { V_XWR = 15, V_ARD = 16, V_BRD, V_XRD, V_VL, V_CO, V_FVO, V_PV, V_TMP, V_T = 24, V_A = 32, V_B = 96, V_ACC = 128, A_ACC = 0, A_PARK = 128 };
#define C10(p, t) #p #t "0", #p #t "1", #p #t "2", #p #t "3", #p #t "4", #p #t "5", #p #t "6", #p #t "7", #p #t "8", #p #t "9"
#define C20_255(p) C10(p, 2), C10(p, 3), C10(p, 4), C10(p, 5), C10(p, 6), C10(p, 7), C10(p, 8), C10(p, 9), C10(p, 10), C10(p, 11), C10(p, 12), C10(p, 13), \
    C10(p, 14), C10(p, 15), C10(p, 16), C10(p, 17), C10(p, 18), C10(p, 19), C10(p, 20), C10(p, 21), C10(p, 22), C10(p, 23), C10(p, 24), \
    #p "250", #p "251", #p "252", #p "253", #p "254", #p "255"
#define OWNED_REGS "v14", "v15", "v16", "v17", "v18", "v19", C20_255(v), C10(a, ), C10(a, 1), C20_255(a)
template <int m, int n, int slot, bool zero> __device__ __forceinline__ void mfma() {
  constexpr int c = (n < 4 ? V_ACC : A_ACC) + m * 16 + (n & 3) * 4, a = V_A + m * 8, b = V_B + slot * 8;
  if constexpr (n < 4 && !zero) asm volatile("v_mfma_f32_16x16x128_f8f6f4 v[%c0:%c1], v[%c2:%c3], v[%c4:%c5], v[%c0:%c1] cbsz:1 blgp:1" :: "n"(c), "n"(c + 3), "n"(a), "n"(a + 7), "n"(b), "n"(b + 7));
  if constexpr (n < 4 && zero) asm volatile("v_mfma_f32_16x16x128_f8f6f4 v[%c0:%c1], v[%c2:%c3], v[%c4:%c5], 0 cbsz:1 blgp:1" :: "n"(c), "n"(c + 3), "n"(a), "n"(a + 7), "n"(b), "n"(b + 7));
  if constexpr (n >= 4 && !zero) asm volatile("v_mfma_f32_16x16x128_f8f6f4 a[%c0:%c1], v[%c2:%c3], v[%c4:%c5], a[%c0:%c1] cbsz:1 blgp:1" :: "n"(c), "n"(c + 3), "n"(a), "n"(a + 7), "n"(b), "n"(b + 7));
  if constexpr (n >= 4 && zero) asm volatile("v_mfma_f32_16x16x128_f8f6f4 a[%c0:%c1], v[%c2:%c3], v[%c4:%c5], 0 cbsz:1 blgp:1" :: "n"(c), "n"(c + 3), "n"(a), "n"(a + 7), "n"(b), "n"(b + 7));
}
template <int dst, int adr, int off, int sw> __device__ __forceinline__ void ldfrag() {
  asm volatile("ds_read_b128 v[%c0:%c1], v%c4 offset:%c5\n\tds_read_b128 v[%c2:%c3], v%c4 offset:%c6"
               :: "n"(dst), "n"(dst + 3), "n"(dst + 4), "n"(dst + 7), "n"(adr), "n"(off + sw * 1024), "n"(off + 1024 - sw * 1024) : "memory");
}
template <int m, int slot> __device__ __forceinline__ void ld_a() { ldfrag<V_A + m * 8, V_ARD, slot * BLK + m * 2048, 0>(); }
template <int r, int slot, int n> __device__ __forceinline__ void ld_b() { ldfrag<V_B + r * 8, V_BRD, slot * BLK + (n >> 2) * 16384 + (n & 3) * 2048, (n >> 1) & 1>(); }
template <int off> __device__ __forceinline__ void set_m0(unsigned base) { asm volatile("s_add_u32 m0, %0, %1" :: "s"(base), "n"(off) : "scc", "memory"); }
template <int k> __device__ __forceinline__ void dma(rsrc_t rs, unsigned so) {
  asm volatile("buffer_load_dwordx4 v%c0, %1, %2 offen offset:%c3 sc0 sc1 lds" :: "n"(V_VL), "s"(rs), "s"(so), "n"(k * 1024) : "memory");
}
template <int n> __device__ __forceinline__ void wait_v() { asm volatile("s_waitcnt vmcnt(%c0)" :: "n"(n > 63 ? 63 : n) : "memory"); }
template <int n> __device__ __forceinline__ void wait_l() { asm volatile("s_waitcnt lgkmcnt(%c0)" :: "n"(n) : "memory"); }
template <int off> __device__ __forceinline__ void x_read() { asm volatile("ds_read_b32 v%c0, v%c1 offset:%c2" :: "n"(V_TMP), "n"(V_XRD), "n"(off) : "memory"); }
template <int nl> __device__ __forceinline__ unsigned x_value() { unsigned r; asm volatile("s_waitcnt lgkmcnt(%c1)\n\tv_readfirstlane_b32 %0, v%c2" : "=s"(r) : "n"(nl), "n"(V_TMP) : "memory"); return r; }
template <int off> __device__ __forceinline__ void x_write(unsigned v) {
  asm volatile("v_mov_b32 v%c0, %3\n\tds_write_b32 v%c1, v%c0 offset:%c2" :: "n"(V_TMP), "n"(V_XWR), "n"(off), "s"(v) : "memory");
}
__device__ __forceinline__ unsigned pick(unsigned cand, unsigned& done) {
  unsigned t;
  asm volatile("s_and_b32 %0, %2, 0x7f7f7f7f\n\ts_cselect_b32 %0, %0, %2\n\ts_ff1_i32_b32 %0, %0\n\ts_bitset1_b32 %1, %0\n\ts_mul_i32 %0, %0, %3" : "=&s"(t), "+s"(done) : "s"(cand), "n"(KT_BYTES) : "scc");
  return t;
}
__device__ __forceinline__ void poll(rsrc_t rf, unsigned so) { asm volatile("buffer_load_dword v%c0, v%c1, %2, %3 offen sc0 sc1" :: "n"(V_PV), "n"(V_FVO), "s"(rf), "s"(so) : "memory"); }
template <int nv> __device__ __forceinline__ unsigned poll_mask(unsigned need) {
  unsigned long long b; asm volatile("s_waitcnt vmcnt(%c1)\n\tv_cmp_le_u32 %0, %2, v%c3" : "=s"(b) : "n"(nv > 63 ? 63 : nv), "s"(need), "n"(V_PV) : "memory");
  return (unsigned)b & (unsigned)(b >> 32);
}
__device__ __forceinline__ void aim_poll(int t) {
  const unsigned lb = (unsigned)(NPA - 1 + (t >> 4)) * 128u, la = (unsigned)(t & 15) * 128u - lb;
  asm volatile("v_lshrrev_b32 v%c0, 2, v%c1\n\tv_add_u32 v%c0, %2, v%c0\n\ts_mov_b64 exec, 0xffffffff\n\tv_add_u32 v%c0, %3, v%c0\n\ts_mov_b64 exec, -1" :: "n"(V_FVO), "n"(V_VL), "s"(lb), "s"(la));
}
template <int t, int u> __device__ __forceinline__ void q_g0(unsigned scale) {
  constexpr int d = V_T + t * 4, c = V_ACC + (u >> 2) * 16 + (u & 3);
  asm volatile("v_mul_f32 v%c0, %8, v%c4\n\tv_mul_f32 v%c1, %8, v%c5\n\tv_mul_f32 v%c2, %8, v%c6\n\tv_mul_f32 v%c3, %8, v%c7\n\t"
               "v_maximum3_f32 v%c0, v%c0, 0, 0\n\tv_maximum3_f32 v%c1, v%c1, 0, 0\n\tv_maximum3_f32 v%c2, v%c2, 0, 0\n\tv_maximum3_f32 v%c3, v%c3, 0, 0"
               :: "n"(d), "n"(d + 1), "n"(d + 2), "n"(d + 3), "n"(c), "n"(c + 4), "n"(c + 8), "n"(c + 12), "s"(scale));
}
template <int t, int base, int u> __device__ __forceinline__ void q_read() {
  constexpr int d = V_T + t * 4, c = base + (u >> 2) * 16 + (u & 3);
  asm volatile("v_accvgpr_read_b32 v%c0, a%c4\n\tv_accvgpr_read_b32 v%c1, a%c5\n\tv_accvgpr_read_b32 v%c2, a%c6\n\tv_accvgpr_read_b32 v%c3, a%c7"
               :: "n"(d), "n"(d + 1), "n"(d + 2), "n"(d + 3), "n"(c), "n"(c + 4), "n"(c + 8), "n"(c + 12));
}
template <int t> __device__ __forceinline__ void q_scale(unsigned scale) {
  constexpr int d = V_T + t * 4;
  asm volatile("v_mul_f32 v%c0, %4, v%c0\n\tv_mul_f32 v%c1, %4, v%c1\n\tv_mul_f32 v%c2, %4, v%c2\n\tv_mul_f32 v%c3, %4, v%c3" :: "n"(d), "n"(d + 1), "n"(d + 2), "n"(d + 3), "s"(scale));
}
template <int t> __device__ __forceinline__ void q_clamp() {
  constexpr int d = V_T + t * 4;
  asm volatile("v_maximum3_f32 v%c0, v%c0, 0, 0\n\tv_maximum3_f32 v%c1, v%c1, 0, 0\n\tv_maximum3_f32 v%c2, v%c2, 0, 0\n\tv_maximum3_f32 v%c3, v%c3, 0, 0" :: "n"(d), "n"(d + 1), "n"(d + 2), "n"(d + 3));
}
template <int t, int u, int g, bool through> __device__ __forceinline__ void q_store(rsrc_t rc) {
  constexpr int d = V_T + t * 4, row = ((u >> 2) * 16 + (u & 3)) * N * 4;
  unsigned so;
  if constexpr (through) asm volatile("s_mov_b32 %0, %5\n\tbuffer_store_dwordx4 v[%c1:%c2], v%c3, %4, %0 offen offset:%c6 sc0 sc1" : "=&s"(so) : "n"(d), "n"(d + 3), "n"(V_CO), "s"(rc), "n"(row), "n"(g * 512) : "memory");
  else asm volatile("s_mov_b32 %0, %5\n\tbuffer_store_dwordx4 v[%c1:%c2], v%c3, %4, %0 offen offset:%c6 nt" : "=&s"(so) : "n"(d), "n"(d + 3), "n"(V_CO), "s"(rc), "n"(row), "n"(g * 512) : "memory");
}
template <int f> __device__ __forceinline__ void park_frag() {
  constexpr int d = A_PARK + f * 4, c = A_ACC + f * 4;
  asm volatile("v_accvgpr_mov_b32 a%c0, a%c4\n\tv_accvgpr_mov_b32 a%c1, a%c5\n\tv_accvgpr_mov_b32 a%c2, a%c6\n\tv_accvgpr_mov_b32 a%c3, a%c7"
               :: "n"(d), "n"(d + 1), "n"(d + 2), "n"(d + 3), "n"(c), "n"(c + 1), "n"(c + 2), "n"(c + 3));
}
template <int first, bool through> __device__ __forceinline__ void drain_now(rsrc_t rc, unsigned scale) {
  static_for<first, 32>([&](auto uc) __attribute__((always_inline)) { constexpr int u = decltype(uc)::value; q_g0<u & 1, u>(scale); q_store<u & 1, u, 0, through>(rc); });
  static_for<0, 32>([&](auto uc) __attribute__((always_inline)) { constexpr int u = decltype(uc)::value; q_read<u & 1, A_ACC, u>(); q_scale<u & 1>(scale); q_clamp<u & 1>(); q_store<u & 1, u, 1, through>(rc); });
}

struct Raw { unsigned r[16]; };
struct Unit { v4u o[4]; unsigned off, so[4], flag; };
__device__ __forceinline__ void unit_load(const u8* x, const u8* w, int piece, int u, int lane, Raw& d) {
  const int panel = piece >> 2, q = piece & 3, g = lane >> 4, c = lane & 15;
  const bool isx = panel < NPA;
  const rsrc_t rs = __builtin_amdgcn_make_buffer_rsrc((void*)(isx ? x : w), 0, isx ? M * KS : KS * N, 0x00020000);
  const unsigned vo = isx ? (unsigned)(c * KS + g * 16) : (unsigned)(g * 16 * N + 4 * c);
  const unsigned so = isx ? (unsigned)((panel * 256 + u * 32) * KS + q * 128) : (unsigned)((q * 128 + (u & 1) * 64) * N + (panel - NPA) * 256 + (u >> 1) * 64);
  const unsigned sa = isx ? 4u : (unsigned)N, sb = isx ? 64u : 4u * N, sc = isx ? 16u * KS : 8u * N;
#pragma unroll
  for (int j = 0; j < 16; ++j) d.r[j] = __builtin_amdgcn_raw_buffer_load_b32(rs, vo, so + (j & 3) * sa + ((j >> 2) & 1) * sb + (j >> 3) * sc, 0);
}
__device__ __forceinline__ void unit_pack(int piece, int u, int lane, const Raw& raw, Unit& t) {
  const int panel = piece >> 2, q = piece & 3;
  t.off = (unsigned)q * (8u * KT_BYTES) + (unsigned)panel * BLK + (unsigned)lane * 16u;
  t.flag = (unsigned)piece * 32u;
  t.so[0] = 0;
  if (panel < NPA) {
#pragma unroll
    for (int i = 0; i < 4; ++i) t.o[i] = (v4u){raw.r[i * 4], raw.r[i * 4 + 1], raw.r[i * 4 + 2], raw.r[i * 4 + 3]};
    t.off += u * 4096; t.so[1] = 1024; t.so[2] = 2048; t.so[3] = 3072;
  } else {
#pragma unroll
    for (int d = 0; d < 4; ++d) {
      const unsigned r0 = raw.r[4 * d], r1 = raw.r[4 * d + 1], r2 = raw.r[4 * d + 2], r3 = raw.r[4 * d + 3];
      const unsigned t0 = __builtin_amdgcn_perm(r1, r0, 0x05010400u), t1 = __builtin_amdgcn_perm(r1, r0, 0x07030602u);
      const unsigned t2 = __builtin_amdgcn_perm(r3, r2, 0x05010400u), t3 = __builtin_amdgcn_perm(r3, r2, 0x07030602u);
      t.o[0][d] = __builtin_amdgcn_perm(t2, t0, 0x05040100u); t.o[1][d] = __builtin_amdgcn_perm(t2, t0, 0x07060302u);
      t.o[2][d] = __builtin_amdgcn_perm(t3, t1, 0x05040100u); t.o[3][d] = __builtin_amdgcn_perm(t3, t1, 0x07060302u);
    }
    t.off += (u >> 1) * 8192 + (u & 1) * 1024; t.so[1] = 2048; t.so[2] = (u & 1) ? 4096 - 1024 : 4096 + 1024; t.so[3] = t.so[2] + 2048;
  }
}
template <int NW, int NAP> __device__ __forceinline__ bool wait_local(u8* myws, unsigned off, unsigned step, unsigned want, unsigned call) {
  gu32* const f = (gu32*)(myws + WS_FLAG);
  const unsigned t0 = (unsigned)__builtin_amdgcn_s_memrealtime();
  for (;;) {
    bool low = false;
#pragma unroll
    for (int i = 0; i < NW; ++i) low |= __hip_atomic_load(f + (off + i * step) / 4, RLX_SYS) < want;
    if ((unsigned)__builtin_amdgcn_readfirstlane(__hip_atomic_load(f + TMO_OFF / 4, RLX_SYS)) == call) return false;
    if (__builtin_amdgcn_ballot_w64(low) == 0) return true;
    if ((unsigned)__builtin_amdgcn_s_memrealtime() - t0 > DEADLINE) { __hip_atomic_store(f + TMO_OFF / 4, call, RLX_SYS); return false; }
    __builtin_amdgcn_s_sleep(NAP);
  }
}

__global__ __launch_bounds__(256, 1) void fused_kernel(const Params p) {
  __shared__ __attribute__((aligned(16))) u8 shm[LDS_BYTES];
  LAS u8* lds = (LAS u8*)shm;
  const int tid = threadIdx.x, wid = __builtin_amdgcn_readfirstlane(tid >> 6), lane = tid & 63;
  const bool comm = blockIdx.x < NCOMM;
  const int gw = blockIdx.x - NCOMM;
  const int n0 = comm ? wid * NCOMM + blockIdx.x : wid * NGEMM + gw, step = comm ? NDW : NGWAVES;
  const u8* px = p.x; const u8* pw = p.w; u8* peer[NR]; int pc[NDIRECT];
#pragma unroll
  for (int i = 0; i < NR; ++i) peer[i] = p.peer[i];
#pragma unroll
  for (int i = 0; i < NDIRECT; ++i) { const int n = n0 + i * step; pc[i] = (int)p.stream[n < NUNIT ? n >> 4 : 0]; }
  asm volatile("" : "+s"(px), "+s"(pw), "+s"(peer[0]), "+s"(peer[1]), "+s"(peer[2]), "+s"(peer[3]), "+s"(peer[4]), "+s"(peer[5]), "+s"(peer[6]), "+s"(peer[7]),
                    "+s"(pc[0]), "+s"(pc[1]), "+s"(pc[2]), "+s"(pc[3]));
#pragma unroll
  for (int i = 0; i < NDIRECT; ++i) pc[i] = (pc[i] >> ((((n0 + i * step) >> 3) & 1) * 16)) & 0xFFFF;
  u8* const myws = peer[NR - 1];
  gu32* const cnt = (gu32*)(myws + WS_CNT + blockIdx.x * 128) + wid;
  const unsigned cv = __hip_atomic_load(cnt, RLX_AGT);
  if (__builtin_expect(comm, 1)) {
    volatile LAS unsigned* const stage = (volatile LAS unsigned*)(lds + LDS_X);
    Raw raw[2];
    unit_load(px, pw, pc[0], n0 & 7, lane, raw[0]);
    unit_load(px, pw, pc[1], n0 & 7, lane, raw[1]);
    unsigned tab[3];
#pragma unroll
    for (int i = 0; i < 3; ++i) tab[i] = p.stream[i * 64 + lane < NPIECE / 2 ? i * 64 + lane : 0];
    if (tid == 0) *stage = 0;
    WAIT_L(0); FBAR;
    const unsigned call = (unsigned)__builtin_amdgcn_readfirstlane(cv) + 1u;
    if (lane == 0) __hip_atomic_store(cnt, call, RLX_AGT);
    if (wid >= DW) {
      wait_local<NPIECE / 64, 32>(myws, lane * 32 + 7 * 4, 64 * 32, call * 8u, call);
      *stage = 1;
      wait_local<1, 32>(myws, SH_OFF, 0, call * (NSH * 4u), call);
      *stage = 2;
      return;
    }
    rsrc_t rs[NR];
#pragma unroll
    for (int i = 0; i < NR; ++i)
      rs[i] = __builtin_amdgcn_make_buffer_rsrc(peer[i] + WS_STG + (size_t)(call & 1u) * STG_BYTES + (unsigned)((6 - i) & 7) * KT_BYTES, 0, STG_BYTES - 7u * KT_BYTES, 0x00020000);
    u8* fl = peer[0];
#pragma unroll
    for (int i = 1; i < NR - 1; ++i) fl = lane == i ? peer[i] : fl;
    fl += WS_FLAG + (unsigned)((6 - lane) & 7) * 4;
    unsigned* const ticket = (unsigned*)(myws + WS_TICKET);
    constexpr int NSTAT = NDIRECT * NDW + NCOMM;
    const unsigned tbase = (call - 1u) * (unsigned)(NTOT - NSTAT + NCOMM) - (unsigned)NSTAT;
    const unsigned vl = (unsigned)lane * 16u;
    unsigned lstep = 1024u, sstep = 1024u;
#define DRAW() asm volatile("s_mov_b64 exec, 1\n\tglobal_atomic_add v255, %0, %1, %2 sc0\n\ts_mov_b64 exec, -1" :: "v"(0u), "v"(1u), "s"(ticket) : "v255", "memory")
#define TICKET() ({ unsigned _t; asm volatile("v_readfirstlane_b32 %0, v255" : "=s"(_t) :: "memory"); _t - tbase; })
#define PUBLISH(h) do { asm volatile("" ::: "memory"); __builtin_amdgcn_s_waitcnt(0x0F70 | ((h) & 15) | (((h) >> 4) << 14)); WAIT_V(h); \
      if (lane < NR - 1 && pflag != ~0u) __hip_atomic_fetch_add((gu32*)(fl + pflag), 1u, RLX_SYS); } while (0)
#define AWAIT(v) do { while (*stage < (v)) __builtin_amdgcn_s_sleep(1); } while (0)
    auto locate = [&](unsigned n, unsigned& off, unsigned& flag) {
      const unsigned pi = n >> 3, w = pi >> 1;
      const unsigned a = __builtin_amdgcn_readlane(tab[0], w & 63), b = __builtin_amdgcn_readlane(tab[1], w & 63), c = __builtin_amdgcn_readlane(tab[2], w & 63);
      const unsigned e = ((w < 64 ? a : w < 128 ? b : c) >> ((pi & 1) * 16)) & 0xFFFFu;
      off = n >= (unsigned)NUNIT ? ((n - NUNIT) / NSH) * (N * 16u) + ((n - NUNIT) % NSH) * 1024u : (e & 3) * (8u * KT_BYTES) + (e >> 2) * BLK + (n & 7) * 4096u;
      flag = n >= (unsigned)NUNIT ? OUT_OFF : e * 32u;
    };
    unsigned n = (unsigned)(n0 + NDIRECT * NDW), off, flag, pflag = ~0u;
    v4u d[4];
#pragma unroll
    for (int i = 0; i < NDIRECT; ++i) {
      Unit t;
      unit_pack(pc[i], n0 & 7, lane, raw[i & 1], t);
      asm volatile("" : "+v"(t.o[0]), "+v"(t.o[1]), "+v"(t.o[2]), "+v"(t.o[3]));
      if (i > 0 && i + 1 < NDIRECT) unit_load(px, pw, pc[i + 1], n0 & 7, lane, raw[(i + 1) & 1]);
#pragma unroll
      for (int k = 0; k < 28; ++k) { const int q = k % 7, j = (k / 7 + q) & 3; __builtin_amdgcn_raw_buffer_store_b128(t.o[j], rs[q], t.off, t.so[j], 17); }
      if (i == NDIRECT - 1 && wid == 0) {
        AWAIT(1);
        locate(n, off, flag);
#pragma unroll
        for (int k = 0; k < 4; ++k) d[k] = __builtin_amdgcn_raw_buffer_load_b128(rs[7], vl, off + k * 1024, 17);
        DRAW();
      }
      PUBLISH(28); pflag = t.flag;
    }
    PUBLISH(0); pflag = ~0u;
    if (wid != 0) return;
    for (;;) {
      const v4u o[4] = {d[0], d[1], d[2], d[3]};
      const unsigned coff = off, cflag = flag;
      const unsigned nn = TICKET();
      const bool more = nn < (unsigned)NTOT, sw = n < (unsigned)NUNIT && nn >= (unsigned)NUNIT;
      if (more) {
        if (sw) { AWAIT(2); rs[7] = __builtin_amdgcn_make_buffer_rsrc(p.share[7], 0, SH_BYTES, 0x00020000); lstep = N * 4u; }
        locate(nn, off, flag);
#pragma unroll
        for (int k = 0; k < 4; ++k) d[k] = __builtin_amdgcn_raw_buffer_load_b128(rs[7], vl, off + k * lstep, 17);
        DRAW();
      }
#pragma unroll
      for (int k = 0; k < 28; ++k) { const int q = k % 7, j = (k / 7 + q) & 3; __builtin_amdgcn_raw_buffer_store_b128(o[j], rs[q], vl, coff + j * sstep, 17); }
      PUBLISH(28);
      pflag = cflag;
      if (!more) break;
      if (sw) {
        sstep = N * 4u;
#pragma unroll
        for (int i = 0; i < NR - 1; ++i) rs[i] = __builtin_amdgcn_make_buffer_rsrc(p.share[i], 0, SH_BYTES, 0x00020000);
      }
      n = nn;
    }
    PUBLISH(0);
    return;
  }
  unsigned call;
  {
    Raw raw[3];
#pragma unroll
    for (int i = 0; i < 3; ++i) unit_load(px, pw, pc[i], n0 & 7, lane, raw[i]);
    call = (unsigned)__builtin_amdgcn_readfirstlane(cv) + 1u;
    if (lane == 0) __hip_atomic_store(cnt, call, RLX_AGT);
    const rsrc_t rl = __builtin_amdgcn_make_buffer_rsrc(myws + WS_STG + (size_t)(call & 1u) * STG_BYTES + 7u * KT_BYTES, 0, STG_BYTES - 7u * KT_BYTES, 0x00020000);
    unsigned fo = 0;
#pragma unroll
    for (int i = 0; i < 3; ++i) {
      if (n0 + i * NGWAVES < NUNIT) {
        Unit t; unit_pack(pc[i], n0 & 7, lane, raw[i], t);
#pragma unroll
        for (int k = 0; k < 4; ++k) __builtin_amdgcn_raw_buffer_store_b128(t.o[k], rl, t.off, t.so[k], 17);
        fo = lane == i ? t.flag : fo;
      }
    }
    WAIT_V(0);
    if (lane < 3 && n0 + lane * NGWAVES < NUNIT) __hip_atomic_fetch_add((gu32*)(myws + WS_FLAG + fo + 7 * 4), 1u, RLX_SYS);
    if (wid == 0) __builtin_amdgcn_fence(__ATOMIC_ACQUIRE, "");
    if (tid == 0) *(LAS float*)(lds + LDS_X + 32) = p.sx[0] * p.sw[0];
    asm volatile("s_waitcnt vmcnt(0) lgkmcnt(0)" ::: "memory");
  }
  const int wr = wid >> 1, wc = wid & 1;
  const rsrc_t rs = __builtin_amdgcn_make_buffer_rsrc(myws + WS_STG + (size_t)(call & 1u) * STG_BYTES, 0, STG_BYTES, 0x00020000);
  const rsrc_t rf = __builtin_amdgcn_make_buffer_rsrc(myws + WS_FLAG, 0, FLAG_BYTES, 0x00020000);
  const unsigned need = call * 8u;
  const unsigned lbase = (unsigned)(size_t)lds;
  const unsigned rinit = wid ? 0xFFFFFFFFu : 0u;
  unsigned long long tlo = p.tile[gw][0] | ((unsigned long long)p.tile[gw][1] << 32); unsigned thi = p.tile[gw][2];
  asm volatile("v_mov_b32 v%c0, %5\n\tv_mov_b32 v%c1, %6\n\tv_mov_b32 v%c2, %7\n\tv_mov_b32 v%c3, %8\n\tv_mov_b32 v%c4, %9\n\tv_mov_b32 v%c10, %11"
               :: "n"(V_ARD), "n"(V_BRD), "n"(V_XRD), "n"(V_VL), "n"(V_CO),
                  "v"(lbase + LDS_A + wr * 16384 + lane * 16), "v"(lbase + LDS_B + wc * 8192 + lane * 16), "v"(lbase + LDS_X), "v"(lane * 16),
                  "v"((4 * (lane >> 4) * N + 4 * (lane & 15)) * 4), "n"(V_XWR), "v"(lbase + LDS_X + wid * 4) : OWNED_REGS, "memory");
  const unsigned mA = lbase + wid * 8192, mB = lbase + wid * 4096;
#define M0_A(slot, h) set_m0<LDS_A + (slot) * BLK + (h) * 4096>(mA)
#define M0_B(slot, h) set_m0<LDS_B + (slot) * BLK + (h) * 16384>(mB)
#define SRC_A(t) ((unsigned)(((t) & 15) * BLK + wid * 8192))
#define SRC_B(t) ((unsigned)((NPA + ((t) >> 4)) * BLK + wid * 4096))
#define ENSURE(cand) do { if (__builtin_expect((cand) == 0, 0)) { \
      const unsigned _t0 = (unsigned)__builtin_amdgcn_s_memrealtime(); \
      for (;;) { unsigned _w; \
        poll(rf, 0u); asm volatile("buffer_load_dword v%c0, off, %1, %2 sc0 sc1" :: "n"(V_TMP), "s"(rf), "s"(TMO_OFF) : "memory"); \
        rdy = poll_mask<0>(need); cand = rdy & ~done; \
        if (cand) break; \
        asm volatile("v_readfirstlane_b32 %0, v%c1" : "=s"(_w) : "n"(V_TMP)); \
        if (_w == call || (unsigned)__builtin_amdgcn_s_memrealtime() - _t0 > DEADLINE) { \
          asm volatile("v_mov_b32 v%c0, %1\n\tbuffer_store_dword v%c0, off, %2, %3 sc0 sc1\n\ts_nop 1" :: "n"(V_TMP), "s"(call), "s"(rf), "s"(TMO_OFF) : "memory"); \
          x_write<32>(0x7FC00000u); cand = ~done; break; } \
        __builtin_amdgcn_s_sleep(2); } \
      __builtin_amdgcn_s_waitcnt(0xC07F); } } while (0)
#define POLL_OFF (rdy == 0xFFFFFFFFu ? 0x40000u : 0u)

  int cur = (int)(tlo & 0xFFFFu);
  unsigned done = 0, rdy = rinit;
  unsigned sA = SRC_A(cur), sB = SRC_B(cur);
  unsigned bsrc;
  aim_poll(cur);
  {
    unsigned c_ = rdy; ENSURE(c_); x_write<0>(pick(c_, done));
    c_ = rdy & ~done; ENSURE(c_); x_write<16>(pick(c_, done));
    WAIT_L(0); FBAR;
    x_read<0>(); const unsigned k0 = x_value<0>();
    x_read<16>(); const unsigned k1 = x_value<0>();
    poll(rf, POLL_OFF);
#define DMA4(m0v, so) do { m0v; asm volatile("s_nop 0"); dma<0>(rs, so); dma<1>(rs, so); dma<2>(rs, so); dma<3>(rs, so); } while (0)
    DMA4(M0_A(0, 0), sA + k0); DMA4(M0_A(0, 1), sA + k0 + 4096); DMA4(M0_B(0, 0), sB + k0); DMA4(M0_B(0, 1), sB + k0 + 16384);
    DMA4(M0_A(1, 0), sA + k1); DMA4(M0_A(1, 1), sA + k1 + 4096); DMA4(M0_B(1, 0), sB + k1);
    bsrc = sB + k1;
    WAIT_V(0); FBAR;
    ld_b<0, 0, 0>(); ld_a<0, 0>(); ld_a<1, 0>(); ld_b<1, 0, 1>(); ld_a<2, 0>(); ld_a<3, 0>(); ld_a<4, 0>(); ld_a<5, 0>(); ld_a<6, 0>(); ld_a<7, 0>();
  }
#define KTILE(par, MODE, PREV, SW) do { \
    unsigned asrc, cand, jp; \
    asm volatile("" ::: OWNED_REGS);        \
    static_for<0, 64>([&](auto gsc) __attribute__((always_inline)) { \
      constexpr int gs = decltype(gsc)::value, np = gs >> 4, s = gs & 15, m = s >> 1, e = s & 1; \
      constexpr int cb = (np & 1) * 2, nb = 2 - cb, z = e ^ (m & 1); \
      if constexpr (gs == 16) { wait_v<nvm(MODE, PREV, dma_slot(15) - 128, 16)>(); wait_l<0>(); BAR; } \
      if constexpr (gs == 48) { wait_v<nvm(MODE, PREV, dma_slot(11) - 64, 48)>(); wait_l<0>(); BAR; } \
      if constexpr (gs <= 1) wait_l<14>(); \
      if constexpr (gs == 4 || gs == 6) wait_l<12>(); \
      if constexpr (gs == 8) wait_l<10>(); \
      if constexpr (gs == 10 || gs == 12 || gs == 14) wait_l<19 - gs>(); \
      if constexpr (gs == 32 || gs == 33) wait_l<2>(); \
      mfma<m, np * 2 + z, cb + z, (MODE) == 2>(); SCHED; \
      if constexpr (gs == 16) x_read<(par) * 16>(); \
      if constexpr (gs == 20) { const unsigned ko = x_value<2>(); bsrc = sB + ko; asrc = sA + ko; } \
      if constexpr ((MODE) == 1 && gs == 20) x_read<32>(); \
      if constexpr ((MODE) == 1 && gs == 22) scale = x_value<2>(); \
      if constexpr (s == 0 || s == 4) { if constexpr (np < 3) ld_b<nb + (s >> 2), par, np * 2 + 2 + (s >> 2)>(); else ld_b<nb + (s >> 2), (par) ^ 1, (s >> 2)>(); } \
      if constexpr (np == 3 && e == 1) ld_a<m, (par) ^ 1>(); \
      if constexpr (gs == 1) rdy |= poll_mask<nvm(MODE, PREV, POLL_SLOT - 64, 1)>(need); \
      if constexpr (gs == 3) { cand = rdy & ~done; ENSURE(cand); } \
      if constexpr (gs == 7) jp = pick(cand, done); \
      if constexpr (gs == 9) x_write<(par) * 16>(jp); \
      if constexpr (gs == 11 && (SW)) { done = 0; rdy = rinit; aim_poll(nxt); } \
      if constexpr (gs == POLL_SLOT) poll(rf, POLL_OFF); \
      if constexpr (dma_at((gs + 1) % 64) >= 0 && dma_at((gs + 1) % 64) % 4 == 0) { constexpr int k = dma_at((gs + 1) % 64), sl = dma_slot(k) > 64 ? (par) ^ 1 : (par); \
        if constexpr (k < 8) M0_A(sl, k >> 2); else M0_B(sl, (k >> 2) - 2); } \
      if constexpr (dma_at(gs) >= 0) { constexpr int k = dma_at(gs); dma<k & 3>(rs, k < 8 ? asrc + (k >> 2) * 4096 : bsrc + ((k >> 2) - 2) * 16384); } \
      if constexpr (gs == 29 && (SW)) { sA = SRC_A(nxt); sB = SRC_B(nxt); } \
      if constexpr (g0_mx(MODE, gs) >= 0) q_g0<g0_mx(MODE, gs) & 1, g0_mx(MODE, gs)>(scale); \
      if constexpr (g0_st(MODE, gs) >= 0) q_store<g0_st(MODE, gs) & 1, g0_st(MODE, gs), 0, false>(rc); \
      if constexpr (park_frag_at(MODE, gs) >= 0) park_frag<park_frag_at(MODE, gs)>(); \
      if constexpr (g1_op(MODE, gs, 0) >= 0) q_read<g1_op(MODE, gs, 0) & 1, A_PARK, g1_op(MODE, gs, 0)>(); \
      if constexpr (g1_op(MODE, gs, 1) >= 0) q_scale<g1_op(MODE, gs, 1) & 1>(scale); \
      if constexpr (g1_op(MODE, gs, 2) >= 0) q_clamp<g1_op(MODE, gs, 2) & 1>(); \
      if constexpr (g1_op(MODE, gs, 3) >= 0) q_store<g1_op(MODE, gs, 3) & 1, g1_op(MODE, gs, 3), 1, false>(rc); \
      SCHED; \
    }); } while (0)
  unsigned scale = 0;
  rsrc_t rc = __builtin_amdgcn_make_buffer_rsrc(p.out, 0, 0, 0x00020000);
  for (int r = 0;; ++r) {
    tlo = (tlo >> 16) | ((unsigned long long)thi << 48); thi = (thi >> 16) | 0xFFFF0000u;
    const int nx = (int)(tlo & 0xFFFFu);
    const bool has_next = nx != 0xFFFF;
    const int nxt = has_next ? nx : cur;
    KTILE(0, 2, 1, false);
    static_for<1, ND + 2>([&](auto tc) __attribute__((always_inline)) { constexpr int t = decltype(tc)::value; KTILE(t & 1, (t <= ND ? 2 + t : 0), 1 + t, false); });
    for (int t = ND + 2; t < NKT - 4; t += 2) {
      KTILE(0, 0, 0, false);
      KTILE(1, 0, 0, false);
    }
    KTILE(0, 0, 0, false);
    KTILE(1, 0, 0, true);
    KTILE(0, 0, 0, false);
    const bool shared = (cur & 15) >= 8 && (unsigned)((cur >> 4) - SC) < (unsigned)NSH;
    float* const cp = p.out + (size_t)((cur & 15) * 256 + wr * 128) * N + (cur >> 4) * 256 + wc * 64;
    constexpr unsigned WAVE_SPAN = 127u * N * 4u + 768u;
    rc = __builtin_amdgcn_make_buffer_rsrc(cp, 0, shared ? 0 : WAVE_SPAN, 0x00020000);
    KTILE(1, 1, 0, false);
    if (shared) {
      drain_now<0, true>(__builtin_amdgcn_make_buffer_rsrc(cp, 0, WAVE_SPAN, 0x00020000), scale);
      asm volatile("v_mov_b32 v%c0, 1\n\ts_waitcnt vmcnt(0)\n\ts_mov_b64 exec, 1\n\tbuffer_atomic_add v%c0, off, %1, %2 sc1\n\ts_mov_b64 exec, -1" :: "n"(V_TMP), "s"(rf), "s"(SH_OFF) : "memory");
    }
    if (!has_next) break;
    cur = nxt;
  }
  drain_now<SL.tail, false>(rc, scale);
  int ln;
  asm volatile("s_waitcnt vmcnt(0) ; end of owned registers\n\tv_mbcnt_lo_u32_b32 %0, -1, 0\n\tv_mbcnt_hi_u32_b32 %0, -1, %0" : "=v"(ln) :: OWNED_REGS, "memory");
  if (gw == WAITER && wid == 0) {
    const bool ok = wait_local<1, 2>(myws, OUT_OFF + (ln < NR - 1 ? ln : 0) * 4, 0, call * (unsigned)NOUT, call);
    __builtin_amdgcn_fence(__ATOMIC_ACQUIRE, "");
    if (!ok && ln == 0) p.out[(size_t)(cur & 15) * 256 * N + (cur >> 4) * 256] = __builtin_nanf("");
  }
}

static void build_schedule(Params& p, int rank) {
  constexpr int W = NGEMM / 8, PR = 2, PC = 3, C0 = 18;
  for (int x = 0; x < 8; ++x) {
    const int xr = x & 3, xc = x >> 2;
    u16 list[128]; int n = 0;
    for (int pass = 0; pass < 2; ++pass)
      for (int ci = 0; ci < 32; ++ci) for (int ri = 0; ri < 4; ++ri) {
        const int pm = xr + 4 * ri, pn = 2 * ci + xc;
        if ((ri < PR && ci < PC) != (pass == 0)) continue;
        if (pm >= 8 && pn >= SC && pn < SC + NSH) {
          if (pn == SC && ri == 2 && xr < NSH) list[n++] = (u16)((8 + rank) | (SC + xr) << 4);
          continue;
        }
        list[n++] = (u16)(pm | pn << 4);
      }
    const int extra = n - 4 * W;
    for (int c = 0; c < W; ++c) {
      int k = 0; u16* t = (u16*)p.tile[c * 8 + x];
      t[k++] = list[c];
      if (c < extra) t[k++] = list[W + c];
      for (int r = 0; r < 3; ++r) t[k++] = list[W + extra + r * W + (c + W - extra) % W];
      while (k <= MAXT) t[k++] = 0xFFFF;
    }
  }
  int s = 0;
  auto put = [&](int panel, int q) { ((u16*)p.stream)[s++] = (u16)(panel << 2 | q); };
  for (int q = 0; q < 4; ++q) for (int i = 0; i < 16; ++i) { if (i < 4 * PR) put(i, q); if (i < 2 * PC) put(NPA + i, q); }
  for (int q = 0; q < 4; ++q) { for (int i = 4 * PR; i < 16; ++i) put(i, q); for (int i = 2 * PC; i < C0; ++i) put(NPA + i, q); }
  const int lim[4] = {C0, 34, 50, 64};
  for (int r = 0; r < 3; ++r) for (int q = 0; q < 4; ++q) for (int i = lim[r]; i < lim[r + 1]; ++i) put(NPA + i, q);
}

extern "C" void kernel_launch(int rank, int nranks, ncclComm_t comm, hipStream_t stream,
                              void* const* d_in, const int* in_sizes, int n_in,
                              void* d_out, int out_size, const int* in_dtypes, int out_dtype,
                              void* const* d_out_all, void* const* d_ws_all, size_t ws_bytes) {
  if (nranks != NR || n_in != 4 || ws_bytes < WS_TOTAL || (unsigned)rank >= (unsigned)NR ||
      in_sizes[0] != M * KS || in_sizes[1] != KS * N || out_size != M * N) { fprintf(stderr, "unexpected problem shape\n"); return; }
  Params p;
  p.x = (const u8*)d_in[0]; p.w = (const u8*)d_in[1]; p.sx = (const float*)d_in[2]; p.sw = (const float*)d_in[3];
  p.out = (float*)d_out;
  for (int i = 0; i < NR; ++i) {
    p.peer[i] = (u8*)d_ws_all[(rank + 1 + i) % NR];
    p.share[i] = (float*)d_out_all[(rank + 1 + i) % NR] + (size_t)(8 + rank) * 256 * N + SC * 256;
  }
  build_schedule(p, rank);
  hipLaunchKernelGGL(fused_kernel, dim3(GRID), dim3(256), 0, stream, p);
  HIP_CHECK(hipGetLastError());
}
```

```cpp
#include <hip/hip_runtime.h>
#include <rccl/rccl.h>
#include <cstdio>

#define HIP_CHECK(expr) do { hipError_t _e = (expr); if (_e != hipSuccess) { \
    fprintf(stderr, "HIP error %s at %s:%d\n", hipGetErrorString(_e), __FILE__, __LINE__); \
    return; } } while (0)

#define LAS __attribute__((address_space(3)))
typedef unsigned char u8;
typedef unsigned short u16;
typedef __attribute__((address_space(1))) unsigned gu32;
typedef unsigned v4u __attribute__((ext_vector_type(4)));
typedef __amdgpu_buffer_rsrc_t rsrc_t;

constexpr int M = 4096, N = 16384, K = 4096, NR = 8, KS = K / NR;
constexpr int NKT = 32;
constexpr int BLK = 32768;
constexpr int NPA = 16, NPB = 64, NPANEL = NPA + NPB;
constexpr unsigned KT_BYTES = NPANEL * BLK;
constexpr unsigned STG_BYTES = NKT * KT_BYTES;
constexpr int GRID = 256, NCOMM = 16, NGEMM = GRID - NCOMM, NGWAVES = NGEMM * 4;
constexpr int DW = 3, NDIRECT = 4, NDW = NCOMM * DW;
constexpr int WAITER = 64;
constexpr int MAXT = 5;
constexpr int NPIECE = NPANEL * 4, NUNIT = NPIECE * 8;
constexpr int SC = 20, NSH = 3, NOUT = NSH * 64, NTOT = NUNIT + NOUT;
constexpr unsigned SH_BYTES = 255u * N * 4u + NSH * 1024u;
constexpr size_t WS_CNT = 0;
constexpr size_t WS_TICKET = GRID * 128;
constexpr size_t WS_FLAG = 65536;
constexpr unsigned TMO_OFF = NPANEL * 128;
constexpr unsigned OUT_OFF = TMO_OFF + 128;
constexpr unsigned SH_OFF = OUT_OFF + 128;
constexpr unsigned ROT_OFF = SH_OFF + 128;
constexpr unsigned FLAG_BYTES = ROT_OFF + 128;
constexpr size_t WS_STG = 1u << 20;
constexpr size_t WS_TOTAL = WS_STG + 2 * (size_t)STG_BYTES;
constexpr unsigned DEADLINE = 50000000u;
constexpr int LDS_A = 0, LDS_B = 2 * BLK, LDS_X = 4 * BLK, LDS_BYTES = LDS_X + 64;

struct Params {
  const u8* x; const u8* w; const float* sx; const float* sw; float* out;
  u8* peer[NR];
  float* share[NR];
  unsigned tile[NGEMM][(MAXT + 1) / 2];
  unsigned stream[NPIECE / 2];
};

#define STR2(x) #x
#define STR(x) STR2(x)
#define WAIT_V(n) asm volatile("s_waitcnt vmcnt(" STR(n) ")" ::: "memory")
#define WAIT_L(n) asm volatile("s_waitcnt lgkmcnt(" STR(n) ")" ::: "memory")
#define BAR __builtin_amdgcn_s_barrier()
#define FBAR do { asm volatile("" ::: "memory"); __builtin_amdgcn_s_barrier(); asm volatile("" ::: "memory"); } while (0)
#define SCHED __builtin_amdgcn_sched_barrier(0)
#define RLX_SYS __ATOMIC_RELAXED, __HIP_MEMORY_SCOPE_SYSTEM
#define RLX_AGT __ATOMIC_RELAXED, __HIP_MEMORY_SCOPE_AGENT

constexpr int dma_slot(int k) { return 22 + 4 * k; }
constexpr int POLL_SLOT = 17;
constexpr int ND = 16;
struct Slots { int dma[64], mx[128], st[128], park[64], tail; bool ok; };
constexpr Slots make_slots() {
  Slots t{};
  for (int i = 0; i < 64; ++i) t.dma[i] = t.park[i] = -1;
  for (int i = 0; i < 128; ++i) t.mx[i] = t.st[i] = -1;
  for (int k = 0; k < 16; ++k) t.dma[dma_slot(k) % 64] = k;
  t.ok = dma_slot(0) >= 22 && dma_slot(12) >= 48 && dma_slot(15) < 64 + 20 && t.dma[POLL_SLOT] < 0;
  int fr[64] = {}, nf = 0;
  for (int w = 23; w < 78; ++w) if (t.dma[w % 64] < 0) fr[nf++] = w;
  for (int u = 0; u < 32; ++u) {
    const int x = fr[u * nf / 32], y = fr[(u + 1) * nf / 32 - 1];
    t.mx[x] = u; t.st[y] = u; t.tail += y < 64;
    if (x < 21 + 2 * (u >> 2) || x >= 64 + 2 * (u >> 2)) t.ok = false;
  }
  int c = 0;
  for (int w = 15; w < 64 && c < 32; ++w) if (t.dma[w] < 0) {
    t.park[w] = ((c >> 1) & 7) * 4 + (c >> 4) * 2 + (c & 1);
    if (w >= 32 + (c >> 4) * 16 + 2 * ((c >> 1) & 7)) t.ok = false;
    ++c;
  }
  if (c != 32) t.ok = false;
  return t;
}
constexpr Slots SL = make_slots();
static_assert(SL.ok, "drain schedule violates a deadline");
static_assert(NDW % 8 == 0 && NGWAVES % 8 == 0 && NCOMM % 8 == 0 && 3 * NGWAVES >= NUNIT, "unit dealing");
constexpr int dma_at(int s) { return SL.dma[s]; }
constexpr int g0_mx(int mode, int s) { return mode == 1 || mode == 2 ? SL.mx[(mode - 1) * 64 + s] : -1; }
constexpr int g0_st(int mode, int s) { return mode == 1 || mode == 2 ? SL.st[(mode - 1) * 64 + s] : -1; }
constexpr int park_frag_at(int mode, int s) { return mode == 2 ? SL.park[s] : -1; }
constexpr int g1_op(int mode, int s, int op) {
  const int k = op == 3 ? (s == 40 ? 0 : s == 44 ? 1 : -1) : (s == 33 + 2 * op ? 0 : s == 39 + 2 * op ? 1 : -1);
  return mode >= 3 && k >= 0 ? (mode - 3) * 2 + k : -1;
}
constexpr int nvm(int mode, int prev, int lo, int hi) {
  int c = 0;
  for (int s = lo + 1; s < hi; ++s) {
    const int q = (s + 128) % 64, md = s >= 0 ? mode : s >= -64 ? prev : 0;
    c += (dma_at(q) >= 0) + (q == POLL_SLOT) + (g0_st(md, q) >= 0) + (g1_op(md, q, 3) >= 0);
  }
  return c;
}
template <int V> struct IC { static constexpr int value = V; };
template <int I, int E, class F> __device__ __forceinline__ void static_for(F&& f) { if constexpr (I < E) { f(IC<I>{}); static_for<I + 1, E>(f); } }

enum : int { V_XWR = 15, V_ARD = 16, V_BRD, V_XRD, V_VL, V_CO, V_FVO, V_PV, V_TMP, V_T = 24, V_A = 32, V_B = 96, V_ACC = 128, A_ACC = 0, A_PARK = 128 };
#define C10(p, t) #p #t "0", #p #t "1", #p #t "2", #p #t "3", #p #t "4", #p #t "5", #p #t "6", #p #t "7", #p #t "8", #p #t "9"
#define C20_255(p) C10(p, 2), C10(p, 3), C10(p, 4), C10(p, 5), C10(p, 6), C10(p, 7), C10(p, 8), C10(p, 9), C10(p, 10), C10(p, 11), C10(p, 12), C10(p, 13), \
    C10(p, 14), C10(p, 15), C10(p, 16), C10(p, 17), C10(p, 18), C10(p, 19), C10(p, 20), C10(p, 21), C10(p, 22), C10(p, 23), C10(p, 24), \
    #p "250", #p "251", #p "252", #p "253", #p "254", #p "255"
#define OWNED_REGS "v14", "v15", "v16", "v17", "v18", "v19", C20_255(v), C10(a, ), C10(a, 1), C20_255(a)
template <int m, int n, int slot, bool zero> __device__ __forceinline__ void mfma() {
  constexpr int c = (n < 4 ? V_ACC : A_ACC) + m * 16 + (n & 3) * 4, a = V_A + m * 8, b = V_B + slot * 8;
  if constexpr (n < 4 && !zero) asm volatile("v_mfma_f32_16x16x128_f8f6f4 v[%c0:%c1], v[%c2:%c3], v[%c4:%c5], v[%c0:%c1] cbsz:1 blgp:1" :: "n"(c), "n"(c + 3), "n"(a), "n"(a + 7), "n"(b), "n"(b + 7));
  if constexpr (n < 4 && zero) asm volatile("v_mfma_f32_16x16x128_f8f6f4 v[%c0:%c1], v[%c2:%c3], v[%c4:%c5], 0 cbsz:1 blgp:1" :: "n"(c), "n"(c + 3), "n"(a), "n"(a + 7), "n"(b), "n"(b + 7));
  if constexpr (n >= 4 && !zero) asm volatile("v_mfma_f32_16x16x128_f8f6f4 a[%c0:%c1], v[%c2:%c3], v[%c4:%c5], a[%c0:%c1] cbsz:1 blgp:1" :: "n"(c), "n"(c + 3), "n"(a), "n"(a + 7), "n"(b), "n"(b + 7));
  if constexpr (n >= 4 && zero) asm volatile("v_mfma_f32_16x16x128_f8f6f4 a[%c0:%c1], v[%c2:%c3], v[%c4:%c5], 0 cbsz:1 blgp:1" :: "n"(c), "n"(c + 3), "n"(a), "n"(a + 7), "n"(b), "n"(b + 7));
}
template <int dst, int adr, int off, int sw> __device__ __forceinline__ void ldfrag() {
  asm volatile("ds_read_b128 v[%c0:%c1], v%c4 offset:%c5\n\tds_read_b128 v[%c2:%c3], v%c4 offset:%c6"
               :: "n"(dst), "n"(dst + 3), "n"(dst + 4), "n"(dst + 7), "n"(adr), "n"(off + sw * 1024), "n"(off + 1024 - sw * 1024) : "memory");
}
template <int m, int slot> __device__ __forceinline__ void ld_a() { ldfrag<V_A + m * 8, V_ARD, slot * BLK + m * 2048, 0>(); }
template <int r, int slot, int n> __device__ __forceinline__ void ld_b() { ldfrag<V_B + r * 8, V_BRD, slot * BLK + (n >> 2) * 16384 + (n & 3) * 2048, (n >> 1) & 1>(); }
template <int off> __device__ __forceinline__ void set_m0(unsigned base) { asm volatile("s_add_u32 m0, %0, %1" :: "s"(base), "n"(off) : "scc", "memory"); }
template <int k> __device__ __forceinline__ void dma(rsrc_t rs, unsigned so) {
  asm volatile("buffer_load_dwordx4 v%c0, %1, %2 offen offset:%c3 sc0 sc1 lds" :: "n"(V_VL), "s"(rs), "s"(so), "n"(k * 1024) : "memory");
}
template <int n> __device__ __forceinline__ void wait_v() { asm volatile("s_waitcnt vmcnt(%c0)" :: "n"(n > 63 ? 63 : n) : "memory"); }
template <int n> __device__ __forceinline__ void wait_l() { asm volatile("s_waitcnt lgkmcnt(%c0)" :: "n"(n) : "memory"); }
template <int off> __device__ __forceinline__ void x_read() { asm volatile("ds_read_b32 v%c0, v%c1 offset:%c2" :: "n"(V_TMP), "n"(V_XRD), "n"(off) : "memory"); }
template <int nl> __device__ __forceinline__ unsigned x_value() { unsigned r; asm volatile("s_waitcnt lgkmcnt(%c1)\n\tv_readfirstlane_b32 %0, v%c2" : "=s"(r) : "n"(nl), "n"(V_TMP) : "memory"); return r; }
template <int off> __device__ __forceinline__ void x_write(unsigned v) {
  asm volatile("v_mov_b32 v%c0, %3\n\tds_write_b32 v%c1, v%c0 offset:%c2" :: "n"(V_TMP), "n"(V_XWR), "n"(off), "s"(v) : "memory");
}
__device__ __forceinline__ unsigned pick(unsigned cand, unsigned& done) {
  unsigned t;
  asm volatile("s_and_b32 %0, %2, 0x7f7f7f7f\n\ts_cselect_b32 %0, %0, %2\n\ts_ff1_i32_b32 %0, %0\n\ts_bitset1_b32 %1, %0\n\ts_mul_i32 %0, %0, %3" : "=&s"(t), "+s"(done) : "s"(cand), "n"(KT_BYTES) : "scc");
  return t;
}
__device__ __forceinline__ void poll(rsrc_t rf, unsigned so) { asm volatile("buffer_load_dword v%c0, v%c1, %2, %3 offen sc0 sc1" :: "n"(V_PV), "n"(V_FVO), "s"(rf), "s"(so) : "memory"); }
template <int nv> __device__ __forceinline__ unsigned poll_mask(unsigned need) {
  unsigned long long b; asm volatile("s_waitcnt vmcnt(%c1)\n\tv_cmp_le_u32 %0, %2, v%c3" : "=s"(b) : "n"(nv > 63 ? 63 : nv), "s"(need), "n"(V_PV) : "memory");
  return (unsigned)b & (unsigned)(b >> 32);
}
__device__ __forceinline__ void aim_poll(int t) {
  const unsigned lb = (unsigned)(NPA - 1 + (t >> 4)) * 128u, la = (unsigned)(t & 15) * 128u - lb;
  asm volatile("v_lshrrev_b32 v%c0, 2, v%c1\n\tv_add_u32 v%c0, %2, v%c0\n\ts_mov_b64 exec, 0xffffffff\n\tv_add_u32 v%c0, %3, v%c0\n\ts_mov_b64 exec, -1" :: "n"(V_FVO), "n"(V_VL), "s"(lb), "s"(la));
}
template <int t, int u> __device__ __forceinline__ void q_g0(unsigned scale) {
  constexpr int d = V_T + t * 4, c = V_ACC + (u >> 2) * 16 + (u & 3);
  asm volatile("v_mul_f32 v%c0, %8, v%c4\n\tv_mul_f32 v%c1, %8, v%c5\n\tv_mul_f32 v%c2, %8, v%c6\n\tv_mul_f32 v%c3, %8, v%c7\n\t"
               "v_maximum3_f32 v%c0, v%c0, 0, 0\n\tv_maximum3_f32 v%c1, v%c1, 0, 0\n\tv_maximum3_f32 v%c2, v%c2, 0, 0\n\tv_maximum3_f32 v%c3, v%c3, 0, 0"
               :: "n"(d), "n"(d + 1), "n"(d + 2), "n"(d + 3), "n"(c), "n"(c + 4), "n"(c + 8), "n"(c + 12), "s"(scale));
}
template <int t, int base, int u> __device__ __forceinline__ void q_read() {
  constexpr int d = V_T + t * 4, c = base + (u >> 2) * 16 + (u & 3);
  asm volatile("v_accvgpr_read_b32 v%c0, a%c4\n\tv_accvgpr_read_b32 v%c1, a%c5\n\tv_accvgpr_read_b32 v%c2, a%c6\n\tv_accvgpr_read_b32 v%c3, a%c7"
               :: "n"(d), "n"(d + 1), "n"(d + 2), "n"(d + 3), "n"(c), "n"(c + 4), "n"(c + 8), "n"(c + 12));
}
template <int t> __device__ __forceinline__ void q_scale(unsigned scale) {
  constexpr int d = V_T + t * 4;
  asm volatile("v_mul_f32 v%c0, %4, v%c0\n\tv_mul_f32 v%c1, %4, v%c1\n\tv_mul_f32 v%c2, %4, v%c2\n\tv_mul_f32 v%c3, %4, v%c3" :: "n"(d), "n"(d + 1), "n"(d + 2), "n"(d + 3), "s"(scale));
}
template <int t> __device__ __forceinline__ void q_clamp() {
  constexpr int d = V_T + t * 4;
  asm volatile("v_maximum3_f32 v%c0, v%c0, 0, 0\n\tv_maximum3_f32 v%c1, v%c1, 0, 0\n\tv_maximum3_f32 v%c2, v%c2, 0, 0\n\tv_maximum3_f32 v%c3, v%c3, 0, 0" :: "n"(d), "n"(d + 1), "n"(d + 2), "n"(d + 3));
}
template <int t, int u, int g, bool through> __device__ __forceinline__ void q_store(rsrc_t rc) {
  constexpr int d = V_T + t * 4, row = ((u >> 2) * 16 + (u & 3)) * N * 4;
  unsigned so;
  if constexpr (through) asm volatile("s_mov_b32 %0, %5\n\tbuffer_store_dwordx4 v[%c1:%c2], v%c3, %4, %0 offen offset:%c6 sc0 sc1" : "=&s"(so) : "n"(d), "n"(d + 3), "n"(V_CO), "s"(rc), "n"(row), "n"(g * 512) : "memory");
  else asm volatile("s_mov_b32 %0, %5\n\tbuffer_store_dwordx4 v[%c1:%c2], v%c3, %4, %0 offen offset:%c6 nt" : "=&s"(so) : "n"(d), "n"(d + 3), "n"(V_CO), "s"(rc), "n"(row), "n"(g * 512) : "memory");
}
template <int f> __device__ __forceinline__ void park_frag() {
  constexpr int d = A_PARK + f * 4, c = A_ACC + f * 4;
  asm volatile("v_accvgpr_mov_b32 a%c0, a%c4\n\tv_accvgpr_mov_b32 a%c1, a%c5\n\tv_accvgpr_mov_b32 a%c2, a%c6\n\tv_accvgpr_mov_b32 a%c3, a%c7"
               :: "n"(d), "n"(d + 1), "n"(d + 2), "n"(d + 3), "n"(c), "n"(c + 1), "n"(c + 2), "n"(c + 3));
}
template <int first, bool through> __device__ __forceinline__ void drain_now(rsrc_t rc, unsigned scale) {
  static_for<first, 32>([&](auto uc) __attribute__((always_inline)) { constexpr int u = decltype(uc)::value; q_g0<u & 1, u>(scale); q_store<u & 1, u, 0, through>(rc); });
  static_for<0, 32>([&](auto uc) __attribute__((always_inline)) { constexpr int u = decltype(uc)::value; q_read<u & 1, A_ACC, u>(); q_scale<u & 1>(scale); q_clamp<u & 1>(); q_store<u & 1, u, 1, through>(rc); });
}

struct Raw { unsigned r[16]; };
struct Unit { v4u o[4]; unsigned off, so[4], flag; };
__device__ __forceinline__ void unit_load(const u8* x, const u8* w, int piece, int u, int lane, Raw& d) {
  const int panel = piece >> 2, q = piece & 3, g = lane >> 4, c = lane & 15;
  const bool isx = panel < NPA;
  const rsrc_t rs = __builtin_amdgcn_make_buffer_rsrc((void*)(isx ? x : w), 0, isx ? M * KS : KS * N, 0x00020000);
  const unsigned vo = isx ? (unsigned)(c * KS + g * 16) : (unsigned)(g * 16 * N + 4 * c);
  const unsigned so = isx ? (unsigned)((panel * 256 + u * 32) * KS + q * 128) : (unsigned)((q * 128 + (u & 1) * 64) * N + (panel - NPA) * 256 + (u >> 1) * 64);
  const unsigned sa = isx ? 4u : (unsigned)N, sb = isx ? 64u : 4u * N, sc = isx ? 16u * KS : 8u * N;
#pragma unroll
  for (int j = 0; j < 16; ++j) d.r[j] = __builtin_amdgcn_raw_buffer_load_b32(rs, vo, so + (j & 3) * sa + ((j >> 2) & 1) * sb + (j >> 3) * sc, 0);
}
__device__ __forceinline__ void unit_pack(int piece, int u, int lane, const Raw& raw, Unit& t) {
  const int panel = piece >> 2, q = piece & 3;
  t.off = (unsigned)q * (8u * KT_BYTES) + (unsigned)panel * BLK + (unsigned)lane * 16u;
  t.flag = (unsigned)piece * 32u;
  t.so[0] = 0;
  if (panel < NPA) {
#pragma unroll
    for (int i = 0; i < 4; ++i) t.o[i] = (v4u){raw.r[i * 4], raw.r[i * 4 + 1], raw.r[i * 4 + 2], raw.r[i * 4 + 3]};
    t.off += u * 4096; t.so[1] = 1024; t.so[2] = 2048; t.so[3] = 3072;
  } else {
#pragma unroll
    for (int d = 0; d < 4; ++d) {
      const unsigned r0 = raw.r[4 * d], r1 = raw.r[4 * d + 1], r2 = raw.r[4 * d + 2], r3 = raw.r[4 * d + 3];
      const unsigned t0 = __builtin_amdgcn_perm(r1, r0, 0x05010400u), t1 = __builtin_amdgcn_perm(r1, r0, 0x07030602u);
      const unsigned t2 = __builtin_amdgcn_perm(r3, r2, 0x05010400u), t3 = __builtin_amdgcn_perm(r3, r2, 0x07030602u);
      t.o[0][d] = __builtin_amdgcn_perm(t2, t0, 0x05040100u); t.o[1][d] = __builtin_amdgcn_perm(t2, t0, 0x07060302u);
      t.o[2][d] = __builtin_amdgcn_perm(t3, t1, 0x05040100u); t.o[3][d] = __builtin_amdgcn_perm(t3, t1, 0x07060302u);
    }
    t.off += (u >> 1) * 8192 + (u & 1) * 1024; t.so[1] = 2048; t.so[2] = (u & 1) ? 4096 - 1024 : 4096 + 1024; t.so[3] = t.so[2] + 2048;
  }
}
template <int NW, int NAP> __device__ __forceinline__ bool wait_local(u8* myws, unsigned off, unsigned step, unsigned want, unsigned call) {
  gu32* const f = (gu32*)(myws + WS_FLAG);
  const unsigned t0 = (unsigned)__builtin_amdgcn_s_memrealtime();
  for (;;) {
    bool low = false;
#pragma unroll
    for (int i = 0; i < NW; ++i) low |= __hip_atomic_load(f + (off + i * step) / 4, RLX_SYS) < want;
    if ((unsigned)__builtin_amdgcn_readfirstlane(__hip_atomic_load(f + TMO_OFF / 4, RLX_SYS)) == call) return false;
    if (__builtin_amdgcn_ballot_w64(low) == 0) return true;
    if ((unsigned)__builtin_amdgcn_s_memrealtime() - t0 > DEADLINE) { __hip_atomic_store(f + TMO_OFF / 4, call, RLX_SYS); return false; }
    __builtin_amdgcn_s_sleep(NAP);
  }
}

__global__ __launch_bounds__(256, 1) void fused_kernel(const Params p) {
  __shared__ __attribute__((aligned(16))) u8 shm[LDS_BYTES];
  LAS u8* lds = (LAS u8*)shm;
  const int tid = threadIdx.x, wid = __builtin_amdgcn_readfirstlane(tid >> 6), lane = tid & 63;
  const bool comm = blockIdx.x < NCOMM;
  const int gw = blockIdx.x - NCOMM;
  const int n0 = comm ? wid * NCOMM + blockIdx.x : wid * NGEMM + gw, step = comm ? NDW : NGWAVES;
  const u8* px = p.x; const u8* pw = p.w; u8* peer[NR]; int pc[NDIRECT];
#pragma unroll
  for (int i = 0; i < NR; ++i) peer[i] = p.peer[i];
#pragma unroll
  for (int i = 0; i < NDIRECT; ++i) { const int n = n0 + i * step; pc[i] = (int)p.stream[n < NUNIT ? n >> 4 : 0]; }
  asm volatile("" : "+s"(px), "+s"(pw), "+s"(peer[0]), "+s"(peer[1]), "+s"(peer[2]), "+s"(peer[3]), "+s"(peer[4]), "+s"(peer[5]), "+s"(peer[6]), "+s"(peer[7]),
                    "+s"(pc[0]), "+s"(pc[1]), "+s"(pc[2]), "+s"(pc[3]));
#pragma unroll
  for (int i = 0; i < NDIRECT; ++i) pc[i] = (pc[i] >> ((((n0 + i * step) >> 3) & 1) * 16)) & 0xFFFF;
  u8* const myws = peer[NR - 1];
  gu32* const cnt = (gu32*)(myws + WS_CNT + blockIdx.x * 128) + wid;
  const unsigned cv = __hip_atomic_load(cnt, RLX_AGT);
  if (__builtin_expect(comm, 1)) {
    volatile LAS unsigned* const stage = (volatile LAS unsigned*)(lds + LDS_X);
    Raw raw[2];
    unit_load(px, pw, pc[0], n0 & 7, lane, raw[0]);
    unit_load(px, pw, pc[1], n0 & 7, lane, raw[1]);
    unsigned tab[3];
#pragma unroll
    for (int i = 0; i < 3; ++i) tab[i] = p.stream[i * 64 + lane < NPIECE / 2 ? i * 64 + lane : 0];
    if (tid == 0) *stage = 0;
    WAIT_L(0); FBAR;
    const unsigned call = (unsigned)__builtin_amdgcn_readfirstlane(cv) + 1u;
    if (lane == 0) __hip_atomic_store(cnt, call, RLX_AGT);
    if (wid >= DW) {
      if (blockIdx.x == 0 && lane == 0) __hip_atomic_store((gu32*)(myws + WS_FLAG + ROT_OFF), call * 8u + (__builtin_amdgcn_s_getreg(20 | (3 << 11)) & 7u), RLX_SYS);
      wait_local<NPIECE / 64, 32>(myws, lane * 32 + 7 * 4, 64 * 32, call * 8u, call);
      *stage = 1;
      wait_local<1, 32>(myws, SH_OFF, 0, call * (NSH * 4u), call);
      *stage = 2;
      return;
    }
    rsrc_t rs[NR];
#pragma unroll
    for (int i = 0; i < NR; ++i)
      rs[i] = __builtin_amdgcn_make_buffer_rsrc(peer[i] + WS_STG + (size_t)(call & 1u) * STG_BYTES + (unsigned)((6 - i) & 7) * KT_BYTES, 0, STG_BYTES - 7u * KT_BYTES, 0x00020000);
    u8* fl = peer[0];
#pragma unroll
    for (int i = 1; i < NR - 1; ++i) fl = lane == i ? peer[i] : fl;
    fl += WS_FLAG + (unsigned)((6 - lane) & 7) * 4;
    unsigned* const ticket = (unsigned*)(myws + WS_TICKET);
    constexpr int NSTAT = NDIRECT * NDW + NCOMM;
    const unsigned tbase = (call - 1u) * (unsigned)(NTOT - NSTAT + NCOMM) - (unsigned)NSTAT;
    const unsigned vl = (unsigned)lane * 16u;
    unsigned lstep = 1024u, sstep = 1024u;
#define DRAW() asm volatile("s_mov_b64 exec, 1\n\tglobal_atomic_add v255, %0, %1, %2 sc0\n\ts_mov_b64 exec, -1" :: "v"(0u), "v"(1u), "s"(ticket) : "v255", "memory")
#define TICKET() ({ unsigned _t; asm volatile("v_readfirstlane_b32 %0, v255" : "=s"(_t) :: "memory"); _t - tbase; })
#define PUBLISH(h) do { asm volatile("" ::: "memory"); __builtin_amdgcn_s_waitcnt(0x0F70 | ((h) & 15) | (((h) >> 4) << 14)); WAIT_V(h); \
      if (lane < NR - 1 && pflag != ~0u) __hip_atomic_fetch_add((gu32*)(fl + pflag), 1u, RLX_SYS); } while (0)
#define AWAIT(v) do { while (*stage < (v)) __builtin_amdgcn_s_sleep(1); } while (0)
    auto locate = [&](unsigned n, unsigned& off, unsigned& flag) {
      const unsigned pi = n >> 3, w = pi >> 1;
      const unsigned a = __builtin_amdgcn_readlane(tab[0], w & 63), b = __builtin_amdgcn_readlane(tab[1], w & 63), c = __builtin_amdgcn_readlane(tab[2], w & 63);
      const unsigned e = ((w < 64 ? a : w < 128 ? b : c) >> ((pi & 1) * 16)) & 0xFFFFu;
      off = n >= (unsigned)NUNIT ? ((n - NUNIT) / NSH) * (N * 16u) + ((n - NUNIT) % NSH) * 1024u : (e & 3) * (8u * KT_BYTES) + (e >> 2) * BLK + (n & 7) * 4096u;
      flag = n >= (unsigned)NUNIT ? OUT_OFF : e * 32u;
    };
    unsigned n = (unsigned)(n0 + NDIRECT * NDW), off, flag, pflag = ~0u;
    v4u d[4];
#pragma unroll
    for (int i = 0; i < NDIRECT; ++i) {
      Unit t;
      unit_pack(pc[i], n0 & 7, lane, raw[i & 1], t);
      asm volatile("" : "+v"(t.o[0]), "+v"(t.o[1]), "+v"(t.o[2]), "+v"(t.o[3]));
      if (i > 0 && i + 1 < NDIRECT) unit_load(px, pw, pc[i + 1], n0 & 7, lane, raw[(i + 1) & 1]);
#pragma unroll
      for (int k = 0; k < 28; ++k) { const int q = k % 7, j = (k / 7 + q) & 3; __builtin_amdgcn_raw_buffer_store_b128(t.o[j], rs[q], t.off, t.so[j], 17); }
      if (i == NDIRECT - 1 && wid == 0) {
        AWAIT(1);
        locate(n, off, flag);
#pragma unroll
        for (int k = 0; k < 4; ++k) d[k] = __builtin_amdgcn_raw_buffer_load_b128(rs[7], vl, off + k * 1024, 17);
        DRAW();
      }
      PUBLISH(28); pflag = t.flag;
    }
    PUBLISH(0); pflag = ~0u;
    if (wid != 0) return;
    for (;;) {
      const v4u o[4] = {d[0], d[1], d[2], d[3]};
      const unsigned coff = off, cflag = flag;
      const unsigned nn = TICKET();
      const bool more = nn < (unsigned)NTOT, sw = n < (unsigned)NUNIT && nn >= (unsigned)NUNIT;
      if (more) {
        if (sw) { AWAIT(2); rs[7] = __builtin_amdgcn_make_buffer_rsrc(p.share[7], 0, SH_BYTES, 0x00020000); lstep = N * 4u; }
        locate(nn, off, flag);
#pragma unroll
        for (int k = 0; k < 4; ++k) d[k] = __builtin_amdgcn_raw_buffer_load_b128(rs[7], vl, off + k * lstep, 17);
        DRAW();
      }
#pragma unroll
      for (int k = 0; k < 28; ++k) { const int q = k % 7, j = (k / 7 + q) & 3; __builtin_amdgcn_raw_buffer_store_b128(o[j], rs[q], vl, coff + j * sstep, 17); }
      PUBLISH(28);
      pflag = cflag;
      if (!more) break;
      if (sw) {
        sstep = N * 4u;
#pragma unroll
        for (int i = 0; i < NR - 1; ++i) rs[i] = __builtin_amdgcn_make_buffer_rsrc(p.share[i], 0, SH_BYTES, 0x00020000);
      }
      n = nn;
    }
    PUBLISH(0);
    return;
  }
  unsigned call;
  {
    Raw raw[3];
#pragma unroll
    for (int i = 0; i < 3; ++i) unit_load(px, pw, pc[i], n0 & 7, lane, raw[i]);
    call = (unsigned)__builtin_amdgcn_readfirstlane(cv) + 1u;
    if (lane == 0) __hip_atomic_store(cnt, call, RLX_AGT);
    const rsrc_t rl = __builtin_amdgcn_make_buffer_rsrc(myws + WS_STG + (size_t)(call & 1u) * STG_BYTES + 7u * KT_BYTES, 0, STG_BYTES - 7u * KT_BYTES, 0x00020000);
    unsigned fo = 0;
#pragma unroll
    for (int i = 0; i < 3; ++i) {
      if (n0 + i * NGWAVES < NUNIT) {
        Unit t; unit_pack(pc[i], n0 & 7, lane, raw[i], t);
#pragma unroll
        for (int k = 0; k < 4; ++k) __builtin_amdgcn_raw_buffer_store_b128(t.o[k], rl, t.off, t.so[k], 17);
        fo = lane == i ? t.flag : fo;
      }
    }
    WAIT_V(0);
    if (lane < 3 && n0 + lane * NGWAVES < NUNIT) __hip_atomic_fetch_add((gu32*)(myws + WS_FLAG + fo + 7 * 4), 1u, RLX_SYS);
    if (wid == 0) __builtin_amdgcn_fence(__ATOMIC_ACQUIRE, "");
    if (tid == 0) *(LAS float*)(lds + LDS_X + 32) = p.sx[0] * p.sw[0];
    asm volatile("s_waitcnt vmcnt(0) lgkmcnt(0)" ::: "memory");
  }
  const int wr = wid >> 1, wc = wid & 1;
  const rsrc_t rs = __builtin_amdgcn_make_buffer_rsrc(myws + WS_STG + (size_t)(call & 1u) * STG_BYTES, 0, STG_BYTES, 0x00020000);
  const rsrc_t rf = __builtin_amdgcn_make_buffer_rsrc(myws + WS_FLAG, 0, FLAG_BYTES, 0x00020000);
  const unsigned need = call * 8u;
  const unsigned lbase = (unsigned)(size_t)lds;
  const unsigned rinit = wid ? 0xFFFFFFFFu : 0u;
  wait_local<1, 2>(myws, ROT_OFF, 0, call * 8u, call);
  const int role = (gw & ~7) | ((gw + __builtin_amdgcn_readfirstlane(__hip_atomic_load((gu32*)(myws + WS_FLAG + ROT_OFF), RLX_SYS))) & 7);
  unsigned long long tlo = p.tile[role][0] | ((unsigned long long)p.tile[role][1] << 32); unsigned thi = p.tile[role][2];
  asm volatile("v_mov_b32 v%c0, %5\n\tv_mov_b32 v%c1, %6\n\tv_mov_b32 v%c2, %7\n\tv_mov_b32 v%c3, %8\n\tv_mov_b32 v%c4, %9\n\tv_mov_b32 v%c10, %11"
               :: "n"(V_ARD), "n"(V_BRD), "n"(V_XRD), "n"(V_VL), "n"(V_CO),
                  "v"(lbase + LDS_A + wr * 16384 + lane * 16), "v"(lbase + LDS_B + wc * 8192 + lane * 16), "v"(lbase + LDS_X), "v"(lane * 16),
                  "v"((4 * (lane >> 4) * N + 4 * (lane & 15)) * 4), "n"(V_XWR), "v"(lbase + LDS_X + wid * 4) : OWNED_REGS, "memory");
  const unsigned mA = lbase + wid * 8192, mB = lbase + wid * 4096;
#define M0_A(slot, h) set_m0<LDS_A + (slot) * BLK + (h) * 4096>(mA)
#define M0_B(slot, h) set_m0<LDS_B + (slot) * BLK + (h) * 16384>(mB)
#define SRC_A(t) ((unsigned)(((t) & 15) * BLK + wid * 8192))
#define SRC_B(t) ((unsigned)((NPA + ((t) >> 4)) * BLK + wid * 4096))
#define ENSURE(cand) do { if (__builtin_expect((cand) == 0, 0)) { \
      const unsigned _t0 = (unsigned)__builtin_amdgcn_s_memrealtime(); \
      for (;;) { unsigned _w; \
        poll(rf, 0u); asm volatile("buffer_load_dword v%c0, off, %1, %2 sc0 sc1" :: "n"(V_TMP), "s"(rf), "s"(TMO_OFF) : "memory"); \
        rdy = poll_mask<0>(need); cand = rdy & ~done; \
        if (cand) break; \
        asm volatile("v_readfirstlane_b32 %0, v%c1" : "=s"(_w) : "n"(V_TMP)); \
        if (_w == call || (unsigned)__builtin_amdgcn_s_memrealtime() - _t0 > DEADLINE) { \
          asm volatile("v_mov_b32 v%c0, %1\n\tbuffer_store_dword v%c0, off, %2, %3 sc0 sc1\n\ts_nop 1" :: "n"(V_TMP), "s"(call), "s"(rf), "s"(TMO_OFF) : "memory"); \
          x_write<32>(0x7FC00000u); cand = ~done; break; } \
        __builtin_amdgcn_s_sleep(2); } \
      __builtin_amdgcn_s_waitcnt(0xC07F); } } while (0)
#define POLL_OFF (rdy == 0xFFFFFFFFu ? 0x40000u : 0u)

  int cur = (int)(tlo & 0xFFFFu);
  unsigned done = 0, rdy = rinit;
  unsigned sA = SRC_A(cur), sB = SRC_B(cur);
  unsigned bsrc;
  aim_poll(cur);
  {
    unsigned c_ = rdy; ENSURE(c_); x_write<0>(pick(c_, done));
    c_ = rdy & ~done; ENSURE(c_); x_write<16>(pick(c_, done));
    WAIT_L(0); FBAR;
    x_read<0>(); const unsigned k0 = x_value<0>();
    x_read<16>(); const unsigned k1 = x_value<0>();
    poll(rf, POLL_OFF);
#define DMA4(m0v, so) do { m0v; asm volatile("s_nop 0"); dma<0>(rs, so); dma<1>(rs, so); dma<2>(rs, so); dma<3>(rs, so); } while (0)
    DMA4(M0_A(0, 0), sA + k0); DMA4(M0_A(0, 1), sA + k0 + 4096); DMA4(M0_B(0, 0), sB + k0); DMA4(M0_B(0, 1), sB + k0 + 16384);
    DMA4(M0_A(1, 0), sA + k1); DMA4(M0_A(1, 1), sA + k1 + 4096); DMA4(M0_B(1, 0), sB + k1);
    bsrc = sB + k1;
    WAIT_V(0); FBAR;
    ld_b<0, 0, 0>(); ld_a<0, 0>(); ld_a<1, 0>(); ld_b<1, 0, 1>(); ld_a<2, 0>(); ld_a<3, 0>(); ld_a<4, 0>(); ld_a<5, 0>(); ld_a<6, 0>(); ld_a<7, 0>();
  }
#define KTILE(par, MODE, PREV, SW) do { \
    unsigned asrc, cand, jp; \
    asm volatile("" ::: OWNED_REGS);        \
    static_for<0, 64>([&](auto gsc) __attribute__((always_inline)) { \
      constexpr int gs = decltype(gsc)::value, np = gs >> 4, s = gs & 15, m = s >> 1, e = s & 1; \
      constexpr int cb = (np & 1) * 2, nb = 2 - cb, z = e ^ (m & 1); \
      if constexpr (gs == 16) { wait_v<nvm(MODE, PREV, dma_slot(15) - 128, 16)>(); wait_l<0>(); BAR; } \
      if constexpr (gs == 48) { wait_v<nvm(MODE, PREV, dma_slot(11) - 64, 48)>(); wait_l<0>(); BAR; } \
      if constexpr (gs <= 1) wait_l<14>(); \
      if constexpr (gs == 4 || gs == 6) wait_l<12>(); \
      if constexpr (gs == 8) wait_l<10>(); \
      if constexpr (gs == 10 || gs == 12 || gs == 14) wait_l<19 - gs>(); \
      if constexpr (gs == 32 || gs == 33) wait_l<2>(); \
      mfma<m, np * 2 + z, cb + z, (MODE) == 2>(); SCHED; \
      if constexpr (gs == 16) x_read<(par) * 16>(); \
      if constexpr (gs == 20) { const unsigned ko = x_value<2>(); bsrc = sB + ko; asrc = sA + ko; } \
      if constexpr ((MODE) == 1 && gs == 20) x_read<32>(); \
      if constexpr ((MODE) == 1 && gs == 22) scale = x_value<2>(); \
      if constexpr (s == 0 || s == 4) { if constexpr (np < 3) ld_b<nb + (s >> 2), par, np * 2 + 2 + (s >> 2)>(); else ld_b<nb + (s >> 2), (par) ^ 1, (s >> 2)>(); } \
      if constexpr (np == 3 && e == 1) ld_a<m, (par) ^ 1>(); \
      if constexpr (gs == 1) rdy |= poll_mask<nvm(MODE, PREV, POLL_SLOT - 64, 1)>(need); \
      if constexpr (gs == 3) { cand = rdy & ~done; ENSURE(cand); } \
      if constexpr (gs == 7) jp = pick(cand, done); \
      if constexpr (gs == 9) x_write<(par) * 16>(jp); \
      if constexpr (gs == 11 && (SW)) { done = 0; rdy = rinit; aim_poll(nxt); } \
      if constexpr (gs == POLL_SLOT) poll(rf, POLL_OFF); \
      if constexpr (dma_at((gs + 1) % 64) >= 0 && dma_at((gs + 1) % 64) % 4 == 0) { constexpr int k = dma_at((gs + 1) % 64), sl = dma_slot(k) > 64 ? (par) ^ 1 : (par); \
        if constexpr (k < 8) M0_A(sl, k >> 2); else M0_B(sl, (k >> 2) - 2); } \
      if constexpr (dma_at(gs) >= 0) { constexpr int k = dma_at(gs); dma<k & 3>(rs, k < 8 ? asrc + (k >> 2) * 4096 : bsrc + ((k >> 2) - 2) * 16384); } \
      if constexpr (gs == 29 && (SW)) { sA = SRC_A(nxt); sB = SRC_B(nxt); } \
      if constexpr (g0_mx(MODE, gs) >= 0) q_g0<g0_mx(MODE, gs) & 1, g0_mx(MODE, gs)>(scale); \
      if constexpr (g0_st(MODE, gs) >= 0) q_store<g0_st(MODE, gs) & 1, g0_st(MODE, gs), 0, false>(rc); \
      if constexpr (park_frag_at(MODE, gs) >= 0) park_frag<park_frag_at(MODE, gs)>(); \
      if constexpr (g1_op(MODE, gs, 0) >= 0) q_read<g1_op(MODE, gs, 0) & 1, A_PARK, g1_op(MODE, gs, 0)>(); \
      if constexpr (g1_op(MODE, gs, 1) >= 0) q_scale<g1_op(MODE, gs, 1) & 1>(scale); \
      if constexpr (g1_op(MODE, gs, 2) >= 0) q_clamp<g1_op(MODE, gs, 2) & 1>(); \
      if constexpr (g1_op(MODE, gs, 3) >= 0) q_store<g1_op(MODE, gs, 3) & 1, g1_op(MODE, gs, 3), 1, false>(rc); \
      SCHED; \
    }); } while (0)
  unsigned scale = 0;
  rsrc_t rc = __builtin_amdgcn_make_buffer_rsrc(p.out, 0, 0, 0x00020000);
  for (int r = 0;; ++r) {
    tlo = (tlo >> 16) | ((unsigned long long)thi << 48); thi = (thi >> 16) | 0xFFFF0000u;
    const int nx = (int)(tlo & 0xFFFFu);
    const bool has_next = nx != 0xFFFF;
    const int nxt = has_next ? nx : cur;
    KTILE(0, 2, 1, false);
    static_for<1, ND + 2>([&](auto tc) __attribute__((always_inline)) { constexpr int t = decltype(tc)::value; KTILE(t & 1, (t <= ND ? 2 + t : 0), 1 + t, false); });
    for (int t = ND + 2; t < NKT - 4; t += 2) {
      KTILE(0, 0, 0, false);
      KTILE(1, 0, 0, false);
    }
    KTILE(0, 0, 0, false);
    KTILE(1, 0, 0, true);
    KTILE(0, 0, 0, false);
    const bool shared = (cur & 15) >= 8 && (unsigned)((cur >> 4) - SC) < (unsigned)NSH;
    float* const cp = p.out + (size_t)((cur & 15) * 256 + wr * 128) * N + (cur >> 4) * 256 + wc * 64;
    constexpr unsigned WAVE_SPAN = 127u * N * 4u + 768u;
    rc = __builtin_amdgcn_make_buffer_rsrc(cp, 0, shared ? 0 : WAVE_SPAN, 0x00020000);
    KTILE(1, 1, 0, false);
    if (shared) {
      drain_now<0, true>(__builtin_amdgcn_make_buffer_rsrc(cp, 0, WAVE_SPAN, 0x00020000), scale);
      asm volatile("v_mov_b32 v%c0, 1\n\ts_waitcnt vmcnt(0)\n\ts_mov_b64 exec, 1\n\tbuffer_atomic_add v%c0, off, %1, %2 sc1\n\ts_mov_b64 exec, -1" :: "n"(V_TMP), "s"(rf), "s"(SH_OFF) : "memory");
    }
    if (!has_next) break;
    cur = nxt;
  }
  drain_now<SL.tail, false>(rc, scale);
  int ln;
  asm volatile("s_waitcnt vmcnt(0) ; end of owned registers\n\tv_mbcnt_lo_u32_b32 %0, -1, 0\n\tv_mbcnt_hi_u32_b32 %0, -1, %0" : "=v"(ln) :: OWNED_REGS, "memory");
  if (gw == WAITER && wid == 0) {
    const bool ok = wait_local<1, 2>(myws, OUT_OFF + (ln < NR - 1 ? ln : 0) * 4, 0, call * (unsigned)NOUT, call);
    __builtin_amdgcn_fence(__ATOMIC_ACQUIRE, "");
    if (!ok && ln == 0) p.out[(size_t)(cur & 15) * 256 * N + (cur >> 4) * 256] = __builtin_nanf("");
  }
}

static void build_schedule(Params& p, int rank) {
  constexpr int W = NGEMM / 8, PR = 2, PC = 3, C0 = 18, SHIFT = 12;
  for (int x = 0; x < 8; ++x) {
    const int xr = x >> 1, xc = (x & 1) ^ 1;
    u16 list[128]; int n = 0;
    for (int pass = 0; pass < 2; ++pass)
      for (int ci = 0; ci < 32; ++ci) for (int ri = 0; ri < 4; ++ri) {
        const int pm = xr + 4 * ri, pn = 2 * ci + xc;
        if ((ri < PR && ci < PC) != (pass == 0)) continue;
        if (pm >= 8 && pn >= SC && pn < SC + NSH) {
          if (pn == SC && ri == 2 && xr < NSH) list[n++] = (u16)((8 + rank) | (SC + xr) << 4);
          continue;
        }
        list[n++] = (u16)(pm | pn << 4);
      }
    const int extra = n - 4 * W;
    for (int c = 0; c < W; ++c) {
      int k = 0; u16* t = (u16*)p.tile[c * 8 + x];
      t[k++] = list[c];
      if (c < extra) t[k++] = list[W + c];
      for (int r = 0; r < 3; ++r) t[k++] = list[W + extra + r * W + (c + W - extra) % W];
      while (k <= MAXT) t[k++] = 0xFFFF;
    }
  }
  int s = 0;
  auto put = [&](int panel, int q) { ((u16*)p.stream)[s++] = (u16)(panel << 2 | q); };
  for (int q = 0; q < 4; ++q) for (int i = 0; i < 16; ++i) { if (i < 4 * PR) put(i, q); if (i < 2 * PC) put(NPA + i, q); }
  for (int q = 0; q < 4; ++q) { for (int i = 4 * PR; i < 16; ++i) put(i, q); for (int i = 2 * PC; i < C0; ++i) put(NPA + i, q); }
  const int lim[4] = {C0, 34, 50, 64};
  for (int r = 0; r < 3; ++r) for (int q = 0; q < 4; ++q) for (int i = lim[r]; i < lim[r + 1]; ++i) put(NPA + i, q);
  u16 a[NPIECE]; u16* st = (u16*)p.stream; constexpr int first = 4 * (4 * PR + 2 * PC), shift = SHIFT;
  auto late = [&](int i) { return i >= first && (a[i] >> 2) >= NPA && ((a[i] >> 2) & 1); };
  for (int i = 0; i < NPIECE; ++i) a[i] = st[i];
  s = 0;
  for (int i = 0; i < NPIECE + shift; ++i) {
    if (i < NPIECE && !late(i)) st[s++] = a[i];
    if (i >= shift && late(i - shift)) st[s++] = a[i - shift];
  }
}

extern "C" void kernel_launch(int rank, int nranks, ncclComm_t comm, hipStream_t stream,
                              void* const* d_in, const int* in_sizes, int n_in,
                              void* d_out, int out_size, const int* in_dtypes, int out_dtype,
                              void* const* d_out_all, void* const* d_ws_all, size_t ws_bytes) {
  if (nranks != NR || n_in != 4 || ws_bytes < WS_TOTAL || (unsigned)rank >= (unsigned)NR ||
      in_sizes[0] != M * KS || in_sizes[1] != KS * N || out_size != M * N) { fprintf(stderr, "unexpected problem shape\n"); return; }
  Params p;
  p.x = (const u8*)d_in[0]; p.w = (const u8*)d_in[1]; p.sx = (const float*)d_in[2]; p.sw = (const float*)d_in[3];
  p.out = (float*)d_out;
  for (int i = 0; i < NR; ++i) {
    p.peer[i] = (u8*)d_ws_all[(rank + 1 + i) % NR];
    p.share[i] = (float*)d_out_all[(rank + 1 + i) % NR] + (size_t)(8 + rank) * 256 * N + SC * 256;
  }
  build_schedule(p, rank);
  hipLaunchKernelGGL(fused_kernel, dim3(GRID), dim3(256), 0, stream, p);
  HIP_CHECK(hipGetLastError());
}
```
